```python
import numpy as np

kernel_name = "gpu_gemm_rs_m4096_k4096_n16384_e4m3e5m2_dqsilu"


ABSMAX_RTOL = 0.05

COMPUTE_DTYPE = "float32"
EPILOGUE = "dequant_silu"
IN_QUANT = None
OUT_QUANT = None
IN_FP8 = "e4m3e5m2"
IN_DTYPES = {"x": "e4m3", "w_mat": "e5m2",
             "scale_x": "float32", "scale_w": "float32"}
WORLD_SIZE = 8

def _snap_fp8(x, mant_bits, max_abs, min_normal):
    a = np.abs(x.astype(np.float64))
    _m, _e = np.frexp(a)
    q = np.where(a >= min_normal, np.ldexp(1.0, _e - 1 - mant_bits),
                 min_normal * 2.0 ** -mant_bits)
    snapped = np.minimum(np.round(a / q) * q, max_abs)
    return (np.sign(x) * snapped).astype(np.float32)

def setup_inputs():

    x = _snap_fp8(np.random.randn(4096, 4096).astype(np.float32), 3, 448.0, 2.0 ** -6)
    w_mat = _snap_fp8(np.random.randn(4096, 16384).astype(np.float32), 2, 57344.0, 2.0 ** -14)
    scale_x = (0.02 * (1.0 + np.random.rand(1))).astype(np.float32)
    scale_w = (0.02 * (1.0 + np.random.rand(1))).astype(np.float32)
    return {"x": x, "w_mat": w_mat, "scale_x": scale_x, "scale_w": scale_w}

def reference(x, w_mat, scale_x, scale_w):
    acc = (x @ w_mat).astype(np.float32)
    y = (acc * (scale_x * scale_w)).astype(np.float32)
    return (y / (1.0 + np.exp(-np.clip(y, -60.0, 60.0)))).astype(np.float32)
```

```cpp
#include <hip/hip_runtime.h>
#include <rccl/rccl.h>
#include <cstdio>
#include <cstdint>

#define HIP_CHECK(expr) do { hipError_t _e = (expr); if (_e != hipSuccess) { \
    fprintf(stderr, "HIP error %s at %s:%d\n", hipGetErrorString(_e), __FILE__, __LINE__); \
    return; } } while (0)

namespace {

typedef unsigned u32;
typedef unsigned long long u64;
typedef unsigned u4 __attribute__((ext_vector_type(4)));
typedef int v2i __attribute__((ext_vector_type(2)));
typedef int v4i __attribute__((ext_vector_type(4)));
typedef int v8i __attribute__((ext_vector_type(8)));
typedef float v4f __attribute__((ext_vector_type(4)));
#define LDS_AS __attribute__((address_space(3)))
#define GLB_AS __attribute__((address_space(1)))

constexpr int NR = 8;
constexpr int MG = 4096, KG = 4096, NG = 16384;
constexpr int KS = KG / NR;
constexpr u32 PANEL = 128 * KS;
constexpr int NA = 16;
constexpr int NB = 32;
#define NREL_CFG 16
constexpr int NREL = NREL_CFG;
constexpr int NDIR = NB - NREL;
static_assert(NREL % 2 == 0 && NREL >= 0 && NREL <= NB, "NREL");

constexpr int WPL = 8;
constexpr int NCOMM = 7 * WPL;
constexpr int GRID = 256;
constexpr int NGEMM = GRID - NCOMM;
constexpr int GPX = NGEMM / 8;
constexpr int THREADS = 512;
static_assert(NCOMM % 8 == 0 && NGEMM % 8 == 0, "roles");

constexpr int NP_W = NDIR + NREL / 2;
constexpr int NP_UNITS = NA + NP_W;
constexpr int PT_UNITS = NA + NB + 3 * NREL;
constexpr int MAXT = (PT_UNITS * 4 + WPL - 1) / WPL;
static_assert((NP_UNITS * 4) % WPL == 0 && (PT_UNITS * 4) % WPL == 0, "pieces");
static_assert(MAXT <= 128, "descriptor table");

constexpr u32 OFF_CTR = 0, OFF_TMO = 128;
constexpr u32 OFF_FX = 4096;
constexpr u32 OFF_FW = 8192;
constexpr u32 OFF_FO = 16384;
constexpr u32 OFF_DBG = 512 << 10;
constexpr u32 OFF_DATA = 1 << 20;
constexpr u32 REL_XG = 0;
constexpr u32 REL_WG = REL_XG + NA * 8 * PANEL;
constexpr u32 REL_ST = REL_WG + NB * 8 * PANEL;
constexpr u32 PAR_BYTES = REL_ST + 12 * NB * 65536;
constexpr u32 WS_NEED = OFF_DATA + 2 * PAR_BYTES;

constexpr u32 X_BYTES = MG * KS;
constexpr u32 W_BYTES = KS * NG;
constexpr u32 OUT_BYTES = (MG / NR) * NG * 4;

constexpr int STAGE = 32768, NST = 4;
constexpr int LDS_CTRL = NST * STAGE;
constexpr int LDS_BYTES = LDS_CTRL + 256;

constexpr u64 DEADLINE = 50000000ull;

constexpr int AUX_SYS = 17;
constexpr int AUX_DEV = 16;

struct Params {
  const char* x;
  const char* w;
  const float* sx;
  const float* sw;
  char* out;
  char* ws[NR];
  int rank;
};

__device__ __forceinline__ u32 rfl(u32 v) { return __builtin_amdgcn_readfirstlane(v); }
__device__ __forceinline__ __amdgpu_buffer_rsrc_t mk_rsrc(const void* p, u32 bytes) {
  return __builtin_amdgcn_make_buffer_rsrc((void*)p, (short)0, (int)bytes, 0x00020000);
}
__device__ __forceinline__ u32 ld_sys(const char* p) {
  return __hip_atomic_load((const GLB_AS u32*)p, __ATOMIC_RELAXED, __HIP_MEMORY_SCOPE_SYSTEM);
}
__device__ __forceinline__ void st_sys(char* p, u32 v) {
  __hip_atomic_store((GLB_AS u32*)p, v, __ATOMIC_RELAXED, __HIP_MEMORY_SCOPE_SYSTEM);
}
__device__ __forceinline__ u32 ld_dev(const char* p) {
  return __hip_atomic_load((const GLB_AS u32*)p, __ATOMIC_RELAXED, __HIP_MEMORY_SCOPE_AGENT);
}
__device__ __forceinline__ void st_dev(char* p, u32 v) {
  __hip_atomic_store((GLB_AS u32*)p, v, __ATOMIC_RELAXED, __HIP_MEMORY_SCOPE_AGENT);
}
#define VMCNT(n) asm volatile("s_waitcnt vmcnt(" #n ")" ::: "memory")
#define RAW_BARRIER() do { asm volatile("s_waitcnt lgkmcnt(0)" ::: "memory"); __builtin_amdgcn_s_barrier(); asm volatile("" ::: "memory"); } while (0)

__device__ __forceinline__ bool give_up(char* myws, u32 seq, u64 t0, int lane) {
  u32 t = rfl(ld_dev(myws + OFF_TMO));
  bool late = (__builtin_amdgcn_s_memrealtime() - t0) > DEADLINE;
  if (t == seq) return true;
  if (late) {
    if (lane == 0) st_dev(myws + OFF_TMO, seq);
    return true;
  }
  return false;
}

#define STAMP(slot) do {} while (0)

__device__ __forceinline__ int grow_of(int p, int a) {
  if (a < 12) return ((p ^ (2 + (a >> 1))) << 2) + ((p & 1) << 1) + (a & 1);
  return (p << 2) + (a - 12);
}

__device__ __forceinline__ void comm_role(const Params& P, char* smem, u32 seq, u32 parbase, u64 t0) {
  const int tid = threadIdx.x, lane = tid & 63;
  const int wave = rfl(tid >> 6);
  const int r = P.rank;
  const int d = 1 + (int)blockIdx.x % 7;
  const int w = (int)blockIdx.x / 7;
  const u32 p = (u32)(r ^ d) & 7u;
  char* myws = P.ws[r];
  char* pws = P.ws[p];
  const int T = (d == 1 ? PT_UNITS : NP_UNITS) * 4 / WPL;

  u4* desc = (u4*)smem;
  u32* cnt = (u32*)(smem + 4096);
  if (tid < T) {
    const int tg = w + WPL * tid, pos = tg >> 2, piece = tg & 3;
    int kind, idx, q = r;
    if (pos < NA) { kind = 0; idx = pos; }
    else if (d == 1) {
      int j = pos - NA;
      if (j < NDIR) { kind = 1; idx = j; }
      else {
        j -= NDIR;
        const int k = j >> 2, i = j & 3;
        idx = NDIR + k;
        if (i == 0) kind = 1;
        else { kind = 2; q = r ^ (2 * i + (k & 1)); }
      }
    } else {
      const int j = pos - NA;
      constexpr int NRL = NREL / 2;
      const int r0 = (j * NRL) / NP_W, r1 = ((j + 1) * NRL) / NP_W;
      kind = 1;
      if (r1 > r0) idx = NDIR + 2 * r0 + (d & 1);
      else idx = j - r0;
    }
    u4 e;
    if (kind == 0) {
      e.x = (u32)grow_of((int)p, idx) * PANEL + piece * 16384u;
      e.y = REL_XG + (u32)(idx * 8 + r) * PANEL + piece * 16384u;
      e.z = OFF_FX + (u32)((idx * 8 + r) * 4 + piece) * 4u;
    } else {
      e.y = REL_WG + (u32)(idx * 8 + q) * PANEL + piece * 16384u;
      e.z = OFF_FW + (u32)((idx * 8 + q) * 4 + piece) * 4u;
      e.x = kind == 1 ? (u32)piece * 128u * NG + ((p >> 1) * NB + (u32)idx) * 128u : e.y;
    }
    e.w = (u32)kind;
    desc[tid] = e;
    cnt[tid] = 0;
  }
  __syncthreads();

  const __amdgpu_buffer_rsrc_t rs_dst = mk_rsrc(pws, WS_NEED);
  const u32 v_lin = (u32)wave * 2048u + (u32)lane * 16u;
  const u32 v_str = (u32)(wave * 16 + (lane >> 3)) * NG + (u32)(lane & 7) * 16u;
  bool bad = false;
  int ready_upto = 0;
  u32 prev_flag = 0;
  if (wave == 0) STAMP(0);

  auto arrive = [&](int t, u32 flag_off) {
    u32 old = 0;
    if (lane == 0) old = __hip_atomic_fetch_add((LDS_AS u32*)(cnt + t), 1u, __ATOMIC_RELAXED, __HIP_MEMORY_SCOPE_WORKGROUP);
    old = rfl(old);
    if (old == 7u && lane == 0) st_sys(pws + flag_off, seq);
  };

  int t = 0;
  for (; t < T; ++t) {
    bool arrived_prev = false;
    if (t >= ready_upto) {
      for (;;) {
        const int ti = t + lane;
        bool ok = true;
        if (ti < T) {
          const u4 e = desc[ti];
          if (e.w == 2u) ok = ld_sys(myws + e.z) >= seq;
        }
        const u64 m = __ballot(ok);
        const int n = (~m == 0ull) ? 64 : __builtin_ctzll(~m);
        if (n > 0) { ready_upto = t + n; break; }
        if (t > 0 && !arrived_prev) { VMCNT(0); arrive(t - 1, prev_flag); arrived_prev = true; }
        if (give_up(myws, seq, t0, lane)) { bad = true; break; }
        __builtin_amdgcn_s_sleep(4);
      }
      asm volatile("" ::: "memory");
      if (bad) { if (arrived_prev) prev_flag = 0xffffffffu; break; }
    }
    const u4 e = desc[t];
    const u32 so = rfl(e.x), dof = rfl(e.y), fo = rfl(e.z), sel = rfl(e.w);
    const char* sbase = sel == 0u ? P.x : (sel == 1u ? P.w : (const char*)myws);
    const u32 sbytes = sel == 0u ? X_BYTES : (sel == 1u ? W_BYTES : WS_NEED);
    const u32 soff = sel == 2u ? so + parbase : so;
    const __amdgpu_buffer_rsrc_t rs_src = mk_rsrc(sbase, sbytes);
    const u32 v0 = sel == 1u ? v_str : v_lin;
    const u32 v1 = sel == 1u ? v_str + 8u * NG : v_lin + 1024u;
    const u4 a0 = __builtin_amdgcn_raw_buffer_load_b128(rs_src, v0, soff, AUX_SYS);
    const u4 a1 = __builtin_amdgcn_raw_buffer_load_b128(rs_src, v1, soff, AUX_SYS);
    VMCNT(0);
    __builtin_amdgcn_raw_buffer_store_b128(a0, rs_dst, v_lin, dof + parbase, AUX_SYS);
    __builtin_amdgcn_raw_buffer_store_b128(a1, rs_dst, v_lin + 1024u, dof + parbase, AUX_SYS);
    if (t > 0 && !arrived_prev) arrive(t - 1, prev_flag);
    prev_flag = fo;
  }
  VMCNT(0);
  if (t > 0 && prev_flag != 0xffffffffu) arrive(t - 1, prev_flag);
  if (wave == 0) STAMP(1);
  if (d == 1) return;

  const __amdgpu_buffer_rsrc_t rs_out = mk_rsrc(P.out, OUT_BYTES);
  const int wl = w * 8 + wave;
  const u32 v_src = (u32)wl * 1024u + (u32)lane * 16u;
  const u32 v_out = (u32)(wl * 2 + (lane >> 5)) * (NG * 4u) + (u32)(lane & 31) * 16u;
  const char* fo = myws + OFF_FO + (u32)d * 256u + (u32)lane * 4u;
  u64 todo = ~0ull, ready = 0ull;
  bool dead = false;
  while (todo) {
    u64 avail = ready & todo;
    if (!avail) {
      ready = __ballot(ld_sys(fo) >= seq);
      avail = ready & todo;
      if (!avail) {
        if (give_up(myws, seq, t0, lane)) { dead = true; ready = ~0ull; avail = todo; }
        else { __builtin_amdgcn_s_sleep(4); continue; }
      }
      asm volatile("" ::: "memory");
    }
    const int i = __builtin_ctzll(avail);
    todo &= ~(1ull << i);
    const int b = i >> 1, tt = i & 1;
    const u32 s_src = parbase + REL_ST + (u32)(((d - 2) * 2 + tt) * NB + b) * 65536u;
    const u32 s_out = (u32)((2 * (p & 1) + tt) * 128) * (NG * 4u) + ((p >> 1) * NB + (u32)b) * 512u;
    u4 v;
    if (!dead) v = __builtin_amdgcn_raw_buffer_load_b128(rs_dst, v_src, s_src, AUX_SYS);
    else v = u4{0x7fc00000u, 0x7fc00000u, 0x7fc00000u, 0x7fc00000u};
    __builtin_amdgcn_raw_buffer_store_b128(v, rs_out, v_out, s_out, AUX_DEV);
  }
  if (wave == 0) STAMP(3);
}

__device__ __forceinline__ void glds16(__amdgpu_buffer_rsrc_t rs, u32 lds, u32 voff, u32 soff) {
  asm volatile("s_mov_b32 m0, %0\n\ts_nop 3\n\tbuffer_load_dwordx4 %1, %2, %3 offen lds"
               :: "s"(rfl(lds)), "v"(voff), "s"(rs), "s"(rfl(soff)) : "m0", "memory");
}

__device__ __forceinline__ void gemm_role(const Params& P, char* smem, u32 seq, u32 parbase, u64 t0) {
  const int tid = threadIdx.x, lane = tid & 63;
  const int wave = rfl(tid >> 6);
  const int r = P.rank;
  char* myws = P.ws[r];
  const int gi = (int)blockIdx.x - NCOMM;
  const int xg = gi & 7, kx = gi >> 3;
  const u32 lds0 = (u32)(uintptr_t)(LDS_AS char*)smem;

  const __amdgpu_buffer_rsrc_t rs_x = mk_rsrc(P.x, X_BYTES);
  const __amdgpu_buffer_rsrc_t rs_w = mk_rsrc(P.w, W_BYTES);
  const __amdgpu_buffer_rsrc_t rs_ws = mk_rsrc(myws, WS_NEED);
  const __amdgpu_buffer_rsrc_t rs_out = mk_rsrc(P.out, OUT_BYTES);
  const float scale = P.sx[0] * P.sw[0];

  u32 vX[2], vWs[2], vWo[2];
#pragma unroll
  for (int u = 0; u < 2; ++u) {
    const int row = wave * 16 + u * 8 + (lane >> 3), slot = lane & 7;
    const int cx = slot ^ ((row >> 1) & 7);
    const int cw = slot ^ (((row >> 1) & 3) | (((row >> 4) & 1) << 2));
    vX[u] = (u32)row * KS + (u32)cx * 16u;
    vWs[u] = (u32)row * 128u + (u32)cw * 16u;
    vWo[u] = (u32)row * NG + (u32)cw * 16u;
  }
  const int wm = wave >> 2, wn = wave & 3, g = lane >> 4, l15 = lane & 15;
  const int aX0 = (wm * 64 + l15) * 128 + ((g ^ (l15 >> 1)) * 16);
  const int aX1 = aX0 ^ 64;
  const int fw = ((l15 >> 2) & 3) | ((g & 1) << 2);
  const int aW0 = 16384 + (16 * g + (l15 >> 1)) * 128 + (((2 * wn) ^ fw) * 16) + (l15 & 1) * 8;
  const int aW1 = aW0 ^ 16;
  volatile LDS_AS u32* ctrl = (volatile LDS_AS u32*)(smem + LDS_CTRL);

  for (int e = kx; e < 2 * NB; e += GPX) {
    const int b = e >> 1;
    const int a = (e & 1) == 0 ? xg : ((((b ^ xg) & 1) == 0 ? 8 : 12) + (xg >> 1));
    u32 ordd;
    if (b < NDIR) ordd = 0x17654320u;
    else ordd = ((b - NDIR) & 1) ? 0x64217530u : 0x75316420u;
    const u32 sX_own = (u32)grow_of(r, a) * PANEL;
    const u32 sW_own = (u32)((r >> 1) * NB + b) * 128u;
    const u32 sX_ws = parbase + REL_XG + (u32)(a * 8) * PANEL;
    const u32 sW_ws = parbase + REL_WG + (u32)(b * 8) * PANEL;
    const char* fptr = myws + (lane < 32 ? OFF_FX + (u32)a * 128u + (u32)lane * 4u : OFF_FW + (u32)b * 128u + (u32)(lane - 32) * 4u);

    v4f acc[2][4];
#pragma unroll
    for (int nb = 0; nb < 2; ++nb)
#pragma unroll
      for (int mb = 0; mb < 4; ++mb) acc[nb][mb] = v4f{0.f, 0.f, 0.f, 0.f};

    auto issue = [&](int kt, int stage) {
      const int dq = (ordd >> (4 * (kt >> 2))) & 7;
      const u32 q = (u32)(r ^ dq);
      const u32 kb = (u32)(kt & 3);
      const u32 ldsb = lds0 + (u32)stage * STAGE + (u32)wave * 2048u;
      if (dq == 0) {
        const u32 sx = sX_own + kb * 128u, sw = sW_own + kb * 128u * NG;
        glds16(rs_x, ldsb, vX[0], sx);
        glds16(rs_x, ldsb + 1024u, vX[1], sx);
        glds16(rs_w, ldsb + 16384u, vWo[0], sw);
        glds16(rs_w, ldsb + 16384u + 1024u, vWo[1], sw);
      } else {
        const u32 sx = sX_ws + q * PANEL + kb * 128u, sw = sW_ws + q * PANEL + kb * 16384u;
        glds16(rs_ws, ldsb, vX[0], sx);
        glds16(rs_ws, ldsb + 1024u, vX[1], sx);
        glds16(rs_ws, ldsb + 16384u, vWs[0], sw);
        glds16(rs_ws, ldsb + 16384u + 1024u, vWs[1], sw);
      }
    };
    auto compute = [&](int stage) {
      const LDS_AS char* sb = (const LDS_AS char*)smem + stage * STAGE;
      v8i xf[4], wf[2];
#pragma unroll
      for (int nb = 0; nb < 2; ++nb) {
        const LDS_AS char* pw = sb + (nb ? aW1 : aW0);
        const v2i t0_ = __builtin_amdgcn_ds_read_tr8_b64_v2i32((LDS_AS v2i*)(pw));
        const v2i t1_ = __builtin_amdgcn_ds_read_tr8_b64_v2i32((LDS_AS v2i*)(pw + 1024));
        const v2i t2_ = __builtin_amdgcn_ds_read_tr8_b64_v2i32((LDS_AS v2i*)(pw + 8192));
        const v2i t3_ = __builtin_amdgcn_ds_read_tr8_b64_v2i32((LDS_AS v2i*)(pw + 9216));
        wf[nb] = v8i{t0_.x, t0_.y, t1_.x, t1_.y, t2_.x, t2_.y, t3_.x, t3_.y};
      }
#pragma unroll
      for (int mb = 0; mb < 4; ++mb) {
        const v4i lo = *(const LDS_AS v4i*)(sb + aX0 + mb * 2048);
        const v4i hi = *(const LDS_AS v4i*)(sb + aX1 + mb * 2048);
        xf[mb] = v8i{lo.x, lo.y, lo.z, lo.w, hi.x, hi.y, hi.z, hi.w};
      }
#pragma unroll
      for (int mb = 0; mb < 4; ++mb)
#pragma unroll
        for (int nb = 0; nb < 2; ++nb)
          acc[nb][mb] = __builtin_amdgcn_mfma_scale_f32_16x16x128_f8f6f4(wf[nb], xf[mb], acc[nb][mb], 1, 0, 0, 0, 0, 0);
    };

    bool bad = false;
    int pos = 0;
    if (wave == 0) STAMP(8 * (e / GPX) + 0);
    while (pos < 8) {
      if (wave == 0) {
        int n = 0;
        for (;;) {
          const bool ok = (((lane & 31) >> 2) == r) || ld_sys(fptr) >= seq;
          const u64 m = __ballot(ok);
          const u32 m32 = (u32)m & (u32)(m >> 32);
          n = 0;
          while (pos + n < 8) {
            const u32 q = (u32)(r ^ ((ordd >> (4 * (pos + n))) & 7));
            if (((m32 >> (4 * q)) & 15u) != 15u) break;
            ++n;
          }
          if (n > 0 && !(pos == 0 && n == 1)) break;
          if (give_up(myws, seq, t0, lane)) { n = 0; break; }
          __builtin_amdgcn_s_sleep(8);
        }
        __builtin_amdgcn_fence(__ATOMIC_ACQUIRE, "");
        VMCNT(0);
        if (lane == 0) *ctrl = (u32)n;
      }
      RAW_BARRIER();
      const int n = (int)rfl(*ctrl);
      if (n == 0) { bad = true; break; }
      const int k0 = (int)rfl((u32)pos * 4u);
      issue(k0, 0); issue(k0 + 1, 1); issue(k0 + 2, 2);
      for (int ig = 0; ig < n; ++ig) {
        const int kt = k0 + ig * 4;
        const bool last = ig == n - 1;
        VMCNT(8); RAW_BARRIER();
        issue(kt + 3, 3);
        compute(0);
        VMCNT(8); RAW_BARRIER();
        if (!last) issue(kt + 4, 0);
        compute(1);
        if (!last) VMCNT(8); else VMCNT(4);
        RAW_BARRIER();
        if (!last) issue(kt + 5, 1);
        compute(2);
        if (!last) VMCNT(8); else VMCNT(0);
        RAW_BARRIER();
        if (!last) issue(kt + 6, 2);
        compute(3);
      }
      pos = (int)rfl((u32)(pos + n));
    }
    if (wave == 0) STAMP(8 * (e / GPX) + 3);

    RAW_BARRIER();
#pragma unroll
    for (int nb = 0; nb < 2; ++nb)
#pragma unroll
      for (int mb = 0; mb < 4; ++mb) {
        v4f o;
#pragma unroll
        for (int j = 0; j < 4; ++j) {
          const float y = acc[nb][mb][j] * scale;
          const float yc = fminf(fmaxf(y, -60.f), 60.f);
          o[j] = y * __builtin_amdgcn_rcpf(1.f + __expf(-yc));
          if (bad) o[j] = __builtin_nanf("");
        }
        const int m = wm * 64 + mb * 16 + l15, n4 = wn * 8 + nb * 4 + g;
        *(LDS_AS v4f*)((LDS_AS char*)smem + m * 512 + ((n4 ^ (m & 7)) * 16)) = o;
      }
    RAW_BARRIER();
    const bool own = a >= 12;
    if (own || !bad) {
#pragma unroll
      for (int i = 0; i < 8; ++i) {
        const int row = (wave * 8 + i) * 2 + (lane >> 5), c = lane & 31;
        const u4 v = *(const LDS_AS u4*)((const LDS_AS char*)smem + row * 512 + ((c ^ (row & 7)) * 16));
        if (own) {
          __builtin_amdgcn_raw_buffer_store_b128(v, rs_out, (u32)row * (NG * 4u) + (u32)c * 16u,
                                                 (u32)((a - 12) * 128) * (NG * 4u) + (u32)((r >> 1) * NB + b) * 512u, AUX_DEV);
        } else {
          __builtin_amdgcn_raw_buffer_store_b128(v, rs_ws, (u32)row * 512u + (u32)c * 16u,
                                                 parbase + REL_ST + (u32)(a * NB + b) * 65536u, AUX_SYS);
        }
      }
    }
    VMCNT(0);
    RAW_BARRIER();
    if (!own && !bad && tid == 0) {
      const int dd = 2 + (a >> 1);
      const u32 o = (u32)(r ^ dd) & 7u;
      st_sys(P.ws[o] + OFF_FO + (u32)dd * 256u + (u32)(b * 2 + (a & 1)) * 4u, seq);
    }
    if (wave == 0) STAMP(8 * (e / GPX) + 4);
  }
}

__global__ void __launch_bounds__(THREADS, 2) fused_gemm_rs(Params P) {
  extern __shared__ __attribute__((aligned(16))) char smem[];
  char* myws = P.ws[P.rank];
  const u64 t0 = __builtin_amdgcn_s_memrealtime();
  const u32 call = rfl(ld_dev(myws + OFF_CTR)) / (u32)GRID;
  const u32 seq = call + 1u;
  const u32 parbase = OFF_DATA + (call & 1u) * PAR_BYTES;
  if ((int)blockIdx.x < NCOMM) comm_role(P, smem, seq, parbase, t0);
  else gemm_role(P, smem, seq, parbase, t0);
  __syncthreads();
  if (threadIdx.x == 0)
    __hip_atomic_fetch_add((GLB_AS u32*)(myws + OFF_CTR), 1u, __ATOMIC_RELAXED, __HIP_MEMORY_SCOPE_AGENT);
}

}

extern "C" void kernel_launch(int rank, int nranks, ncclComm_t comm,
                              hipStream_t stream,
                              void* const* d_in, const int* in_sizes, int n_in,
                              void* d_out, int out_size,
                              const int* in_dtypes, int out_dtype,
                              void* const* d_out_all,
                              void* const* d_ws_all, size_t ws_bytes) {
  (void)comm; (void)in_dtypes; (void)out_dtype; (void)d_out_all;
  if (nranks != NR || n_in != 4 || rank < 0 || rank >= NR || ws_bytes < (size_t)WS_NEED ||
      in_sizes[0] != (int)X_BYTES || in_sizes[1] != (int)W_BYTES || out_size != (int)(OUT_BYTES / 4)) {
    fprintf(stderr, "kernel_launch: unexpected problem shape\n");
    return;
  }
  HIP_CHECK(hipFuncSetAttribute((const void*)fused_gemm_rs, hipFuncAttributeMaxDynamicSharedMemorySize, LDS_BYTES));
  Params P;
  P.x = (const char*)d_in[0];
  P.w = (const char*)d_in[1];
  P.sx = (const float*)d_in[2];
  P.sw = (const float*)d_in[3];
  P.out = (char*)d_out;
  for (int i = 0; i < NR; ++i) P.ws[i] = (char*)d_ws_all[i];
  P.rank = rank;
  hipLaunchKernelGGL(fused_gemm_rs, dim3(GRID), dim3(THREADS), LDS_BYTES, stream, P);
  HIP_CHECK(hipGetLastError());
}
```

```cpp
#include <hip/hip_runtime.h>
#include <rccl/rccl.h>
#include <cstdio>
#include <cstdint>

#define HIP_CHECK(expr) do { hipError_t _e = (expr); if (_e != hipSuccess) { \
    fprintf(stderr, "HIP error %s at %s:%d\n", hipGetErrorString(_e), __FILE__, __LINE__); \
    return; } } while (0)

namespace {

typedef unsigned u32;
typedef unsigned long long u64;
typedef unsigned u4 __attribute__((ext_vector_type(4)));
typedef int v2i __attribute__((ext_vector_type(2)));
typedef int v4i __attribute__((ext_vector_type(4)));
typedef int v8i __attribute__((ext_vector_type(8)));
typedef float v4f __attribute__((ext_vector_type(4)));
#define LDS_AS __attribute__((address_space(3)))
#define GLB_AS __attribute__((address_space(1)))

constexpr int NR = 8;
constexpr int MG = 4096, KG = 4096, NG = 16384;
constexpr int KS = KG / NR;
constexpr u32 PANEL = 128 * KS;
constexpr int NA = 16;
constexpr int NB = 32;
#define WPL_CFG 8
constexpr int WPL = WPL_CFG;
constexpr int NCOMM = 7 * WPL;
constexpr int GRID = 256;
constexpr int NGEMM = GRID - NCOMM;
constexpr int GPX = NGEMM / 8;
constexpr int THREADS = 576;
static_assert(NCOMM % 8 == 0 && NGEMM % 8 == 0, "roles");

constexpr int MAXU = 128;
constexpr int MAXP = 4 * MAXU;

constexpr u32 OFF_CTR = 0, OFF_TMO = 128, OFF_ZERO = 256;
constexpr u32 OFF_TK = 1024;
constexpr u32 OFF_FX = 4096;
constexpr u32 OFF_FW = 8192;
constexpr u32 OFF_FO = 16384;
constexpr u32 OFF_DONE = 24576;
constexpr u32 OFF_DBG = 512 << 10;
constexpr u32 OFF_ST = 1 << 20;
constexpr u32 OFF_DATA = OFF_ST + 12 * NB * 65536;
constexpr u32 SRC_BYTES = (NA + NB) * PANEL;
constexpr u32 REL_XG = 0;
constexpr u32 REL_WG = NA * PANEL;
constexpr u32 PAR_BYTES = 8 * SRC_BYTES;
constexpr u32 WS_NEED = OFF_DATA + 2 * PAR_BYTES;

constexpr u32 X_BYTES = MG * KS;
constexpr u32 W_BYTES = KS * NG;
constexpr u32 OUT_BYTES = (MG / NR) * NG * 4;

constexpr int STAGE = 32768, NST = 4;
constexpr int LDS_CTRL = NST * STAGE;
constexpr int LDS_BYTES = LDS_CTRL + 256;

constexpr u64 DEADLINE = 50000000ull;

constexpr int AUX_SYS = 17;
constexpr int AUX_DEV = 16;

struct Params {
  const char* x;
  const char* w;
  const float* sx;
  const float* sw;
  char* out[NR];
  char* ws[NR];
  int rank;
  int ndir;
  int la;
  int grp;
  int nunits[3];
  u32 ord[3][MAXU / 4];
  u32 tiles[2][16];
};

__device__ __forceinline__ u32 rfl(u32 v) { return __builtin_amdgcn_readfirstlane(v); }
__device__ __forceinline__ __amdgpu_buffer_rsrc_t mk_rsrc(const void* p, u32 bytes) {
  return __builtin_amdgcn_make_buffer_rsrc((void*)p, (short)0, (int)bytes, 0x00020000);
}
__device__ __forceinline__ u32 ld_sys(const char* p) {
  return __hip_atomic_load((const GLB_AS u32*)p, __ATOMIC_RELAXED, __HIP_MEMORY_SCOPE_SYSTEM);
}
__device__ __forceinline__ void st_sys(char* p, u32 v) {
  __hip_atomic_store((GLB_AS u32*)p, v, __ATOMIC_RELAXED, __HIP_MEMORY_SCOPE_SYSTEM);
}
__device__ __forceinline__ u32 ld_dev(const char* p) {
  return __hip_atomic_load((const GLB_AS u32*)p, __ATOMIC_RELAXED, __HIP_MEMORY_SCOPE_AGENT);
}
__device__ __forceinline__ void st_dev(char* p, u32 v) {
  __hip_atomic_store((GLB_AS u32*)p, v, __ATOMIC_RELAXED, __HIP_MEMORY_SCOPE_AGENT);
}
#define VMCNT(n) asm volatile("s_waitcnt vmcnt(" #n ")" ::: "memory")
#define SCHED_FENCE() do { asm volatile("" ::: "memory"); __builtin_amdgcn_sched_barrier(0); } while (0)
#define RAW_BARRIER() do { asm volatile("s_waitcnt lgkmcnt(0)" ::: "memory"); __builtin_amdgcn_s_barrier(); asm volatile("" ::: "memory"); } while (0)

__device__ __forceinline__ bool give_up(char* myws, u32 seq, u64 t0) {
  const u32 t = rfl(ld_dev(myws + OFF_TMO));
  const bool late = (__builtin_amdgcn_s_memrealtime() - t0) > DEADLINE;
  if (t == seq) return true;
  if (late) {
    st_dev(myws + OFF_TMO, seq);
    return true;
  }
  return false;
}

#define STAMP(slot) do {} while (0)
#define PVAL(slot, v) do {} while (0)
#define PNOW() 0ull

__device__ __forceinline__ int grow_of(int p, int a) {
  if (a < 12) return ((p ^ (2 + (a >> 1))) << 2) + ((p & 1) << 1) + (a & 1);
  return (p << 2) + (a - 12);
}

constexpr u32 K_X = 1, K_W = 2, K_FWD = 3, K_OUT = 4, K_END = 5;

__device__ __forceinline__ void comm_role(const Params& P, char* smem, u32 call, u32 parbase, u64 t0) {
  const int tid = threadIdx.x, lane = tid & 63;
  const int wave = rfl(tid >> 6);
  const int r = P.rank;
  const u32 seq = call + 1u;
  const int d = 1 + (int)blockIdx.x % 7;
  const int w = (int)blockIdx.x / 7;
  const u32 p = (u32)(r ^ d) & 7u;
  char* myws = P.ws[r];
  char* pws = P.ws[p];
  const int cls = d == 1 ? 0 : 1 + (d & 1);
  const int NP = 4 * P.nunits[cls];
  const int LA = P.la, NDIR = P.ndir;

  volatile LDS_AS u4* desc = (volatile LDS_AS u4*)smem;
  volatile LDS_AS u32* cnt = (volatile LDS_AS u32*)(smem + (MAXP + 4) * 16);
  volatile LDS_AS u32* fail = cnt + MAXP + 3;
  for (int i = tid; i < MAXP + 4; i += THREADS) { desc[i] = u4{0u, 0u, 0u, 0u}; cnt[i] = 0u; }
  __syncthreads();
  if (wave == 0) STAMP(0);

  if (wave == 8) {
    const __amdgpu_buffer_rsrc_t rs_my = mk_rsrc(myws, WS_NEED);
    u64 stall = 0;
    u32 woff = OFF_FO + (u32)d * 1024u + (u32)lane * 16u;
    if (d == 1) {
      const int fi = lane < 3 * (NB - NDIR) ? lane : 0;
      const int k = fi / 3, i = fi - 3 * k;
      woff = OFF_FW + (u32)(((NDIR + k) * 8 + (r ^ (2 * i + 2 + (k & 1)))) * 16);
    }
    const u32 tab = P.ord[cls][lane & 31];
    u64 rdy = 0;
    auto poll = [&]() {
      const u4 f = __builtin_amdgcn_raw_buffer_load_b128(rs_my, woff, 0, AUX_SYS);
      rdy |= __ballot(f.x >= seq && f.y >= seq && f.z >= seq && f.w >= seq);
    };
    auto post = [&](int s, u32 e0, u32 e1, u32 e2, u32 e3) {
      volatile LDS_AS u32* dp = (volatile LDS_AS u32*)(desc + s);
      if (lane == 0) { dp[0] = e0; dp[1] = e1; dp[2] = e2; dp[3] = e3; }
    };
    const int G = P.grp, NS = (LA + G - 1) / G;
    const int NT = (NP + G * WPL - 1) / (G * WPL) * WPL;
    const u32 tkbase = call * (u32)(NT - WPL * NS + WPL) - (u32)(WPL * NS);
    char* tkptr = myws + OFF_TK + (u32)d * 128u;
    bool dead = false;
    int s = 0, tj = 0, g = G - 1;
    u64 rtt = 0;
    for (int k = 0;;) {
      if (++g == G) {
        g = 0;
        if (k < NS) tj = w + WPL * k;
        else {
          if (!dead && s >= LA) while (rfl(cnt[s - LA]) == 0u) __builtin_amdgcn_s_sleep(1);
          const u64 tr0 = PNOW();
          const u4 f = __builtin_amdgcn_raw_buffer_load_b128(rs_my, woff, 0, AUX_SYS);
          u32 old = 0;
          if (lane == 0) old = __hip_atomic_fetch_add((GLB_AS u32*)tkptr, 1u, __ATOMIC_RELAXED, __HIP_MEMORY_SCOPE_AGENT);
          tj = (int)(rfl(old) - tkbase);
          rdy |= __ballot(f.x >= seq && f.y >= seq && f.z >= seq && f.w >= seq);
          rtt += PNOW() - tr0;
        }
        ++k;
        if ((u32)tj >= (u32)NT) break;
      }
      const int tk = (tj / WPL) * G * WPL + tj % WPL + g * WPL;
      if (dead || tk >= NP) continue;
      const int pos = tk >> 2, piece = tk & 3;
      const u32 ent = ((u32)__builtin_amdgcn_readlane((int)tab, pos >> 2) >> (8 * (pos & 3))) & 255u;
      const u32 kind = (ent >> 6) + 1u;
      int idx = (int)(ent & 63u), q = r, wi = -1;
      if (kind == K_OUT) wi = idx;
      if (kind == K_FWD) {
        wi = idx;
        const int kk = idx / 3, i = idx - 3 * kk;
        q = r ^ (2 * i + 2 + (kk & 1));
        idx = NDIR + kk;
      }
      u32 e_src, e_dst, e_flag;
      if (kind == K_X) {
        e_src = (u32)grow_of((int)p, idx) * PANEL + piece * 16384u;
        e_dst = parbase + REL_XG + (u32)r * SRC_BYTES + (u32)idx * PANEL + piece * 16384u;
        e_flag = OFF_FX + (u32)((idx * 8 + r) * 4 + piece) * 4u;
      } else if (kind == K_OUT) {
        const int b = idx >> 1, tt = idx & 1;
        e_src = OFF_ST + (u32)(((d - 2) * 2 + tt) * NB + b) * 65536u + piece * 16384u;
        e_dst = (u32)((2 * (r & 1) + tt) * 128 + piece * 32) * (NG * 4u) + (u32)((r >> 1) * NB + b) * 512u;
        e_flag = 0u;
      } else {
        e_dst = parbase + REL_WG + (u32)q * SRC_BYTES + (u32)idx * PANEL + piece * 16384u;
        e_flag = OFF_FW + (u32)((idx * 8 + q) * 4 + piece) * 4u;
        e_src = kind == K_W ? (u32)piece * 128u * NG + ((p >> 1) * NB + (u32)idx) * 128u : e_dst;
      }
      if (wi >= 0 && !((rdy >> wi) & 1ull)) {
        const u64 tw0 = PNOW();
        for (;;) {
          poll();
          if ((rdy >> wi) & 1ull) break;
          if (give_up(myws, seq, t0)) { dead = true; break; }
          __builtin_amdgcn_s_sleep(2);
        }
        stall += PNOW() - tw0;
        if (dead) { *fail = 1u; post(s, 0u, 0u, 0u, K_END); continue; }
      }
      asm volatile("" ::: "memory");
      {
        const u64 sa = (u64)(kind == K_X ? P.x : (kind == K_W ? P.w : (const char*)myws)) + e_src;
        const u64 da = (u64)(kind == K_OUT ? P.out[p] : pws) + e_dst;
        post(s, (u32)sa, (u32)da, (u32)(sa >> 32) | (u32)(da >> 32) << 16, e_flag << 8 | kind);
      }
      ++s;
    }
    if (!dead) post(s, 0u, 0u, 0u, K_END);
    PVAL(4, stall);
    PVAL(5, (u64)s);
    PVAL(6, rtt);
  } else {
    const __amdgpu_buffer_rsrc_t rs_pws = mk_rsrc(pws, WS_NEED);
    const u32 v_lin = (u32)wave * 2048u + (u32)lane * 16u;
    const u32 v_str = (u32)(wave * 16 + (lane >> 3)) * NG + (u32)(lane & 7) * 16u;
    const u32 v_out = (u32)(wave * 4 + (lane >> 5)) * (NG * 4u) + (u32)(lane & 31) * 16u;
    const u32 cnt0 = (u32)(uintptr_t)cnt;
    struct D { u32 slo, shi, dlo, dhi, sstep, dstep, flag, kind, vs, vd; };
    auto fetch = [&](int t) -> D {
      const u4 e = desc[t];
      D c;
      c.slo = rfl(e.x); c.dlo = rfl(e.y);
      const u32 h = rfl(e.z), k = rfl(e.w);
      c.shi = h & 0xffffu; c.dhi = h >> 16;
      c.kind = k & 255u; c.flag = k >> 8;
      c.sstep = c.kind == K_W ? 8u * NG : 1024u;
      c.dstep = c.kind == K_OUT ? 2u * NG * 4u : 1024u;
      c.vs = c.kind == K_W ? v_str : v_lin;
      c.vd = c.kind == K_OUT ? v_out : v_lin;
      asm volatile("" :: "s"(c.slo), "s"(c.shi), "s"(c.dlo), "s"(c.dhi), "s"(c.sstep), "s"(c.dstep), "v"(c.vs), "v"(c.vd));
      return c;
    };
    auto arrive = [&](int t, u32 flag) {
      u32 old;
      u64 keep;
      asm volatile("s_mov_b64 %1, exec\n\ts_mov_b64 exec, 1\n\tds_add_rtn_u32 %0, %2, %3\n\ts_waitcnt lgkmcnt(0)\n\ts_mov_b64 exec, %1"
                   : "=&v"(old), "=&s"(keep) : "v"(cnt0 + 4u * (u32)t), "v"(1u) : "memory");
      if (rfl(old) == 7u && flag != 0u)
        __builtin_amdgcn_raw_buffer_store_b32(seq, rs_pws, 0, flag, AUX_SYS);
    };
    auto load_step = [&](const D& e, int u) -> u4 {
      return __builtin_amdgcn_raw_buffer_load_b128(mk_rsrc((const void*)((u64)e.shi << 32 | e.slo), 0x200000u), e.vs, u ? e.sstep : 0u, AUX_SYS);
    };
    auto store_step = [&](const D& e, int u, const u4& a) {
      __builtin_amdgcn_raw_buffer_store_b128(a, mk_rsrc((const void*)((u64)e.dhi << 32 | e.dlo), 0x200000u), e.vd, u ? e.dstep : 0u, AUX_SYS);
    };
    auto usable = [](const D& e) { return e.kind != 0u && e.kind != K_END; };
    int t = 0;
    u64 nruns = 0;
    for (;;) {
      ++nruns;
      D C = fetch(t);
      while (C.kind == 0u) { __builtin_amdgcn_s_sleep(1); C = fetch(t); }
      if (C.kind == K_END) break;
      u4 a0 = load_step(C, 0);
      SCHED_FENCE();
      u4 a1 = load_step(C, 1);
      D N = fetch(t + 1);
      const int tfirst = t;
      u32 fprev = 0u;
      for (;;) {
        const bool more = usable(N);
        D L = more ? N : C;
        asm volatile("" :: "s"(L.slo), "s"(L.shi), "s"(L.sstep), "v"(L.vs));
        VMCNT(1);
        store_step(C, 0, a0);
        a0 = load_step(L, 0);
        if (t != tfirst) arrive(t - 1, fprev);
        D N2 = fetch(t + 2);
        VMCNT(1);
        store_step(C, 1, a1);
        a1 = load_step(L, 1);
        fprev = C.flag;
        ++t;
        if (!more) break;
        if (N2.kind == 0u) N2 = fetch(t + 1);
        C = N;
        N = N2;
      }
      VMCNT(0);
      arrive(t - 1, fprev);
    }
    PVAL(40 + wave, nruns);
  }
  __syncthreads();
  if (wave == 0) STAMP(1);
  if (d == 1 || wave != 0) return;
  const u32 doff = OFF_DONE + (u32)(d * WPL + w) * 64u;
  if (rfl(*fail) == 0u) st_sys(pws + doff, seq);
  while (rfl(ld_sys(myws + doff)) < seq) {
    if (give_up(myws, seq, t0)) {
      st_sys(P.out[r] + (u32)((r >> 1) * NB) * 512u, 0x7fc00000u);
      break;
    }
    __builtin_amdgcn_s_sleep(2);
  }
  STAMP(3);
}

__device__ __forceinline__ void glds16(__amdgpu_buffer_rsrc_t rs, u32 lds, u32 voff, u32 soff) {
  asm volatile("s_mov_b32 m0, %0\n\ts_nop 3\n\tbuffer_load_dwordx4 %1, %2, %3 offen lds"
               :: "s"(rfl(lds)), "v"(voff), "s"(rs), "s"(rfl(soff)) : "m0", "memory");
}

__device__ __forceinline__ void gemm_role(const Params& P, char* smem, u32 seq, u32 parbase, u64 t0) {
  const int tid = threadIdx.x, lane = tid & 63;
  const int wave = rfl(tid >> 6);
  const int r = P.rank;
  char* myws = P.ws[r];
  const int gi = (int)blockIdx.x - NCOMM;
  const int xg = gi & 7, kx = gi >> 3;
  const u32 lds0 = (u32)(uintptr_t)(LDS_AS char*)smem;

  const __amdgpu_buffer_rsrc_t rs_x = mk_rsrc(P.x, X_BYTES);
  const __amdgpu_buffer_rsrc_t rs_w = mk_rsrc(P.w, W_BYTES);
  const __amdgpu_buffer_rsrc_t rs_ws = mk_rsrc(myws, WS_NEED);
  const __amdgpu_buffer_rsrc_t rs_out = mk_rsrc(P.out[r], OUT_BYTES);
  const float scale = P.sx[0] * P.sw[0];

  u32 vX[2], vWs[2], vWo[2];
#pragma unroll
  for (int u = 0; u < 2; ++u) {
    const int row = wave * 16 + u * 8 + (lane >> 3), slot = lane & 7;
    const int cx = slot ^ ((row >> 1) & 7);
    const int cw = slot ^ (((row >> 1) & 3) | (((row >> 4) & 1) << 2));
    vX[u] = (u32)row * KS + (u32)cx * 16u;
    vWs[u] = (u32)row * 128u + (u32)cw * 16u;
    vWo[u] = (u32)row * NG + (u32)cw * 16u;
  }
  const int wm = wave >> 2, wn = wave & 3, g = lane >> 4, l15 = lane & 15;
  const int aX0 = (wm * 64 + l15) * 128 + ((g ^ (l15 >> 1)) * 16);
  const int aX1 = aX0 ^ 64;
  const int fw = ((l15 >> 2) & 3) | ((g & 1) << 2);
  const int aW0 = 16384 + (16 * g + (l15 >> 1)) * 128 + (((2 * wn) ^ fw) * 16) + (l15 & 1) * 8;
  const int aW1 = aW0 ^ 16;
  volatile LDS_AS u32* ctrl = (volatile LDS_AS u32*)(smem + LDS_CTRL);

  for (int e = kx; e < 2 * NB; e += GPX) {
    const u32 ent = (P.tiles[xg & 1][e >> 2] >> (8 * (e & 3))) & 255u;
    const int b = (int)(ent & 31u);
    const int a = (ent >> 5) == 0u ? xg : ((((b ^ xg) & 1) == 0 ? 8 : 12) + (xg >> 1));
    const int NDIR = P.ndir;
    u32 ordd;
    if (b < NDIR) ordd = 0x17654320u;
    else ordd = ((b - NDIR) & 1) ? 0x64217530u : 0x75316420u;
    const u32 sX_own = (u32)grow_of(r, a) * PANEL;
    const u32 sW_own = (u32)((r >> 1) * NB + b) * 128u;
    const u32 sX_ws = parbase + REL_XG + (u32)a * PANEL;
    const u32 sW_ws = parbase + REL_WG + (u32)b * PANEL;
    const char* fptr = myws + (lane < 32 ? OFF_FX + (u32)a * 128u + (u32)lane * 4u : OFF_FW + (u32)b * 128u + (u32)(lane - 32) * 4u);

    v4f acc[2][4];
#pragma unroll
    for (int nb = 0; nb < 2; ++nb)
#pragma unroll
      for (int mb = 0; mb < 4; ++mb) acc[nb][mb] = v4f{0.f, 0.f, 0.f, 0.f};

    auto issue = [&](int kt, int stage) {
      const int dq = (ordd >> (4 * (kt >> 2))) & 7;
      const u32 q = (u32)(r ^ dq);
      const u32 kb = (u32)(kt & 3);
      const u32 ldsb = lds0 + (u32)stage * STAGE + (u32)wave * 2048u;
      if (dq == 0) {
        const u32 sx = sX_own + kb * 128u, sw = sW_own + kb * 128u * NG;
        glds16(rs_x, ldsb, vX[0], sx);
        glds16(rs_x, ldsb + 1024u, vX[1], sx);
        glds16(rs_w, ldsb + 16384u, vWo[0], sw);
        glds16(rs_w, ldsb + 16384u + 1024u, vWo[1], sw);
      } else {
        const u32 sx = sX_ws + q * SRC_BYTES + kb * 128u, sw = sW_ws + q * SRC_BYTES + kb * 16384u;
        glds16(rs_ws, ldsb, vX[0], sx);
        glds16(rs_ws, ldsb + 1024u, vX[1], sx);
        glds16(rs_ws, ldsb + 16384u, vWs[0], sw);
        glds16(rs_ws, ldsb + 16384u + 1024u, vWs[1], sw);
      }
    };
    auto compute = [&](int stage) {
      const LDS_AS char* sb = (const LDS_AS char*)smem + stage * STAGE;
      v8i xf[4], wf[2];
#pragma unroll
      for (int nb = 0; nb < 2; ++nb) {
        const LDS_AS char* pw = sb + (nb ? aW1 : aW0);
        const v2i t0_ = __builtin_amdgcn_ds_read_tr8_b64_v2i32((LDS_AS v2i*)(pw));
        const v2i t1_ = __builtin_amdgcn_ds_read_tr8_b64_v2i32((LDS_AS v2i*)(pw + 1024));
        const v2i t2_ = __builtin_amdgcn_ds_read_tr8_b64_v2i32((LDS_AS v2i*)(pw + 8192));
        const v2i t3_ = __builtin_amdgcn_ds_read_tr8_b64_v2i32((LDS_AS v2i*)(pw + 9216));
        wf[nb] = v8i{t0_.x, t0_.y, t1_.x, t1_.y, t2_.x, t2_.y, t3_.x, t3_.y};
      }
#pragma unroll
      for (int mb = 0; mb < 4; ++mb) {
        const v4i lo = *(const LDS_AS v4i*)(sb + aX0 + mb * 2048);
        const v4i hi = *(const LDS_AS v4i*)(sb + aX1 + mb * 2048);
        xf[mb] = v8i{lo.x, lo.y, lo.z, lo.w, hi.x, hi.y, hi.z, hi.w};
      }
#pragma unroll
      for (int mb = 0; mb < 4; ++mb)
#pragma unroll
        for (int nb = 0; nb < 2; ++nb)
          acc[nb][mb] = __builtin_amdgcn_mfma_scale_f32_16x16x128_f8f6f4(wf[nb], xf[mb], acc[nb][mb], 1, 0, 0, 0, 0, 0);
    };

    bool bad = false;
    int pos = 0;
    if (wave == 0) STAMP(8 * (e / GPX) + 0);
    while (pos < 8) {
      if (wave == 0) {
        int n = 0;
        for (;;) {
          const bool ok = (((lane & 31) >> 2) == r) || ld_sys(fptr) >= seq;
          const u64 m = __ballot(ok);
          const u32 m32 = (u32)m & (u32)(m >> 32);
          n = 0;
          while (pos + n < 8) {
            const u32 q = (u32)(r ^ ((ordd >> (4 * (pos + n))) & 7));
            if (((m32 >> (4 * q)) & 15u) != 15u) break;
            ++n;
          }
          if (n > 0 && !(pos == 0 && n == 1)) break;
          if (give_up(myws, seq, t0)) { n = 0; break; }
          __builtin_amdgcn_s_sleep(8);
        }
        __builtin_amdgcn_fence(__ATOMIC_ACQUIRE, "");
        VMCNT(0);
        if (lane == 0) *ctrl = (u32)n;
      }
      RAW_BARRIER();
      const int n = (int)rfl(*ctrl);
      if (n == 0) { bad = true; break; }
      const int k0 = (int)rfl((u32)pos * 4u);
      issue(k0, 0); issue(k0 + 1, 1); issue(k0 + 2, 2);
      for (int ig = 0; ig < n; ++ig) {
        const int kt = k0 + ig * 4;
        const bool last = ig == n - 1;
        VMCNT(8); RAW_BARRIER();
        issue(kt + 3, 3);
        compute(0);
        VMCNT(8); RAW_BARRIER();
        if (!last) issue(kt + 4, 0);
        compute(1);
        if (!last) VMCNT(8); else VMCNT(4);
        RAW_BARRIER();
        if (!last) issue(kt + 5, 1);
        compute(2);
        if (!last) VMCNT(8); else VMCNT(0);
        RAW_BARRIER();
        if (!last) issue(kt + 6, 2);
        compute(3);
      }
      pos = (int)rfl((u32)(pos + n));
    }
    if (wave == 0) STAMP(8 * (e / GPX) + 3);

    RAW_BARRIER();
#pragma unroll
    for (int nb = 0; nb < 2; ++nb)
#pragma unroll
      for (int mb = 0; mb < 4; ++mb) {
        v4f o;
#pragma unroll
        for (int j = 0; j < 4; ++j) {
          const float y = acc[nb][mb][j] * scale;
          const float yc = fminf(fmaxf(y, -60.f), 60.f);
          o[j] = y * __builtin_amdgcn_rcpf(1.f + __expf(-yc));
          if (bad) o[j] = __builtin_nanf("");
        }
        const int m = wm * 64 + mb * 16 + l15, n4 = wn * 8 + nb * 4 + g;
        *(LDS_AS v4f*)((LDS_AS char*)smem + m * 512 + ((n4 ^ (m & 7)) * 16)) = o;
      }
    RAW_BARRIER();
    const bool own = a >= 12;
    if (own || !bad) {
#pragma unroll
      for (int i = 0; i < 8; ++i) {
        const int row = (wave * 8 + i) * 2 + (lane >> 5), c = lane & 31;
        const u4 v = *(const LDS_AS u4*)((const LDS_AS char*)smem + row * 512 + ((c ^ (row & 7)) * 16));
        if (own) {
          __builtin_amdgcn_raw_buffer_store_b128(v, rs_out, (u32)row * (NG * 4u) + (u32)c * 16u,
                                                 (u32)((a - 12) * 128) * (NG * 4u) + (u32)((r >> 1) * NB + b) * 512u, AUX_DEV);
        } else {
          __builtin_amdgcn_raw_buffer_store_b128(v, rs_ws, (u32)row * 512u + (u32)c * 16u,
                                                 OFF_ST + (u32)(a * NB + b) * 65536u, AUX_DEV);
        }
      }
    }
    VMCNT(0);
    RAW_BARRIER();
    if (!own && !bad && tid < 4) {
      const int dd = 2 + (a >> 1);
      st_dev(myws + OFF_FO + (u32)dd * 1024u + (u32)(b * 2 + (a & 1)) * 16u + (u32)tid * 4u, seq);
    }
    if (wave == 0) STAMP(8 * (e / GPX) + 4);
  }
}

__global__ void __launch_bounds__(THREADS, 3) fused_gemm_rs(Params P) {
  extern __shared__ __attribute__((aligned(16))) char smem[];
  const bool comm = (int)blockIdx.x < NCOMM;
  if (!comm && rfl(threadIdx.x >> 6) == 8u) return;
  char* myws = P.ws[P.rank];
  const u64 t0 = __builtin_amdgcn_s_memrealtime();
  const u32 call = rfl(ld_dev(myws + OFF_CTR)) / (u32)GRID;
  const u32 parbase = OFF_DATA + (call & 1u) * PAR_BYTES;
  if (comm) comm_role(P, smem, call, parbase, t0);
  else gemm_role(P, smem, call + 1u, parbase, t0);
  if (threadIdx.x == 0)
    __hip_atomic_fetch_add((GLB_AS u32*)(myws + OFF_CTR), 1u, __ATOMIC_RELAXED, __HIP_MEMORY_SCOPE_AGENT);
}

struct Tune { int nrel, la, ownx_after, own_delay, relay_start, grp, rep; };

void build_schedule(Params& P, const Tune& t) {
  const int ndir = NB - t.nrel;
  P.ndir = ndir;
  P.la = t.la;
  P.grp = t.grp;
  for (int cls = 0; cls < 3; ++cls) {
    unsigned char L[MAXU] = {};
    int n = 0, nw = 0;
    bool ownx = false;
    auto put = [&](u32 kind, int idx) { if (n < MAXU) L[n++] = (unsigned char)(((kind - 1u) << 6) | (u32)idx); };
    auto putw = [&](int b) {
      if (!ownx && nw >= t.ownx_after) { for (int a = 12; a < 16; ++a) put(K_X, a); ownx = true; }
      put(K_W, b); ++nw;
    };
    for (int a = 0; a < 12; ++a) put(K_X, a);
    if (cls == 0) {
      for (int b = 0; b < ndir; ++b) putw(b);
      for (int k = 0; k < t.nrel; ++k) {
        putw(ndir + k);
        for (int i = 0; i < 3; ++i) put(K_FWD, 3 * k + i);
      }
    } else {
      const int par = cls - 1;
      const int nr = (t.nrel + 1 - par) / 2;
      const int lead = t.relay_start < ndir ? t.relay_start : ndir;
      for (int b = 0; b < lead; ++b) putw(b);
      const int rest = ndir - lead, tot = rest + nr;
      for (int j = 0; j < tot; ++j) {
        const int r0 = j * nr / tot, r1 = (j + 1) * nr / tot;
        if (r1 > r0) putw(ndir + 2 * r0 + par); else putw(lead + j - r0);
      }
    }
    if (!ownx) for (int a = 12; a < 16; ++a) put(K_X, a);
    if (cls != 0) for (int o = 0; o < 2 * NB; ++o) put(K_OUT, o);
    P.nunits[cls] = n;
    for (int i = 0; i < MAXU / 4; ++i)
      P.ord[cls][i] = (u32)L[4 * i] | (u32)L[4 * i + 1] << 8 | (u32)L[4 * i + 2] << 16 | (u32)L[4 * i + 3] << 24;
  }
  for (int xp = 0; xp < 2; ++xp) {
    unsigned char T[2 * NB];
    int n = 0;
    for (int key = 0; key < NB + t.own_delay; ++key) {
      if (key < NB) {
        T[n++] = (unsigned char)key;
        if (((key ^ xp) & 1) == 0) T[n++] = (unsigned char)(32 | key);
      }
      const int ob = key - t.own_delay;
      if (ob >= 0 && ob < NB && ((ob ^ xp) & 1) != 0) T[n++] = (unsigned char)(32 | ob);
    }
    for (int i = 0; i < 16; ++i)
      P.tiles[xp][i] = (u32)T[4 * i] | (u32)T[4 * i + 1] << 8 | (u32)T[4 * i + 2] << 16 | (u32)T[4 * i + 3] << 24;
  }
}

}

extern "C" void kernel_launch(int rank, int nranks, ncclComm_t comm,
                              hipStream_t stream,
                              void* const* d_in, const int* in_sizes, int n_in,
                              void* d_out, int out_size,
                              const int* in_dtypes, int out_dtype,
                              void* const* d_out_all,
                              void* const* d_ws_all, size_t ws_bytes) {
  (void)comm; (void)in_dtypes; (void)out_dtype;
  if (nranks != NR || n_in != 4 || rank < 0 || rank >= NR || ws_bytes < (size_t)WS_NEED ||
      in_sizes[0] != (int)X_BYTES || in_sizes[1] != (int)W_BYTES || out_size != (int)(OUT_BYTES / 4)) {
    fprintf(stderr, "kernel_launch: unexpected problem shape\n");
    return;
  }
  static bool attr_set[NR] = {};
  if (!attr_set[rank]) {
    HIP_CHECK(hipFuncSetAttribute((const void*)fused_gemm_rs, hipFuncAttributeMaxDynamicSharedMemorySize, LDS_BYTES));
    attr_set[rank] = true;
  }
  Params P;
  P.x = (const char*)d_in[0];
  P.w = (const char*)d_in[1];
  P.sx = (const float*)d_in[2];
  P.sw = (const float*)d_in[3];
  for (int i = 0; i < NR; ++i) { P.ws[i] = (char*)d_ws_all[i]; P.out[i] = (char*)d_out_all[i]; }
  P.out[rank] = (char*)d_out;
  P.rank = rank;
  Tune t = {14, 3, 8, 6, 4, 2, 0};
  build_schedule(P, t);
  hipLaunchKernelGGL(fused_gemm_rs, dim3(GRID), dim3(THREADS), LDS_BYTES, stream, P);
  HIP_CHECK(hipGetLastError());
}
```

```cpp
#include <hip/hip_runtime.h>
#include <rccl/rccl.h>
#include <cstdio>
#include <cstdint>

#define HIP_CHECK(expr) do { hipError_t _e = (expr); if (_e != hipSuccess) { \
    fprintf(stderr, "HIP error %s at %s:%d\n", hipGetErrorString(_e), __FILE__, __LINE__); \
    return; } } while (0)

namespace {

typedef unsigned u32;
typedef unsigned long long u64;
typedef unsigned u4 __attribute__((ext_vector_type(4)));
typedef int v2i __attribute__((ext_vector_type(2)));
typedef int v4i __attribute__((ext_vector_type(4)));
typedef int v8i __attribute__((ext_vector_type(8)));
typedef float v4f __attribute__((ext_vector_type(4)));
#define LDS_AS __attribute__((address_space(3)))
#define GLB_AS __attribute__((address_space(1)))

constexpr int NR = 8;
constexpr int MG = 4096, KG = 4096, NG = 16384;
constexpr int KS = KG / NR;
constexpr u32 PANEL = 128 * KS;
constexpr int NA = 16;
constexpr int NB = 32;
#define WPL_CFG 8
constexpr int WPL = WPL_CFG;
constexpr int NCOMM = 7 * WPL;
constexpr int GRID = 256;
constexpr int NGEMM = GRID - NCOMM;
constexpr int GPX = NGEMM / 8;
constexpr int THREADS = 640;

constexpr int MAXU = 128;
constexpr int MAXP = 4 * MAXU;

constexpr u32 OFF_CTR = 0, OFF_TMO = 128, OFF_ZERO = 256;
constexpr u32 OFF_TK = 1024;
constexpr u32 OFF_FX = 4096;
constexpr u32 OFF_FW = 8192;
constexpr u32 OFF_FO = 16384;
constexpr u32 OFF_DONE = 24576;
constexpr u32 OFF_FI = 32768;
constexpr u32 OFF_DBG = 512 << 10;
constexpr u32 OFF_ST = 1 << 20;
constexpr u32 OFF_DATA = OFF_ST + 12 * NB * 65536;
constexpr u32 SRC_BYTES = (NA + NB) * PANEL;
constexpr u32 REL_XG = 0;
constexpr u32 REL_WG = NA * PANEL;
constexpr u32 PAR_BYTES = 8 * SRC_BYTES;
constexpr u32 WS_NEED = OFF_DATA + 2 * PAR_BYTES;

constexpr u32 X_BYTES = MG * KS;
constexpr u32 W_BYTES = KS * NG;
constexpr u32 OUT_BYTES = (MG / NR) * NG * 4;

constexpr int STAGE = 32768, NST = 4;
constexpr int LDS_CTRL = NST * STAGE;
constexpr int LDS_BYTES = LDS_CTRL + 256;

constexpr u64 DEADLINE = 50000000ull;

constexpr int AUX_SYS = 17;
constexpr int AUX_DEV = 16;

struct Params {
  const char* x;
  const char* w;
  const float* sx;
  const float* sw;
  char* out[NR];
  char* ws[NR];
  int rank;
  int ndir;
  int la;
  int grp;
  int pull_from[3];
  int nunits[3];
  u32 ord[3][MAXU / 4];
  u32 tiles[2][16];
  u32 role[GRID / 4];
  u32 gpx[2];
};

__device__ __forceinline__ u32 rfl(u32 v) { return __builtin_amdgcn_readfirstlane(v); }
__device__ __forceinline__ __amdgpu_buffer_rsrc_t mk_rsrc(const void* p, u32 bytes) {
  return __builtin_amdgcn_make_buffer_rsrc((void*)p, (short)0, (int)bytes, 0x00020000);
}
__device__ __forceinline__ u32 ld_sys(const char* p) {
  return __hip_atomic_load((const GLB_AS u32*)p, __ATOMIC_RELAXED, __HIP_MEMORY_SCOPE_SYSTEM);
}
__device__ __forceinline__ void st_sys(char* p, u32 v) {
  __hip_atomic_store((GLB_AS u32*)p, v, __ATOMIC_RELAXED, __HIP_MEMORY_SCOPE_SYSTEM);
}
__device__ __forceinline__ u32 ld_dev(const char* p) {
  return __hip_atomic_load((const GLB_AS u32*)p, __ATOMIC_RELAXED, __HIP_MEMORY_SCOPE_AGENT);
}
__device__ __forceinline__ void st_dev(char* p, u32 v) {
  __hip_atomic_store((GLB_AS u32*)p, v, __ATOMIC_RELAXED, __HIP_MEMORY_SCOPE_AGENT);
}
#define VMCNT(n) asm volatile("s_waitcnt vmcnt(" #n ")" ::: "memory")
#define SCHED_FENCE() do { asm volatile("" ::: "memory"); __builtin_amdgcn_sched_barrier(0); } while (0)
#define RAW_BARRIER() do { asm volatile("s_waitcnt lgkmcnt(0)" ::: "memory"); __builtin_amdgcn_s_barrier(); asm volatile("" ::: "memory"); } while (0)

__device__ __forceinline__ bool give_up(char* myws, u32 seq, u64 t0) {
  const u32 t = rfl(ld_dev(myws + OFF_TMO));
  const bool late = (__builtin_amdgcn_s_memrealtime() - t0) > DEADLINE;
  if (t == seq) return true;
  if (late) {
    st_dev(myws + OFF_TMO, seq);
    return true;
  }
  return false;
}

#define STAMP(slot) do {} while (0)
#define PVAL(slot, v) do {} while (0)
#define PNOW() 0ull

__device__ __forceinline__ int grow_of(int p, int a) {
  if (a < 12) return ((p ^ (2 + (a >> 1))) << 2) + ((p & 1) << 1) + (a & 1);
  return (p << 2) + (a - 12);
}

constexpr u32 K_X = 1, K_W = 2, K_FWD = 3, K_OUT = 4, K_END = 5, K_PULL = 6;

__device__ __forceinline__ u32 late_rfl(u32 v) {
  u32 s;
  asm volatile("v_readfirstlane_b32 %0, %1" : "=s"(s) : "v"(v));
  return s;
}

__device__ __forceinline__ void comm_role(const Params& P, char* smem, u32 role, u32 ctr_v, u64 t0) {
  const int tid = threadIdx.x, lane = tid & 63;
  const int wave = rfl(tid >> 6);
  const int r = P.rank;
  const int d = (int)(role >> 3) & 7;
  const int w = (int)role & 7;
  const u32 p = (u32)(r ^ d) & 7u;
  char* myws = P.ws[r];
  char* pws = P.ws[p];
  const int cls = d == 1 ? 0 : 1 + (d & 1);
  const int NP = 4 * P.nunits[cls];
  const int LA = P.la, NDIR = P.ndir;

  volatile LDS_AS u4* desc = (volatile LDS_AS u4*)smem;
  volatile LDS_AS u32* cnt = (volatile LDS_AS u32*)(smem + (MAXP + 4) * 16);
  volatile LDS_AS u32* fail = cnt + MAXP + 3;
  volatile LDS_AS u32* npush = cnt + MAXP + 2;
  const int G = P.grp, NS = (LA + G - 1) / G;
  for (int i = tid; i < MAXP + 4; i += THREADS) {
    u4 e = u4{0u, 0u, 0u, 0u};
    if (i < NS * G) {
      const int tk = w + WPL * i, a = tk >> 2, piece = tk & 3;
      const u64 sa = (u64)P.x + (u32)grow_of((int)p, a) * PANEL + piece * 16384u;
      const u64 da = (u64)pws + OFF_DATA + REL_XG + (u32)r * SRC_BYTES + (u32)a * PANEL + piece * 16384u;
      e = u4{(u32)sa, (u32)da, (u32)(sa >> 32) | (u32)(da >> 32) << 16, (OFF_FX + (u32)((a * 8 + r) * 4 + piece) * 4u) << 8 | K_X};
    }
    desc[i] = e;
    cnt[i] = 0u;
  }
  RAW_BARRIER();
  if (wave == 0) STAMP(0);

  if (wave == 9) {
    const u32 seq = late_rfl(ctr_v) / (u32)GRID + 1u;
    const __amdgpu_buffer_rsrc_t rs_pws = mk_rsrc(pws, WS_NEED);
    const u32 doff = OFF_DONE + (u32)(d * WPL + w) * 64u;
    bool pre = d == 1;
    int t = 0;
    for (;; ++t) {
      u32 k, np;
      for (;;) {
        np = rfl(*npush);
        k = rfl(((volatile LDS_AS u32*)(desc + t))[3]);
        if (!pre && np != 0u && (u32)t + 2u >= np) { st_sys(pws + doff + 4, seq); pre = true; }
        if (np == (u32)t + 1u || (k != 0u && rfl(cnt[t]) == 8u)) break;
        __builtin_amdgcn_s_sleep(1);
      }
      if (np == (u32)t + 1u) break;
      if ((k >> 8) != 0u) __builtin_amdgcn_raw_buffer_store_b32(seq, rs_pws, 0, k >> 8, AUX_SYS);
    }
    STAMP(1);
    if (d != 1) {
      if (rfl(*fail) == 0u) st_sys(pws + doff, seq);
      bool ok = true;
      while (rfl(ld_sys(myws + doff)) < seq) {
        if (give_up(myws, seq, t0)) { ok = false; break; }
        __builtin_amdgcn_s_sleep(1);
      }
      for (;; ++t) {
        u32 k;
        while ((k = rfl(((volatile LDS_AS u32*)(desc + t))[3])) == 0u) __builtin_amdgcn_s_sleep(2);
        if (k == K_END) break;
      }
      if (!ok || rfl(*fail) != 0u)
        st_sys(P.out[r] + (u32)((r >> 1) * NB) * 512u, 0x7fc00000u);
      STAMP(3);
    }
    if (lane == 0) __hip_atomic_fetch_add((GLB_AS u32*)(myws + OFF_CTR), 1u, __ATOMIC_RELAXED, __HIP_MEMORY_SCOPE_AGENT);
  } else if (wave == 8) {
    const __amdgpu_buffer_rsrc_t rs_my = mk_rsrc(myws, WS_NEED);
    u64 stall = 0;
    const u32 call = late_rfl(ctr_v) / (u32)GRID;
    const u32 seq = call + 1u, parbase = (call & 1u) * PAR_BYTES;
    const int pull_from = P.pull_from[cls];
    u32 woff = (lane >= pull_from ? OFF_FI : OFF_FO) + (u32)d * 1024u + (u32)lane * 16u;
    if (d == 1) {
      const int fi = lane < 3 * (NB - NDIR) ? lane : 0;
      const int k = fi / 3, i = fi - 3 * k;
      woff = OFF_FW + (u32)(((NDIR + k) * 8 + (r ^ (2 * i + 2 + (k & 1)))) * 16);
    }
    const u32 tab = P.ord[cls][lane & 31];
    u64 rdy = 0;
    auto poll = [&]() {
      const u4 f = __builtin_amdgcn_raw_buffer_load_b128(rs_my, woff, 0, AUX_SYS);
      rdy |= __ballot(f.x >= seq && f.y >= seq && f.z >= seq && f.w >= seq);
    };
    auto post = [&](int s, u32 e0, u32 e1, u32 e2, u32 e3) {
      volatile LDS_AS u32* dp = (volatile LDS_AS u32*)(desc + s);
      if (lane == 0) { dp[0] = e0; dp[1] = e1; dp[2] = e2; dp[3] = e3; }
    };
    const int NT = (NP + G * WPL - 1) / (G * WPL) * WPL;
    const u32 tkbase = call * (u32)(NT - WPL * NS + WPL) - (u32)(WPL * NS);
    char* tkptr = myws + OFF_TK + (u32)d * 128u;
    bool dead = false, pulling = false;
    int s = NS * G, tj = 0, g = G - 1;
    u64 rtt = 0;
    for (;;) {
      if (++g == G) {
        g = 0;
        if (!dead) while (rfl(cnt[s - LA]) == 0u) __builtin_amdgcn_s_sleep(1);
        const u64 tr0 = PNOW();
        const u4 f = __builtin_amdgcn_raw_buffer_load_b128(rs_my, woff, 0, AUX_SYS);
        u32 old = 0;
        if (lane == 0) old = __hip_atomic_fetch_add((GLB_AS u32*)tkptr, 1u, __ATOMIC_RELAXED, __HIP_MEMORY_SCOPE_AGENT);
        tj = (int)(rfl(old) - tkbase);
        rdy |= __ballot(f.x >= seq && f.y >= seq && f.z >= seq && f.w >= seq);
        rtt += PNOW() - tr0;
        if ((u32)tj >= (u32)NT) break;
      }
      const int tk = (tj / WPL) * G * WPL + tj % WPL + g * WPL;
      if (dead || tk >= NP) continue;
      const int pos = tk >> 2, piece = tk & 3;
      const u32 ent = ((u32)__builtin_amdgcn_readlane((int)tab, pos >> 2) >> (8 * (pos & 3))) & 255u;
      u32 kind = (ent >> 6) + 1u;
      int idx = (int)(ent & 63u), q = r, wi = -1;
      if (kind == K_OUT) {
        wi = idx;
        if (idx >= pull_from) {
          kind = K_PULL;
          if (!pulling) {
            pulling = true;
            *npush = (u32)s + 1u;
            const char* gate = myws + OFF_DONE + (u32)(d * WPL + w) * 64u + 4u;
            while (rfl(ld_sys(gate)) < seq) {
              if (give_up(myws, seq, t0)) break;
              __builtin_amdgcn_s_sleep(1);
            }
          }
        }
      }
      if (kind == K_FWD) {
        wi = idx;
        const int kk = idx / 3, i = idx - 3 * kk;
        q = r ^ (2 * i + 2 + (kk & 1));
        idx = NDIR + kk;
      }
      u32 e_src, e_dst, e_flag;
      if (kind == K_X) {
        e_src = (u32)grow_of((int)p, idx) * PANEL + piece * 16384u;
        e_dst = OFF_DATA + REL_XG + (u32)r * SRC_BYTES + (u32)idx * PANEL + piece * 16384u;
        e_flag = OFF_FX + (u32)((idx * 8 + r) * 4 + piece) * 4u;
      } else if (kind == K_OUT) {
        const int b = idx >> 1, tt = idx & 1;
        e_src = OFF_ST + (u32)(((d - 2) * 2 + tt) * NB + b) * 65536u + piece * 16384u;
        e_dst = (u32)((2 * (r & 1) + tt) * 128 + piece * 32) * (NG * 4u) + (u32)((r >> 1) * NB + b) * 512u;
        e_flag = 0u;
      } else if (kind == K_PULL) {
        const int b = idx >> 1, tt = idx & 1;
        e_src = OFF_ST + (u32)(((d - 2) * 2 + tt) * NB + b) * 65536u + piece * 16384u;
        e_dst = (u32)((2 * (p & 1) + tt) * 128 + piece * 32) * (NG * 4u) + ((p >> 1) * NB + (u32)b) * 512u;
        e_flag = 0u;
      } else {
        e_dst = OFF_DATA + REL_WG + (u32)q * SRC_BYTES + (u32)idx * PANEL + piece * 16384u;
        e_flag = OFF_FW + (u32)((idx * 8 + q) * 4 + piece) * 4u;
        e_src = kind == K_W ? (u32)piece * 128u * NG + ((p >> 1) * NB + (u32)idx) * 128u : e_dst + parbase;
      }
      if (wi >= 0 && !((rdy >> wi) & 1ull)) {
        const u64 tw0 = PNOW();
        for (;;) {
          poll();
          if ((rdy >> wi) & 1ull) break;
          if (give_up(myws, seq, t0)) { dead = true; break; }
          __builtin_amdgcn_s_sleep(2);
        }
        stall += PNOW() - tw0;
        if (dead) { *fail = 1u; if (!pulling) *npush = (u32)s + 1u; post(s, 0u, 0u, 0u, K_END); continue; }
      }
      asm volatile("" ::: "memory");
      {
        const u64 sa = (u64)(kind == K_X ? P.x : (kind == K_W ? P.w : (kind == K_PULL ? (const char*)pws : (const char*)myws))) + e_src;
        const u64 da = (u64)(kind == K_OUT ? P.out[p] : (kind == K_PULL ? P.out[r] : pws)) + e_dst;
        post(s, (u32)sa, (u32)da, (u32)(sa >> 32) | (u32)(da >> 32) << 16, e_flag << 8 | kind);
      }
      ++s;
    }
    if (!dead) { if (!pulling) *npush = (u32)s + 1u; post(s, 0u, 0u, 0u, K_END); }
    PVAL(4, stall);
    PVAL(5, (u64)s);
    PVAL(6, rtt);
  } else {
    const u32 v_lin = (u32)wave * 2048u + (u32)lane * 16u;
    const u32 v_str = (u32)(wave * 16 + (lane >> 3)) * NG + (u32)(lane & 7) * 16u;
    const u32 v_out = (u32)(wave * 4 + (lane >> 5)) * (NG * 4u) + (u32)(lane & 31) * 16u;
    const u32 cnt0 = (u32)(uintptr_t)cnt;
    struct D { u32 slo, shi, dlo, dhi, sstep, kind, vs, vd; };
    auto fetch = [&](int t) -> D {
      const u4 e = desc[t];
      D c;
      c.slo = rfl(e.x); c.dlo = rfl(e.y);
      const u32 h = rfl(e.z);
      c.shi = h & 0xffffu; c.dhi = h >> 16;
      c.kind = rfl(e.w) & 255u;
      c.sstep = c.kind == K_W ? 8u * NG : 1024u;
      c.vs = c.kind == K_W ? v_str : v_lin;
      c.vd = c.kind >= K_OUT ? v_out : v_lin;
      asm volatile("" :: "s"(c.slo), "s"(c.shi), "s"(c.dlo), "s"(c.dhi), "s"(c.sstep), "v"(c.vs), "v"(c.vd));
      return c;
    };
    auto arrive = [&](int t) {
      u64 keep;
      asm volatile("s_mov_b64 %0, exec\n\ts_mov_b64 exec, 1\n\tds_add_u32 %1, %2\n\ts_mov_b64 exec, %0"
                   : "=&s"(keep) : "v"(cnt0 + 4u * (u32)t), "v"(1u) : "memory");
    };
    auto load_step = [&](const D& e, int u) -> u4 {
      return __builtin_amdgcn_raw_buffer_load_b128(mk_rsrc((const void*)((u64)e.shi << 32 | e.slo), 0x200000u), e.vs, u ? e.sstep : 0u, AUX_SYS);
    };
    auto store_step = [&](const D& e, u32 so, const u4& a) {
      __builtin_amdgcn_raw_buffer_store_b128(a, mk_rsrc((const void*)((u64)e.dhi << 32 | e.dlo), PAR_BYTES + 0x200000u), e.vd, so, AUX_SYS);
      asm volatile("s_nop 1" ::: "memory");
    };
    auto usable = [](const D& e) { return e.kind != 0u && e.kind < K_END; };
    int t = 0;
    u64 nruns = 0;
    for (;;) {
      ++nruns;
      D C = fetch(t);
      while (C.kind == 0u) { __builtin_amdgcn_s_sleep(1); C = fetch(t); }
      if (C.kind == K_END) break;
      u4 a0 = load_step(C, 0);
      if (C.kind == K_PULL) {
        D N = fetch(t + 1);
        for (;;) {
          const bool more = N.kind == K_PULL;
          D L = N;
          if (!more) { L.slo = C.dlo; L.shi = C.dhi; L.vs = C.vd; }
          asm volatile("" :: "s"(L.slo), "s"(L.shi), "v"(L.vs));
          VMCNT(1);
          const u4 a1 = load_step(C, 1);
          store_step(C, 0u, a0);
          D N2 = fetch(t + 2);
          VMCNT(1);
          a0 = load_step(L, 0);
          store_step(C, 2u * NG * 4u, a1);
          arrive(t);
          ++t;
          if (!more) break;
          if (N2.kind == 0u) N2 = fetch(t + 1);
          C = N;
          N = N2;
        }
        VMCNT(0);
        continue;
      }
      SCHED_FENCE();
      u4 a1 = load_step(C, 1);
      SCHED_FENCE();
      const u32 paroff = ((late_rfl(ctr_v) / (u32)GRID) & 1u) * PAR_BYTES;
      D N = fetch(t + 1);
      const int tfirst = t;
      for (;;) {
        const bool more = usable(N);
        D L = more ? N : C;
        const u32 so0 = C.kind == K_OUT ? 0u : paroff;
        const u32 so1 = so0 + (C.kind == K_OUT ? 2u * NG * 4u : 1024u);
        asm volatile("" :: "s"(L.slo), "s"(L.shi), "s"(L.sstep), "v"(L.vs), "s"(so0), "s"(so1));
        VMCNT(1);
        store_step(C, so0, a0);
        a0 = load_step(L, 0);
        if (t != tfirst) arrive(t - 1);
        D N2 = fetch(t + 2);
        VMCNT(1);
        store_step(C, so1, a1);
        a1 = load_step(L, 1);
        ++t;
        if (!more) break;
        if (N2.kind == 0u) N2 = fetch(t + 1);
        C = N;
        N = N2;
      }
      VMCNT(0);
      arrive(t - 1);
    }
    PVAL(40 + wave, nruns);
  }
}

__device__ __forceinline__ void glds16(__amdgpu_buffer_rsrc_t rs, u32 lds, u32 voff, u32 soff) {
  asm volatile("s_mov_b32 m0, %0\n\ts_nop 3\n\tbuffer_load_dwordx4 %1, %2, %3 offen lds"
               :: "s"(rfl(lds)), "v"(voff), "s"(rs), "s"(rfl(soff)) : "m0", "memory");
}

__device__ __forceinline__ void gemm_role(const Params& P, char* smem, u32 role, u32 seq, u32 parbase, u64 t0) {
  const int tid = threadIdx.x, lane = tid & 63;
  const int wave = rfl(tid >> 6);
  const int r = P.rank;
  char* myws = P.ws[r];
  const int xg = (int)blockIdx.x & 7, kx = (int)role;
  const int gpx = (int)(P.gpx[xg >> 2] >> (8 * (xg & 3))) & 255;
  const u32 lds0 = (u32)(uintptr_t)(LDS_AS char*)smem;

  const __amdgpu_buffer_rsrc_t rs_x = mk_rsrc(P.x, X_BYTES);
  const __amdgpu_buffer_rsrc_t rs_w = mk_rsrc(P.w, W_BYTES);
  const __amdgpu_buffer_rsrc_t rs_ws = mk_rsrc(myws, WS_NEED);
  const float scale = P.sx[0] * P.sw[0];

  u32 vX[2], vWs[2], vWo[2];
#pragma unroll
  for (int u = 0; u < 2; ++u) {
    const int row = wave * 16 + u * 8 + (lane >> 3), slot = lane & 7;
    const int cx = slot ^ ((row >> 1) & 7);
    const int cw = slot ^ (((row >> 1) & 3) | (((row >> 4) & 1) << 2));
    vX[u] = (u32)row * KS + (u32)cx * 16u;
    vWs[u] = (u32)row * 128u + (u32)cw * 16u;
    vWo[u] = (u32)row * NG + (u32)cw * 16u;
  }
  const int wm = wave >> 2, wn = wave & 3, g = lane >> 4, l15 = lane & 15;
  const int aX0 = (wm * 64 + l15) * 128 + ((g ^ (l15 >> 1)) * 16);
  const int aX1 = aX0 ^ 64;
  const int fw = ((l15 >> 2) & 3) | ((g & 1) << 2);
  const int aW0 = 16384 + (16 * g + (l15 >> 1)) * 128 + (((2 * wn) ^ fw) * 16) + (l15 & 1) * 8;
  const int aW1 = aW0 ^ 16;
  volatile LDS_AS u32* ctrl = (volatile LDS_AS u32*)(smem + LDS_CTRL);

  for (int e = kx; e < 2 * NB; e += gpx) {
    const u32 ent = (P.tiles[xg & 1][e >> 2] >> (8 * (e & 3))) & 255u;
    const int b = (int)(ent & 31u);
    const int a = (ent >> 5) == 0u ? xg : ((((b ^ xg) & 1) == 0 ? 8 : 12) + (xg >> 1));
    const int NDIR = P.ndir;
    u32 pref;
    if (b < NDIR) pref = 0x17654320u;
    else pref = ((b - NDIR) & 1) ? 0x64217530u : 0x75316420u;
    u32 ordd = 0;
    const u32 sX_own = (u32)grow_of(r, a) * PANEL;
    const u32 sW_own = (u32)((r >> 1) * NB + b) * 128u;
    const u32 sX_ws = parbase + REL_XG + (u32)a * PANEL;
    const u32 sW_ws = parbase + REL_WG + (u32)b * PANEL;
    const u32 foff = lane < 32 ? OFF_FX + (u32)a * 128u + (u32)lane * 4u : OFF_FW + (u32)b * 128u + (u32)(lane - 32) * 4u;

    v4f acc[2][4];
#pragma unroll
    for (int nb = 0; nb < 2; ++nb)
#pragma unroll
      for (int mb = 0; mb < 4; ++mb) acc[nb][mb] = v4f{0.f, 0.f, 0.f, 0.f};

    auto issue = [&](int kt, int stage) {
      const int dq = (ordd >> (4 * (kt >> 2))) & 7;
      const u32 q = (u32)(r ^ dq);
      const u32 kb = (u32)(kt & 3);
      const u32 ldsb = lds0 + (u32)stage * STAGE + (u32)wave * 2048u;
      if (dq == 0) {
        const u32 sx = sX_own + kb * 128u, sw = sW_own + kb * 128u * NG;
        glds16(rs_x, ldsb, vX[0], sx);
        glds16(rs_x, ldsb + 1024u, vX[1], sx);
        glds16(rs_w, ldsb + 16384u, vWo[0], sw);
        glds16(rs_w, ldsb + 16384u + 1024u, vWo[1], sw);
      } else {
        const u32 sx = sX_ws + q * SRC_BYTES + kb * 128u, sw = sW_ws + q * SRC_BYTES + kb * 16384u;
        glds16(rs_ws, ldsb, vX[0], sx);
        glds16(rs_ws, ldsb + 1024u, vX[1], sx);
        glds16(rs_ws, ldsb + 16384u, vWs[0], sw);
        glds16(rs_ws, ldsb + 16384u + 1024u, vWs[1], sw);
      }
    };
    auto compute = [&](int stage) {
      const LDS_AS char* sb = (const LDS_AS char*)smem + stage * STAGE;
      v8i xf[4], wf[2];
#pragma unroll
      for (int nb = 0; nb < 2; ++nb) {
        const LDS_AS char* pw = sb + (nb ? aW1 : aW0);
        const v2i t0_ = __builtin_amdgcn_ds_read_tr8_b64_v2i32((LDS_AS v2i*)(pw));
        const v2i t1_ = __builtin_amdgcn_ds_read_tr8_b64_v2i32((LDS_AS v2i*)(pw + 1024));
        const v2i t2_ = __builtin_amdgcn_ds_read_tr8_b64_v2i32((LDS_AS v2i*)(pw + 8192));
        const v2i t3_ = __builtin_amdgcn_ds_read_tr8_b64_v2i32((LDS_AS v2i*)(pw + 9216));
        wf[nb] = v8i{t0_.x, t0_.y, t1_.x, t1_.y, t2_.x, t2_.y, t3_.x, t3_.y};
      }
#pragma unroll
      for (int mb = 0; mb < 4; ++mb) {
        const v4i lo = *(const LDS_AS v4i*)(sb + aX0 + mb * 2048);
        const v4i hi = *(const LDS_AS v4i*)(sb + aX1 + mb * 2048);
        xf[mb] = v8i{lo.x, lo.y, lo.z, lo.w, hi.x, hi.y, hi.z, hi.w};
      }
#pragma unroll
      for (int mb = 0; mb < 4; ++mb)
#pragma unroll
        for (int nb = 0; nb < 2; ++nb)
          acc[nb][mb] = __builtin_amdgcn_mfma_scale_f32_16x16x128_f8f6f4(wf[nb], xf[mb], acc[nb][mb], 1, 0, 0, 0, 0, 0);
    };

    u32 taken = 0;
    int nass = 0;
    auto assign = [&](u32 f) {
      const u64 m = __ballot((((lane & 31) >> 2) == r) || f >= seq);
      const u32 m32 = (u32)m & (u32)(m >> 32);
      for (int i = 0; i < 8; ++i) {
        const u32 dq = (pref >> (4 * i)) & 7u;
        if (!((taken >> dq) & 1u) && ((m32 >> (4u * ((u32)r ^ dq))) & 15u) == 15u) {
          taken |= 1u << dq;
          ordd |= (8u | dq) << (4 * nass);
          ++nass;
        }
      }
    };
    auto wait_slices = [&](int n) {
      for (;;) {
        assign(__builtin_amdgcn_raw_buffer_load_b32(rs_ws, foff, 0, AUX_SYS));
        if (nass >= n) break;
        if (give_up(myws, seq, t0)) {
          while (nass < 8) { ordd |= 8u << (4 * nass); ++nass; }
          ctrl[1] = 1u;
          break;
        }
        __builtin_amdgcn_s_sleep(4);
      }
    };
    auto need = [&](int sl) {
      if ((ordd >> (4 * sl)) & 8u) return;
      if (wave == 0) { wait_slices(sl + 1); ctrl[0] = ordd; }
      else {
        ordd = rfl(ctrl[0]);
        while (!((ordd >> (4 * sl)) & 8u)) { __builtin_amdgcn_s_sleep(1); ordd = rfl(ctrl[0]); }
      }
      asm volatile("" ::: "memory");
    };

    if (wave == 0) {
      STAMP(8 * (e / gpx) + 0);
      ctrl[1] = 0u;
      wait_slices(2);
      __builtin_amdgcn_fence(__ATOMIC_ACQUIRE, "");
      VMCNT(0);
      ctrl[0] = ordd;
      STAMP(8 * (e / gpx) + 1);
    }
    RAW_BARRIER();
    ordd = rfl(ctrl[0]);
    issue(0, 0); issue(1, 1); issue(2, 2);
    for (int sl = 0; sl < 8; ++sl) {
      const int kt = sl * 4;
      const bool last = sl == 7;
      VMCNT(8); RAW_BARRIER();
      u32 pf;
      {
        const u32 po = (wave == 0 && nass < 8) ? foff : 0xfffffff0u;
        asm volatile("buffer_load_dword %0, %1, %2, 0 offen sc0 sc1" : "=v"(pf) : "v"(po), "s"(rs_ws) : "memory");
      }
      issue(kt + 3, 3);
      compute(0);
      VMCNT(8); RAW_BARRIER();
      if (!last) { need(sl + 1); issue(kt + 4, 0); }
      compute(1);
      if (!last) VMCNT(8); else VMCNT(4);
      RAW_BARRIER();
      if (!last) issue(kt + 5, 1);
      compute(2);
      if (!last) VMCNT(8); else VMCNT(0);
      asm volatile("" : "+v"(pf));
      if (wave == 0 && nass < 8) { assign(pf); ctrl[0] = ordd; }
      RAW_BARRIER();
      if (!last) issue(kt + 6, 2);
      compute(3);
    }
    if (wave == 0) STAMP(8 * (e / gpx) + 3);

    RAW_BARRIER();
    const bool bad = rfl(ctrl[1]) != 0u;
#pragma unroll
    for (int nb = 0; nb < 2; ++nb)
#pragma unroll
      for (int mb = 0; mb < 4; ++mb) {
        v4f o;
#pragma unroll
        for (int j = 0; j < 4; ++j) {
          const float y = acc[nb][mb][j] * scale;
          const float yc = fminf(fmaxf(y, -60.f), 60.f);
          o[j] = y * __builtin_amdgcn_rcpf(1.f + __expf(-yc));
          if (bad) o[j] = __builtin_nanf("");
        }
        const int m = wm * 64 + mb * 16 + l15, n4 = wn * 8 + nb * 4 + g;
        *(LDS_AS v4f*)((LDS_AS char*)smem + m * 512 + ((n4 ^ (m & 7)) * 16)) = o;
      }
    RAW_BARRIER();
    {
      const bool own = a >= 12;
      const __amdgpu_buffer_rsrc_t rs_o = mk_rsrc(own ? P.out[r] : myws, own ? OUT_BYTES : WS_NEED);
      const u32 rstride = own ? NG * 4u : 512u;
      const u32 so = own ? (u32)(a - 12) * 128u * (NG * 4u) + (u32)((r >> 1) * NB + b) * 512u : OFF_ST + (u32)(a * NB + b) * 65536u;
      const u32 vo = (u32)(wave * 16 + (lane >> 5)) * rstride + (u32)(lane & 31) * 16u;
      u4 v[8];
#pragma unroll
      for (int i = 0; i < 8; ++i) {
        const int row = (wave * 8 + i) * 2 + (lane >> 5), c = lane & 31;
        v[i] = *(const LDS_AS u4*)((const LDS_AS char*)smem + row * 512 + ((c ^ (row & 7)) * 16));
      }
#pragma unroll
      for (int i = 0; i < 8; ++i)
        __builtin_amdgcn_raw_buffer_store_b128(v[i], rs_o, vo, so + (u32)(2 * i) * rstride, AUX_SYS);
      VMCNT(0);
      RAW_BARRIER();
      if (!own && !bad && tid < 4) {
        const u32 dd = 2u + (u32)(a >> 1);
        char* fb = b * 2 + (a & 1) >= P.pull_from[1 + (dd & 1u)] ? P.ws[((u32)r ^ dd) & 7u] + OFF_FI : myws + OFF_FO;
        st_sys(fb + dd * 1024u + (u32)(b * 2 + (a & 1)) * 16u + (u32)tid * 4u, seq);
      }
    }
    if (wave == 0) STAMP(8 * (e / gpx) + 4);
  }
}

__global__ void __launch_bounds__(THREADS, 3) fused_gemm_rs(Params P) {
  extern __shared__ __attribute__((aligned(16))) char smem[];
  const u32 role = (P.role[blockIdx.x >> 2] >> (8 * (blockIdx.x & 3))) & 255u;
  const bool comm = role >= 128u;
  if (!comm && rfl(threadIdx.x >> 6) >= 8u) return;
  char* myws = P.ws[P.rank];
  const u64 t0 = __builtin_amdgcn_s_memrealtime();
  const u32 ctr_v = ld_dev(myws + OFF_CTR);
  if (comm) {
    comm_role(P, smem, role, ctr_v, t0);
  } else {
    const u32 call = rfl(ctr_v) / (u32)GRID;
    gemm_role(P, smem, role, call + 1u, OFF_DATA + (call & 1u) * PAR_BYTES, t0);
    if (threadIdx.x == 0)
      __hip_atomic_fetch_add((GLB_AS u32*)(myws + OFF_CTR), 1u, __ATOMIC_RELAXED, __HIP_MEMORY_SCOPE_AGENT);
  }
}

struct Tune { int nrel, la, ownx_after, own_delay, relay_start, grp, pull_from, aff, rep; };

void build_schedule(Params& P, const Tune& t) {
  const int ndir = NB - t.nrel;
  P.ndir = ndir;
  P.la = t.la;
  P.grp = t.grp;
  for (int cls = 0; cls < 3; ++cls) {
    unsigned char L[MAXU] = {};
    int n = 0, nw = 0;
    bool ownx = false;
    auto put = [&](u32 kind, int idx) { if (n < MAXU) L[n++] = (unsigned char)(((kind - 1u) << 6) | (u32)idx); };
    auto putw = [&](int b) {
      if (!ownx && nw >= t.ownx_after) { for (int a = 12; a < 16; ++a) put(K_X, a); ownx = true; }
      put(K_W, b); ++nw;
    };
    for (int a = 0; a < 12; ++a) put(K_X, a);
    if (cls == 0) {
      for (int b = 0; b < ndir; ++b) putw(b);
      for (int k = 0; k < t.nrel; ++k) {
        putw(ndir + k);
        for (int i = 0; i < 3; ++i) put(K_FWD, 3 * k + i);
      }
    } else {
      const int par = cls - 1;
      const int nr = (t.nrel + 1 - par) / 2;
      const int lead = t.relay_start < ndir ? t.relay_start : ndir;
      for (int b = 0; b < lead; ++b) putw(b);
      const int rest = ndir - lead, tot = rest + nr;
      for (int j = 0; j < tot; ++j) {
        const int r0 = j * nr / tot, r1 = (j + 1) * nr / tot;
        if (r1 > r0) putw(ndir + 2 * r0 + par); else putw(lead + j - r0);
      }
    }
    if (!ownx) for (int a = 12; a < 16; ++a) put(K_X, a);
    int po = 2 * t.pull_from;
    while (po < 2 * NB && ((n + po) * 4) % (t.grp * WPL) != 0) ++po;
    P.pull_from[cls] = po;
    if (cls != 0) for (int o = 0; o < 2 * NB; ++o) put(K_OUT, o);
    P.nunits[cls] = n;
    for (int i = 0; i < MAXU / 4; ++i)
      P.ord[cls][i] = (u32)L[4 * i] | (u32)L[4 * i + 1] << 8 | (u32)L[4 * i + 2] << 16 | (u32)L[4 * i + 3] << 24;
  }
  {
    static_assert(WPL == 8, "role byte");
    unsigned char R[GRID];
    int used[8] = {};
    for (int i = 0; i < GRID; ++i) R[i] = 0;
    static const unsigned char lowhalf[8] = {0x4c, 0x70, 0x16, 0x2a, 0x2a, 0x16, 0x70, 0x4c};
    for (int w = 0; w < WPL; ++w)
      for (int d = 1; d < 8; ++d) {
        int x = (w * 7 + d - 1) & 7;
        if (t.aff == 1) x = (((lowhalf[P.rank] >> d) & 1) ? 0 : 4) + ((w + d) & 3);
        if (t.aff == 2) x = (((lowhalf[P.rank] >> d) & 1) ? 4 : 0) + ((w + d) & 3);
        R[x + 8 * used[x]++] = (unsigned char)(128 | d << 3 | w);
      }
    P.gpx[0] = P.gpx[1] = 0;
    for (int x = 0; x < 8; ++x) {
      for (int j = used[x]; j < GRID / 8; ++j) R[x + 8 * j] = (unsigned char)(j - used[x]);
      P.gpx[x >> 2] |= (u32)(GRID / 8 - used[x]) << (8 * (x & 3));
    }
    for (int i = 0; i < GRID / 4; ++i)
      P.role[i] = (u32)R[4 * i] | (u32)R[4 * i + 1] << 8 | (u32)R[4 * i + 2] << 16 | (u32)R[4 * i + 3] << 24;
  }
  for (int xp = 0; xp < 2; ++xp) {
    unsigned char T[2 * NB];
    int n = 0;
    for (int key = 0; key < NB + t.own_delay; ++key) {
      if (key < NB) {
        T[n++] = (unsigned char)key;
        if (((key ^ xp) & 1) == 0) T[n++] = (unsigned char)(32 | key);
      }
      const int ob = key - t.own_delay;
      if (ob >= 0 && ob < NB && ((ob ^ xp) & 1) != 0) T[n++] = (unsigned char)(32 | ob);
    }
    for (int i = 0; i < 16; ++i)
      P.tiles[xp][i] = (u32)T[4 * i] | (u32)T[4 * i + 1] << 8 | (u32)T[4 * i + 2] << 16 | (u32)T[4 * i + 3] << 24;
  }
}

}

extern "C" void kernel_launch(int rank, int nranks, ncclComm_t comm,
                              hipStream_t stream,
                              void* const* d_in, const int* in_sizes, int n_in,
                              void* d_out, int out_size,
                              const int* in_dtypes, int out_dtype,
                              void* const* d_out_all,
                              void* const* d_ws_all, size_t ws_bytes) {
  (void)comm; (void)in_dtypes; (void)out_dtype;
  if (nranks != NR || n_in != 4 || rank < 0 || rank >= NR || ws_bytes < (size_t)WS_NEED ||
      in_sizes[0] != (int)X_BYTES || in_sizes[1] != (int)W_BYTES || out_size != (int)(OUT_BYTES / 4)) {
    fprintf(stderr, "kernel_launch: unexpected problem shape\n");
    return;
  }
  static bool attr_set[NR] = {};
  if (!attr_set[rank]) {
    HIP_CHECK(hipFuncSetAttribute((const void*)fused_gemm_rs, hipFuncAttributeMaxDynamicSharedMemorySize, LDS_BYTES));
    attr_set[rank] = true;
  }
  Params P;
  P.x = (const char*)d_in[0];
  P.w = (const char*)d_in[1];
  P.sx = (const float*)d_in[2];
  P.sw = (const float*)d_in[3];
  for (int i = 0; i < NR; ++i) { P.ws[i] = (char*)d_ws_all[i]; P.out[i] = (char*)d_out_all[i]; }
  P.out[rank] = (char*)d_out;
  P.rank = rank;
  Tune t = {14, 4, 40, 32, 4, 2, 20, 0, 0};
  build_schedule(P, t);
  hipLaunchKernelGGL(fused_gemm_rs, dim3(GRID), dim3(THREADS), LDS_BYTES, stream, P);
  HIP_CHECK(hipGetLastError());
}
```

```cpp
#include <hip/hip_runtime.h>
#include <rccl/rccl.h>
#include <cstdio>
#include <cstdint>

#define HIP_CHECK(expr) do { hipError_t _e = (expr); if (_e != hipSuccess) { \
    fprintf(stderr, "HIP error %s at %s:%d\n", hipGetErrorString(_e), __FILE__, __LINE__); \
    return; } } while (0)

namespace {

typedef unsigned u32;
typedef unsigned long long u64;
typedef unsigned u4 __attribute__((ext_vector_type(4)));
typedef int v2i __attribute__((ext_vector_type(2)));
typedef int v4i __attribute__((ext_vector_type(4)));
typedef int v8i __attribute__((ext_vector_type(8)));
typedef float v4f __attribute__((ext_vector_type(4)));
#define LDS_AS __attribute__((address_space(3)))
#define GLB_AS __attribute__((address_space(1)))

constexpr int NR = 8;
constexpr int MG = 4096, KG = 4096, NG = 16384;
constexpr int KS = KG / NR;
constexpr u32 PANEL = 128 * KS;
constexpr int NA = 16;
constexpr int NB = 32;
#define WPL_CFG 8
constexpr int WPL = WPL_CFG;
constexpr int NCOMM = 7 * WPL;
constexpr int GRID = 256;
constexpr int NGEMM = GRID - NCOMM;
constexpr int GPX = NGEMM / 8;
constexpr int THREADS = 640;

#define NWK_CFG 6
constexpr int NWK = NWK_CFG;
constexpr int NS = 3;
constexpr int MAXU = 128;
constexpr int NSLOT = 512;

constexpr u32 OFF_CTR = 0, OFF_TMO = 128, OFF_ZERO = 256;
constexpr u32 OFF_TK = 1024;
constexpr u32 OFF_FX = 4096;
constexpr u32 OFF_FW = 8192;
constexpr u32 OFF_FO = 16384;
constexpr u32 OFF_DONE = 24576;
constexpr u32 OFF_FI = 32768;
constexpr u32 OFF_DBG = 512 << 10;
constexpr u32 OFF_ST = 1 << 20;
constexpr u32 OFF_DATA = OFF_ST + 12 * NB * 65536;
constexpr u32 SRC_BYTES = (NA + NB) * PANEL;
constexpr u32 REL_XG = 0;
constexpr u32 REL_WG = NA * PANEL;
constexpr u32 PAR_BYTES = 8 * SRC_BYTES;
constexpr u32 WS_NEED = OFF_DATA + 2 * PAR_BYTES;

constexpr u32 X_BYTES = MG * KS;
constexpr u32 W_BYTES = KS * NG;
constexpr u32 OUT_BYTES = (MG / NR) * NG * 4;

constexpr int STAGE = 32768, NST = 4;
constexpr int LDS_CTRL = NST * STAGE;
constexpr int LDS_BYTES = LDS_CTRL + 256;

constexpr u64 DEADLINE = 50000000ull;

constexpr int AUX_SYS = 17;
constexpr int AUX_DEV = 16;

struct Params {
  const char* x;
  const char* w;
  char* wsd[NR];
  int rank;
  int ndir;
  int pull_from[3];
  int nunits[3];
  int margin[2];
  int pre_adv;
  const float* sx;
  const float* sw;
  char* outd[NR];
  u32 ord[3][MAXU / 4];
  u32 tiles[2][16];
};

__device__ __forceinline__ u32 rfl(u32 v) { return __builtin_amdgcn_readfirstlane(v); }
__device__ __forceinline__ __amdgpu_buffer_rsrc_t mk_rsrc(const void* p, u32 bytes) {
  return __builtin_amdgcn_make_buffer_rsrc((void*)p, (short)0, (int)bytes, 0x00020000);
}
__device__ __forceinline__ u32 ld_sys(const char* p) {
  return __hip_atomic_load((const GLB_AS u32*)p, __ATOMIC_RELAXED, __HIP_MEMORY_SCOPE_SYSTEM);
}
__device__ __forceinline__ void st_sys(char* p, u32 v) {
  __hip_atomic_store((GLB_AS u32*)p, v, __ATOMIC_RELAXED, __HIP_MEMORY_SCOPE_SYSTEM);
}
__device__ __forceinline__ u32 ld_dev(const char* p) {
  return __hip_atomic_load((const GLB_AS u32*)p, __ATOMIC_RELAXED, __HIP_MEMORY_SCOPE_AGENT);
}
__device__ __forceinline__ void st_dev(char* p, u32 v) {
  __hip_atomic_store((GLB_AS u32*)p, v, __ATOMIC_RELAXED, __HIP_MEMORY_SCOPE_AGENT);
}
#define VMCNT(n) asm volatile("s_waitcnt vmcnt(" #n ")" ::: "memory")
#define DLY_PUSH 0
#define DLY_PULL 0
#define STR2(x) #x
#define STR(x) STR2(x)
#define VMCNT1_PUSH() asm volatile("s_waitcnt vmcnt(1)\n\ts_sleep " STR(DLY_PUSH) ::: "memory")
#define VMCNT1_PULL() asm volatile("s_waitcnt vmcnt(1)\n\ts_sleep " STR(DLY_PULL) ::: "memory")
#define SCHED_FENCE() do { asm volatile("" ::: "memory"); __builtin_amdgcn_sched_barrier(0); } while (0)
#define RAW_BARRIER() do { asm volatile("s_waitcnt lgkmcnt(0)" ::: "memory"); __builtin_amdgcn_s_barrier(); asm volatile("" ::: "memory"); } while (0)

__device__ __forceinline__ bool give_up(char* myws, u32 seq, u64 t0, u32& round) {
  if ((++round & 7u) != 0u) return false;
  const u32 t = rfl(ld_dev(myws + OFF_TMO));
  const bool late = (__builtin_amdgcn_s_memrealtime() - t0) > DEADLINE;
  if (t == seq) return true;
  if (late) {
    st_dev(myws + OFF_TMO, seq);
    return true;
  }
  return false;
}

#define STAMP(slot) do {} while (0)
#define PVAL(slot, v) do {} while (0)
#define PNOW() 0ull

__device__ __forceinline__ int grow_of(int p, int a) {
  if (a < 12) return ((p ^ (2 + (a >> 1))) << 2) + ((p & 1) << 1) + (a & 1);
  return (p << 2) + (a - 12);
}

constexpr u32 K_X = 1, K_W = 2, K_FWD = 3, K_OUT = 4, K_END = 5, K_PULL = 6;

__device__ __forceinline__ u32 late_rfl(u32 v) {
  u32 s;
  asm volatile("v_readfirstlane_b32 %0, %1" : "=s"(s) : "v"(v));
  return s;
}

__device__ __forceinline__ void comm_role(const Params& P, char* smem, u64 t0) {
  const int tid = threadIdx.x, lane = tid & 63;
  const int wave = rfl(tid >> 6);
  const int w = (int)blockIdx.x / 7;
  const int d = (int)blockIdx.x - 7 * w + 1;
  const int r = P.rank;
  const char* px = P.x;
  char* myws = P.wsd[0];
  char* pws = P.wsd[d];
  asm volatile("" :: "s"(r), "s"(px), "s"(myws), "s"(pws));
  const u32 ctr_v = ld_dev(myws + OFF_CTR);
  const u32 p = (u32)(r ^ d) & 7u;
  u32 round = 0;
  u4 a0, a1;
  {
    const __amdgpu_buffer_rsrc_t rs = mk_rsrc(px + (u32)grow_of((int)p, w >> 2) * PANEL + (w & 3) * 16384u, 16384u);
    const u32 vo = (u32)(wave & 7) * 2048u + (u32)lane * 16u;
    a0 = __builtin_amdgcn_raw_buffer_load_b128(rs, vo, 0, AUX_SYS);
    a1 = __builtin_amdgcn_raw_buffer_load_b128(rs, vo, 1024, AUX_SYS);
    SCHED_FENCE();
  }

  volatile LDS_AS u4* desc = (volatile LDS_AS u4*)smem;
  volatile LDS_AS u32* cnt = (volatile LDS_AS u32*)(smem + NSLOT * 16);
  volatile LDS_AS u32* fail = cnt + NSLOT - 1;
  volatile LDS_AS u32* npush = cnt + NSLOT - 2;
  volatile LDS_AS u32* cctr = cnt + NSLOT - 3;
  static_assert(NSLOT <= THREADS && WPL * NS <= 48 && 2 * NWK <= 8 * NS, "descriptor init");
  if (tid < NSLOT) {
    const int i = tid;
    u4 e = u4{0u, 0u, 0u, 0u};
    if (i < NS) {
      const int tk = w + WPL * i, a = tk >> 2, piece = tk & 3;
      const u64 sa = (u64)px + (((p ^ (u32)(2 + (a >> 1))) << 2) + ((p & 1u) << 1) + (u32)(a & 1)) * PANEL + piece * 16384u;
      const u64 da = (u64)pws + OFF_DATA + REL_XG + (u32)r * SRC_BYTES + (u32)a * PANEL + piece * 16384u;
      e = u4{(u32)sa, (u32)da, (u32)(sa >> 32) | (u32)(da >> 32) << 16, (OFF_FX + (u32)((a * 8 + r) * 4 + piece) * 4u) << 8 | K_X};
    }
    desc[i] = e;
    cnt[i] = i == NSLOT - 3 ? 2u * NWK : 0u;
  }
  RAW_BARRIER();
  if (wave == 0) STAMP(0);

  if (__builtin_expect(wave == 9, 0)) {
    const u32 seq = late_rfl(ctr_v) / (u32)GRID + 1u;
    const __amdgpu_buffer_rsrc_t rs_pws = mk_rsrc(pws, WS_NEED);
    const u32 doff = OFF_DONE + (u32)(d * WPL + w) * 64u;
    int t = 0;
    for (;; ++t) {
      u32 k, np;
      for (;;) {
        np = rfl(*npush);
        k = rfl(((volatile LDS_AS u32*)(desc + t))[3]);
        if (np == (u32)t + 1u || (k != 0u && rfl(cnt[t]) == 8u)) break;
        __builtin_amdgcn_s_sleep(1);
      }
      if (np == (u32)t + 1u) break;
      if ((k >> 8) != 0u) __builtin_amdgcn_raw_buffer_store_b32(seq, rs_pws, 0, k >> 8, AUX_SYS);
    }
    STAMP(1);
    if (d != 1) {
      if (rfl(*fail) == 0u) st_sys(pws + doff, seq);
      bool ok = true;
      while (rfl(ld_sys(myws + doff)) < seq) {
        if (give_up(myws, seq, t0, round)) { ok = false; break; }
        __builtin_amdgcn_s_sleep(1);
      }
      for (;; ++t) {
        u32 k;
        while ((k = rfl(((volatile LDS_AS u32*)(desc + t))[3])) == 0u) __builtin_amdgcn_s_sleep(2);
        if (k == K_END) break;
      }
      if (!ok || rfl(*fail) != 0u)
        st_sys(P.outd[0] + (u32)((r >> 1) * NB) * 512u, 0x7fc00000u);
      STAMP(3);
    }
    if (lane == 0) __hip_atomic_fetch_add((GLB_AS u32*)(myws + OFF_CTR), 1u, __ATOMIC_RELAXED, __HIP_MEMORY_SCOPE_AGENT);
  } else if (__builtin_expect(wave == 8, 0)) {
    const int cls = d == 1 ? 0 : 1 + (d & 1);
    const int NP = 4 * P.nunits[cls];
    const int NDIR = P.ndir;
    const __amdgpu_buffer_rsrc_t rs_my = mk_rsrc(myws, WS_NEED);
    u64 stall = 0;
    const u32 call = late_rfl(ctr_v) / (u32)GRID;
    const u32 seq = call + 1u, parbase = (call & 1u) * PAR_BYTES;
    const int pull_from = P.pull_from[cls];
    u32 woff = (lane >= pull_from ? OFF_FI : OFF_FO) + (u32)d * 1024u + (u32)lane * 16u;
    if (d == 1) {
      const int fi = lane < 3 * (NB - NDIR) ? lane : 0;
      const int k = fi / 3, i = fi - 3 * k;
      woff = OFF_FW + (u32)(((NDIR + k) * 8 + (r ^ (2 * i + 2 + (k & 1)))) * 16);
    }
    const u32 tab = P.ord[cls][lane & 31];
    u64 rdy = 0;
    auto poll = [&]() {
      const u4 f = __builtin_amdgcn_raw_buffer_load_b128(rs_my, woff, 0, AUX_SYS);
      rdy |= __ballot(f.x >= seq && f.y >= seq && f.z >= seq && f.w >= seq);
    };
    auto post = [&](int s, u32 e0, u32 e1, u32 e2, u32 e3) {
      volatile LDS_AS u32* dp = (volatile LDS_AS u32*)(desc + s);
      if (lane == 0) { dp[0] = e0; dp[1] = e1; dp[2] = e2; dp[3] = e3; }
    };
    auto post_end = [&](int s) {
      if (lane < 4) ((volatile LDS_AS u32*)(desc + s + lane))[3] = K_END;
    };
    const u32 tkbase = call * (u32)(NP - WPL * NS + WPL) - (u32)(WPL * NS);
    char* tkptr = myws + OFF_TK + (u32)d * 128u;
    const int margin = P.margin[d != 1];
    const int first_pull = d == 1 ? 0x10000 : NP - 4 * (2 * NB - pull_from), first_pre = first_pull - P.pre_adv;
    bool dead = false, pulling = false;
    int s = NS;
    u64 rtt = 0;
    for (;;) {
      if (!dead) while ((int)rfl(*cctr) + margin <= 8 * s) __builtin_amdgcn_s_sleep(1);
      const u64 tr0 = PNOW();
      const u4 f = __builtin_amdgcn_raw_buffer_load_b128(rs_my, woff, 0, AUX_SYS);
      u32 old = 0;
      if (lane == 0) old = __hip_atomic_fetch_add((GLB_AS u32*)tkptr, 1u, __ATOMIC_RELAXED, __HIP_MEMORY_SCOPE_AGENT);
      const int tk = (int)(rfl(old) - tkbase);
      rdy |= __ballot(f.x >= seq && f.y >= seq && f.z >= seq && f.w >= seq);
      rtt += PNOW() - tr0;
      if ((u32)tk >= (u32)NP) break;
      if ((u32)(tk - first_pre) < (u32)WPL) st_sys(pws + OFF_DONE + (u32)(d * WPL + tk - first_pre) * 64u + 4u, seq);
      if (dead) continue;
      if (tk >= first_pull && !pulling) { pulling = true; *npush = (u32)s + 1u; }
      if ((u32)(tk - first_pull) < (u32)WPL) {
        const char* gate = myws + OFF_DONE + (u32)(d * WPL + tk - first_pull) * 64u + 4u;
        while (rfl(ld_sys(gate)) < seq) {
          if (give_up(myws, seq, t0, round)) break;
          __builtin_amdgcn_s_sleep(1);
        }
      }
      const int pos = tk >> 2, piece = tk & 3;
      const u32 ent = ((u32)__builtin_amdgcn_readlane((int)tab, pos >> 2) >> (8 * (pos & 3))) & 255u;
      u32 kind = (ent >> 6) + 1u;
      int idx = (int)(ent & 63u), q = r, wi = -1;
      if (kind == K_OUT) {
        wi = idx;
        if (pulling) kind = K_PULL;
      }
      if (kind == K_FWD) {
        wi = idx;
        const int kk = idx / 3, i = idx - 3 * kk;
        q = r ^ (2 * i + 2 + (kk & 1));
        idx = NDIR + kk;
      }
      u32 e_src, e_dst, e_flag;
      if (kind == K_X) {
        e_src = (u32)grow_of((int)p, idx) * PANEL + piece * 16384u;
        e_dst = OFF_DATA + REL_XG + (u32)r * SRC_BYTES + (u32)idx * PANEL + piece * 16384u;
        e_flag = OFF_FX + (u32)((idx * 8 + r) * 4 + piece) * 4u;
      } else if (kind == K_OUT) {
        const int b = idx >> 1, tt = idx & 1;
        e_src = OFF_ST + (u32)(((d - 2) * 2 + tt) * NB + b) * 65536u + piece * 16384u;
        e_dst = (u32)((2 * (r & 1) + tt) * 128 + piece * 32) * (NG * 4u) + (u32)((r >> 1) * NB + b) * 512u;
        e_flag = 0u;
      } else if (kind == K_PULL) {
        const int b = idx >> 1, tt = idx & 1;
        e_src = OFF_ST + (u32)(((d - 2) * 2 + tt) * NB + b) * 65536u + piece * 16384u;
        e_dst = (u32)((2 * (p & 1) + tt) * 128 + piece * 32) * (NG * 4u) + ((p >> 1) * NB + (u32)b) * 512u;
        e_flag = 0u;
      } else {
        e_dst = OFF_DATA + REL_WG + (u32)q * SRC_BYTES + (u32)idx * PANEL + piece * 16384u;
        e_flag = OFF_FW + (u32)((idx * 8 + q) * 4 + piece) * 4u;
        e_src = kind == K_W ? (u32)piece * 128u * NG + ((p >> 1) * NB + (u32)idx) * 128u : e_dst + parbase;
      }
      if (wi >= 0 && !((rdy >> wi) & 1ull)) {
        const u64 tw0 = PNOW();
        for (;;) {
          poll();
          if ((rdy >> wi) & 1ull) break;
          if (give_up(myws, seq, t0, round)) { dead = true; break; }
          __builtin_amdgcn_s_sleep(2);
        }
        stall += PNOW() - tw0;
        if (dead) { *fail = 1u; if (!pulling) *npush = (u32)s + 1u; post_end(s); continue; }
      }
      asm volatile("" ::: "memory");
      {
        const u64 sa = (u64)(kind == K_X ? px : (kind == K_W ? P.w : (kind == K_PULL ? (const char*)pws : (const char*)myws))) + e_src;
        const u64 da = (u64)(kind == K_OUT ? P.outd[d] : (kind == K_PULL ? P.outd[0] : pws)) + e_dst;
        post(s, (u32)sa, (u32)da, (u32)(sa >> 32) | (u32)(da >> 32) << 16, e_flag << 8 | kind);
      }
      ++s;
    }
    if (!dead) { if (!pulling) *npush = (u32)s + 1u; post_end(s); }
    PVAL(4, stall);
    PVAL(5, (u64)s);
    PVAL(6, rtt);
  } else if (wave < NWK) {
    const u32 v_lin = (u32)lane * 16u;
    const u32 v_str = (u32)(lane >> 3) * NG + (u32)(lane & 7) * 16u;
    const u32 v_out = (u32)(lane >> 5) * (NG * 4u) + (u32)(lane & 31) * 16u;
    const u32 v_gat = (u32)((lane & 31) >> 2) * 1024u + (u32)(lane >> 5) * 64u + (u32)(lane & 3) * 16u;
    const u32 cnt0 = (u32)(uintptr_t)cnt;
    const u32 paroff = ((late_rfl(ctr_v) / (u32)GRID) & 1u) * PAR_BYTES;
    struct D { u32 slo, shi, dlo, dhi, s0, s1, d0, d1, kind, vs, vd; };
    auto fetch = [=](u32 j) -> D {
      const u4 e = desc[j >> 3];
      const u32 sub = j & 7u;
      D c;
      c.slo = rfl(e.x); c.dlo = rfl(e.y);
      const u32 h = rfl(e.z);
      c.shi = h & 0xffffu; c.dhi = h >> 16;
      c.kind = rfl(e.w) & 255u;
      const bool isw = c.kind == K_W, iso = c.kind >= K_OUT;
      c.s0 = isw ? sub * (16u * NG) : (iso ? (sub >> 2) * 8192u + (sub & 3u) * 256u : sub * 2048u);
      c.s1 = c.s0 + (isw ? 8u * NG : (iso ? 128u : 1024u));
      c.d0 = iso ? sub * (4u * NG * 4u) : paroff + sub * 2048u;
      c.d1 = c.d0 + (iso ? 2u * NG * 4u : 1024u);
      c.vs = c.vd = v_lin;
      if (isw) c.vs = v_str;
      if (iso) { c.vs = v_gat; c.vd = v_out; }
      asm volatile("" :: "s"(c.slo), "s"(c.shi), "s"(c.dlo), "s"(c.dhi), "s"(c.s0), "s"(c.s1), "s"(c.d0), "s"(c.d1), "v"(c.vs), "v"(c.vd));
      return c;
    };
    auto take = [=]() -> u32 {
      u64 keep;
      u32 v;
      asm volatile("s_mov_b64 %0, exec\n\ts_mov_b64 exec, 1\n\tds_add_rtn_u32 %1, %2, %3\n\ts_mov_b64 exec, %0\n\ts_waitcnt lgkmcnt(0)"
                   : "=&s"(keep), "=&v"(v) : "v"(cnt0 + 4u * (NSLOT - 3)), "v"(1u) : "memory");
      return rfl(v);
    };
    auto arrive = [&](u32 j) {
      u64 keep;
      asm volatile("s_mov_b64 %0, exec\n\ts_mov_b64 exec, 1\n\tds_add_u32 %1, %2\n\ts_mov_b64 exec, %0"
                   : "=&s"(keep) : "v"(cnt0 + 4u * (j >> 3)), "v"(1u) : "memory");
    };
    auto load_step = [&](const D& e, u32 so) -> u4 {
      return __builtin_amdgcn_raw_buffer_load_b128(mk_rsrc((const void*)((u64)e.shi << 32 | e.slo), 0x200000u), e.vs, so, AUX_SYS);
    };
    auto store_step = [&](const D& e, u32 so, const u4& a) {
      __builtin_amdgcn_raw_buffer_store_b128(a, mk_rsrc((const void*)((u64)e.dhi << 32 | e.dlo), PAR_BYTES + 0x200000u), e.vd, so, AUX_SYS);
      asm volatile("s_nop 1" ::: "memory");
    };
    constexpr u32 NONE = 0xffffffffu;
    u32 ja = (u32)wave, jb = (u32)wave + NWK, owed = NONE;
    bool first = true;
    u64 nruns = 0;
    for (;;) {
      ++nruns;
      D C = fetch(ja);
      while (C.kind == 0u) { __builtin_amdgcn_s_sleep(1); C = fetch(ja); }
      if (C.kind == K_END) break;
      if (!first) a0 = load_step(C, C.s0);
      SCHED_FENCE();
      D N;
      if (C.kind != K_PULL) {
        if (!first) a1 = load_step(C, C.s1);
        first = false;
        SCHED_FENCE();
        N = fetch(jb);
        for (;;) {
          VMCNT1_PUSH();
          store_step(C, C.d0, a0);
          if (__builtin_expect(N.kind == 0u, 0)) N = fetch(jb);
          const bool nxt = N.kind != 0u && N.kind != K_END;
          if (nxt) a0 = load_step(N, N.s0);
          if (owed != NONE) arrive(owed);
          owed = ja;
          ja = jb;
          jb = take();
          const D N2 = fetch(jb);
          if (nxt) VMCNT1_PUSH(); else VMCNT(0);
          store_step(C, C.d1, a1);
          C = N;
          N = N2;
          if (C.kind - 1u >= K_OUT) break;
          a1 = load_step(C, C.s1);
          if (N.kind == 0u) N = fetch(jb);
        }
        if (C.kind != K_PULL) {
          VMCNT(0);
          arrive(owed);
          owed = NONE;
          continue;
        }
      } else {
        N = fetch(jb);
      }
      for (;;) {
        VMCNT1_PULL();
        a1 = load_step(C, C.s1);
        store_step(C, C.d0, a0);
        const u32 jc = take();
        const D N2 = fetch(jc);
        VMCNT1_PULL();
        if (__builtin_expect(N.kind == 0u, 0)) N = fetch(jb);
        const bool nxt = N.kind == K_PULL;
        if (nxt) a0 = load_step(N, N.s0);
        store_step(C, C.d1, a1);
        if (owed != NONE) { arrive(owed); owed = NONE; }
        ja = jb;
        jb = jc;
        if (!nxt) break;
        C = N;
        N = N2;
      }
    }
    PVAL(40 + wave, nruns);
  }
}

__device__ __forceinline__ void glds16(__amdgpu_buffer_rsrc_t rs, u32 lds, u32 voff, u32 soff) {
  asm volatile("s_mov_b32 m0, %0\n\ts_nop 3\n\tbuffer_load_dwordx4 %1, %2, %3 offen lds"
               :: "s"(rfl(lds)), "v"(voff), "s"(rs), "s"(rfl(soff)) : "m0", "memory");
}

__device__ __forceinline__ void gemm_role(const Params& P, char* smem, u32 seq, u32 parbase, u64 t0) {
  const int tid = threadIdx.x, lane = tid & 63;
  const int wave = rfl(tid >> 6);
  const int r = P.rank;
  char* myws = P.wsd[0];
  const int xg = (int)blockIdx.x & 7, kx = ((int)blockIdx.x - NCOMM) >> 3;
  constexpr int gpx = GPX;
  const u32 lds0 = (u32)(uintptr_t)(LDS_AS char*)smem;

  const __amdgpu_buffer_rsrc_t rs_x = mk_rsrc(P.x, X_BYTES);
  const __amdgpu_buffer_rsrc_t rs_w = mk_rsrc(P.w, W_BYTES);
  const __amdgpu_buffer_rsrc_t rs_ws = mk_rsrc(myws, WS_NEED);
  const float scale = P.sx[0] * P.sw[0];

  u32 vX[2], vWs[2], vWo[2];
#pragma unroll
  for (int u = 0; u < 2; ++u) {
    const int row = wave * 16 + u * 8 + (lane >> 3), slot = lane & 7;
    const int cx = slot ^ ((row >> 1) & 7);
    const int cw = slot ^ (((row >> 1) & 3) | (((row >> 4) & 1) << 2));
    vX[u] = (u32)row * KS + (u32)cx * 16u;
    vWs[u] = (u32)row * 128u + (u32)cw * 16u;
    vWo[u] = (u32)row * NG + (u32)cw * 16u;
  }
  const int wm = wave >> 2, wn = wave & 3, g = lane >> 4, l15 = lane & 15;
  const int aX0 = (wm * 64 + l15) * 128 + ((g ^ (l15 >> 1)) * 16);
  const int aX1 = aX0 ^ 64;
  const int fw = ((l15 >> 2) & 3) | ((g & 1) << 2);
  const int aW0 = 16384 + (16 * g + (l15 >> 1)) * 128 + (((2 * wn) ^ fw) * 16) + (l15 & 1) * 8;
  const int aW1 = aW0 ^ 16;
  volatile LDS_AS u32* ctrl = (volatile LDS_AS u32*)(smem + LDS_CTRL);
  u32 round = 0;

  for (int e = kx; e < 2 * NB; e += gpx) {
    const u32 ent = (P.tiles[xg & 1][e >> 2] >> (8 * (e & 3))) & 255u;
    const int b = (int)(ent & 31u);
    const int a = (ent >> 5) == 0u ? xg : ((((b ^ xg) & 1) == 0 ? 8 : 12) + (xg >> 1));
    const int NDIR = P.ndir;
    u32 pref;
    if (b < NDIR) pref = 0x17654320u;
    else pref = ((b - NDIR) & 1) ? 0x64217530u : 0x75316420u;
    u32 ordd = 0;
    const u32 sX_own = (u32)grow_of(r, a) * PANEL;
    const u32 sW_own = (u32)((r >> 1) * NB + b) * 128u;
    const u32 sX_ws = parbase + REL_XG + (u32)a * PANEL;
    const u32 sW_ws = parbase + REL_WG + (u32)b * PANEL;
    const u32 foff = lane < 32 ? OFF_FX + (u32)a * 128u + (u32)lane * 4u : OFF_FW + (u32)b * 128u + (u32)(lane - 32) * 4u;

    v4f acc[2][4];
#pragma unroll
    for (int nb = 0; nb < 2; ++nb)
#pragma unroll
      for (int mb = 0; mb < 4; ++mb) acc[nb][mb] = v4f{0.f, 0.f, 0.f, 0.f};

    auto issue = [&](int kt, int stage) {
      const int dq = (ordd >> (4 * (kt >> 2))) & 7;
      const u32 q = (u32)(r ^ dq);
      const u32 kb = (u32)(kt & 3);
      const u32 ldsb = lds0 + (u32)stage * STAGE + (u32)wave * 2048u;
      if (dq == 0) {
        const u32 sx = sX_own + kb * 128u, sw = sW_own + kb * 128u * NG;
        glds16(rs_x, ldsb, vX[0], sx);
        glds16(rs_x, ldsb + 1024u, vX[1], sx);
        glds16(rs_w, ldsb + 16384u, vWo[0], sw);
        glds16(rs_w, ldsb + 16384u + 1024u, vWo[1], sw);
      } else {
        const u32 sx = sX_ws + q * SRC_BYTES + kb * 128u, sw = sW_ws + q * SRC_BYTES + kb * 16384u;
        glds16(rs_ws, ldsb, vX[0], sx);
        glds16(rs_ws, ldsb + 1024u, vX[1], sx);
        glds16(rs_ws, ldsb + 16384u, vWs[0], sw);
        glds16(rs_ws, ldsb + 16384u + 1024u, vWs[1], sw);
      }
    };
    auto compute = [&](int stage) {
      const LDS_AS char* sb = (const LDS_AS char*)smem + stage * STAGE;
      v8i xf[4], wf[2];
#pragma unroll
      for (int nb = 0; nb < 2; ++nb) {
        const LDS_AS char* pw = sb + (nb ? aW1 : aW0);
        const v2i t0_ = __builtin_amdgcn_ds_read_tr8_b64_v2i32((LDS_AS v2i*)(pw));
        const v2i t1_ = __builtin_amdgcn_ds_read_tr8_b64_v2i32((LDS_AS v2i*)(pw + 1024));
        const v2i t2_ = __builtin_amdgcn_ds_read_tr8_b64_v2i32((LDS_AS v2i*)(pw + 8192));
        const v2i t3_ = __builtin_amdgcn_ds_read_tr8_b64_v2i32((LDS_AS v2i*)(pw + 9216));
        wf[nb] = v8i{t0_.x, t0_.y, t1_.x, t1_.y, t2_.x, t2_.y, t3_.x, t3_.y};
      }
#pragma unroll
      for (int mb = 0; mb < 4; ++mb) {
        const v4i lo = *(const LDS_AS v4i*)(sb + aX0 + mb * 2048);
        const v4i hi = *(const LDS_AS v4i*)(sb + aX1 + mb * 2048);
        xf[mb] = v8i{lo.x, lo.y, lo.z, lo.w, hi.x, hi.y, hi.z, hi.w};
      }
#pragma unroll
      for (int mb = 0; mb < 4; ++mb)
#pragma unroll
        for (int nb = 0; nb < 2; ++nb)
          acc[nb][mb] = __builtin_amdgcn_mfma_scale_f32_16x16x128_f8f6f4(wf[nb], xf[mb], acc[nb][mb], 1, 0, 0, 0, 0, 0);
    };

    u32 taken = 0;
    int nass = 0;
    auto assign = [&](u32 f) {
      const u64 m = __ballot((((lane & 31) >> 2) == r) || f >= seq);
      const u32 m32 = (u32)m & (u32)(m >> 32);
      for (int i = 0; i < 8; ++i) {
        const u32 dq = (pref >> (4 * i)) & 7u;
        if (!((taken >> dq) & 1u) && ((m32 >> (4u * ((u32)r ^ dq))) & 15u) == 15u) {
          taken |= 1u << dq;
          ordd |= (8u | dq) << (4 * nass);
          ++nass;
        }
      }
    };
    auto wait_slices = [&](int n) {
      for (;;) {
        assign(__builtin_amdgcn_raw_buffer_load_b32(rs_ws, foff, 0, AUX_SYS));
        if (nass >= n) break;
        if (give_up(myws, seq, t0, round)) {
          while (nass < 8) { ordd |= 8u << (4 * nass); ++nass; }
          ctrl[1] = 1u;
          break;
        }
        __builtin_amdgcn_s_sleep(4);
      }
    };
    auto avail = [&](int sl) { return ((ordd >> (4 * sl)) & 8u) != 0u; };

    if (wave == 0) { STAMP(8 * (e / gpx) + 0); ctrl[0] = 0u; ctrl[1] = 0u; }
    for (int sl = 0; sl < 8;) {
      if (wave == 0) {
        wait_slices(sl == 0 ? 2 : sl + 1);
        if (sl == 0) {
          __builtin_amdgcn_fence(__ATOMIC_ACQUIRE, "");
          VMCNT(0);
          STAMP(8 * (e / gpx) + 1);
        }
        ctrl[2] = ordd;
      }
      RAW_BARRIER();
      ordd = rfl(ctrl[2]);
      issue(4 * sl, 0); issue(4 * sl + 1, 1); issue(4 * sl + 2, 2);
      for (bool last = false; !last; ++sl) {
        const int kt = sl * 4;
        VMCNT(8); RAW_BARRIER();
        u32 pf;
        {
          const u32 po = (wave == 0 && nass < 8) ? foff : 0xfffffff0u;
          asm volatile("buffer_load_dword %0, %1, %2, 0 offen sc0 sc1" : "=v"(pf) : "v"(po), "s"(rs_ws) : "memory");
        }
        issue(kt + 3, 3);
        compute(0);
        last = sl == 7 || !avail(sl + 1);
        VMCNT(8); RAW_BARRIER();
        if (!last) issue(kt + 4, 0);
        compute(1);
        if (!last) VMCNT(8); else VMCNT(4);
        RAW_BARRIER();
        if (!last) issue(kt + 5, 1);
        compute(2);
        if (!last) VMCNT(8); else VMCNT(0);
        asm volatile("" : "+v"(pf));
        if (wave == 0 && nass < 8) { assign(pf); ctrl[0] = ordd; }
        RAW_BARRIER();
        if (!avail(7)) ordd |= rfl(ctrl[0]);
        if (last && sl < 7 && avail(sl + 1)) {
          issue(kt + 4, 0); issue(kt + 5, 1);
          last = false;
        }
        if (!last) issue(kt + 6, 2);
        compute(3);
      }
    }
    if (wave == 0) STAMP(8 * (e / gpx) + 3);

    {
      const bool bad = rfl(ctrl[1]) != 0u;
      const bool own = a >= 12;
      const __amdgpu_buffer_rsrc_t rs_o = mk_rsrc(own ? P.outd[0] : myws, own ? OUT_BYTES : WS_NEED);
      const u32 rs = own ? NG * 4u : 64u, rbs = own ? 16u * NG * 4u : 8192u, cbs = own ? 64u : 1024u;
      const u32 so = (own ? (u32)(a - 12) * 128u * (NG * 4u) + (u32)((r >> 1) * NB + b) * 512u : OFF_ST + (u32)(a * NB + b) * 65536u) +
                     (u32)wm * 4u * rbs + (u32)wn * 2u * cbs;
      const u32 vo = (u32)l15 * rs + (u32)g * 16u;
      const float sc = bad ? __builtin_nanf("") : scale;
#pragma unroll
      for (int mb = 0; mb < 4; ++mb)
#pragma unroll
        for (int nb = 0; nb < 2; ++nb) {
          const v4f y = acc[nb][mb] * sc;
          v4f o;
#pragma unroll
          for (int j = 0; j < 4; ++j)
            o[j] = __builtin_amdgcn_exp2f(__builtin_amdgcn_fmed3f(y[j], -60.f, 60.f) * -1.44269504088896f);
          o = o + 1.f;
#pragma unroll
          for (int j = 0; j < 4; ++j) o[j] = __builtin_amdgcn_rcpf(o[j]);
          o = o * y;
          asm volatile("s_nop 4\n\tbuffer_store_dwordx4 %0, %1, %2, %3 offen sc0 sc1\n\ts_nop 1"
                       :: "v"(o), "v"(vo), "s"(rs_o), "s"(so + (u32)mb * rbs + (u32)nb * cbs) : "memory");
        }
      VMCNT(0);
      RAW_BARRIER();
      if (!own && !bad && tid < 4) {
        const u32 dd = 2u + (u32)(a >> 1);
        char* fb = b * 2 + (a & 1) >= P.pull_from[1 + (dd & 1u)] ? P.wsd[dd] + OFF_FI : myws + OFF_FO;
        st_sys(fb + dd * 1024u + (u32)(b * 2 + (a & 1)) * 16u + (u32)tid * 4u, seq);
      }
    }
    if (wave == 0) STAMP(8 * (e / gpx) + 4);
  }
}

__global__ void __launch_bounds__(THREADS, 3) fused_gemm_rs(Params P) {
  extern __shared__ __attribute__((aligned(16))) char smem[];
  const u64 t0 = __builtin_amdgcn_s_memrealtime();
  if (__builtin_expect(blockIdx.x < (u32)NCOMM, 1)) {
    comm_role(P, smem, t0);
  } else {
    if (rfl(threadIdx.x >> 6) >= 8u) return;
    char* myws = P.wsd[0];
    const u32 call = rfl(ld_dev(myws + OFF_CTR)) / (u32)GRID;
    gemm_role(P, smem, call + 1u, OFF_DATA + (call & 1u) * PAR_BYTES, t0);
    if (threadIdx.x == 0)
      __hip_atomic_fetch_add((GLB_AS u32*)(myws + OFF_CTR), 1u, __ATOMIC_RELAXED, __HIP_MEMORY_SCOPE_AGENT);
  }
}

struct Tune { int nrel, margin_p, ownx_after, own_delay, relay_start, margin_o, pull_from, pre_adv, cq, rep; };

void build_schedule(Params& P, const Tune& t) {
  const int ndir = NB - t.nrel;
  P.ndir = ndir;
  P.margin[0] = t.margin_p;
  P.margin[1] = t.margin_o;
  P.pre_adv = t.pre_adv - 8;
  for (int cls = 0; cls < 3; ++cls) {
    unsigned char L[MAXU] = {};
    int n = 0, nw = 0;
    bool ownx = false;
    auto put = [&](u32 kind, int idx) { if (n < MAXU) L[n++] = (unsigned char)(((kind - 1u) << 6) | (u32)idx); };
    auto putw = [&](int b) {
      if (!ownx && nw >= t.ownx_after) { for (int a = 12; a < 16; ++a) put(K_X, a); ownx = true; }
      put(K_W, b); ++nw;
    };
    for (int a = 0; a < 12; ++a) put(K_X, a);
    if (cls == 0) {
      for (int b = 0; b < ndir; ++b) putw(b);
      for (int key = -3 * t.cq; key <= 4 * (t.nrel - 1); ++key)
        for (int j = 3; j >= 0; --j) {
          const int k4 = key + t.cq * j;
          if (k4 < 0 || (k4 & 3) != 0 || k4 / 4 >= t.nrel) continue;
          if (j == 3) putw(ndir + k4 / 4); else put(K_FWD, 3 * (k4 / 4) + 2 - j);
        }
    } else {
      const int par = cls - 1;
      const int nr = (t.nrel + 1 - par) / 2;
      const int lead = t.relay_start < ndir ? t.relay_start : ndir;
      for (int b = 0; b < lead; ++b) putw(b);
      const int rest = ndir - lead, tot = rest + nr;
      for (int j = 0; j < tot; ++j) {
        const int r0 = j * nr / tot, r1 = (j + 1) * nr / tot;
        if (r1 > r0) putw(ndir + 2 * r0 + par); else putw(lead + j - r0);
      }
    }
    if (!ownx) for (int a = 12; a < 16; ++a) put(K_X, a);
    P.pull_from[cls] = 2 * t.pull_from;
    if (cls != 0) for (int o = 0; o < 2 * NB; ++o) put(K_OUT, o);
    P.nunits[cls] = n;
    for (int i = 0; i < MAXU / 4; ++i)
      P.ord[cls][i] = (u32)L[4 * i] | (u32)L[4 * i + 1] << 8 | (u32)L[4 * i + 2] << 16 | (u32)L[4 * i + 3] << 24;
  }
  for (int xp = 0; xp < 2; ++xp) {
    unsigned char T[2 * NB];
    int n = 0;
    for (int key = 0; key < NB + t.own_delay; ++key) {
      if (key < NB) {
        T[n++] = (unsigned char)key;
        if (((key ^ xp) & 1) == 0) T[n++] = (unsigned char)(32 | key);
      }
      const int ob = key - t.own_delay;
      if (ob >= 0 && ob < NB && ((ob ^ xp) & 1) != 0) T[n++] = (unsigned char)(32 | ob);
    }
    for (int i = 0; i < 16; ++i)
      P.tiles[xp][i] = (u32)T[4 * i] | (u32)T[4 * i + 1] << 8 | (u32)T[4 * i + 2] << 16 | (u32)T[4 * i + 3] << 24;
  }
}

}

extern "C" void kernel_launch(int rank, int nranks, ncclComm_t comm,
                              hipStream_t stream,
                              void* const* d_in, const int* in_sizes, int n_in,
                              void* d_out, int out_size,
                              const int* in_dtypes, int out_dtype,
                              void* const* d_out_all,
                              void* const* d_ws_all, size_t ws_bytes) {
  (void)comm; (void)in_dtypes; (void)out_dtype;
  if (nranks != NR || n_in != 4 || rank < 0 || rank >= NR || ws_bytes < (size_t)WS_NEED ||
      in_sizes[0] != (int)X_BYTES || in_sizes[1] != (int)W_BYTES || out_size != (int)(OUT_BYTES / 4)) {
    fprintf(stderr, "kernel_launch: unexpected problem shape\n");
    return;
  }
  static bool attr_set[NR] = {};
  if (!attr_set[rank]) {
    HIP_CHECK(hipFuncSetAttribute((const void*)fused_gemm_rs, hipFuncAttributeMaxDynamicSharedMemorySize, LDS_BYTES));
    attr_set[rank] = true;
  }
  Params P;
  P.x = (const char*)d_in[0];
  P.w = (const char*)d_in[1];
  P.sx = (const float*)d_in[2];
  P.sw = (const float*)d_in[3];
  for (int i = 0; i < NR; ++i) { P.wsd[i] = (char*)d_ws_all[rank ^ i]; P.outd[i] = (char*)d_out_all[rank ^ i]; }
  P.outd[0] = (char*)d_out;
  P.rank = rank;
  Tune t = {16, 2, 40, 32, 4, 2, 0, 6, 3, 0};
  build_schedule(P, t);
  hipLaunchKernelGGL(fused_gemm_rs, dim3(GRID), dim3(THREADS), LDS_BYTES, stream, P);
  HIP_CHECK(hipGetLastError());
}
```

```cpp
#include <hip/hip_runtime.h>
#include <rccl/rccl.h>
#include <cstdio>
#include <cstdint>

#define HIP_CHECK(expr) do { hipError_t _e = (expr); if (_e != hipSuccess) { \
    fprintf(stderr, "HIP error %s at %s:%d\n", hipGetErrorString(_e), __FILE__, __LINE__); \
    return; } } while (0)

namespace {

typedef unsigned u32;
typedef unsigned long long u64;
typedef unsigned u4 __attribute__((ext_vector_type(4)));
typedef int v2i __attribute__((ext_vector_type(2)));
typedef int v4i __attribute__((ext_vector_type(4)));
typedef int v8i __attribute__((ext_vector_type(8)));
typedef float v4f __attribute__((ext_vector_type(4)));
#define LDS_AS __attribute__((address_space(3)))
#define GLB_AS __attribute__((address_space(1)))

constexpr int NR = 8;
constexpr int MG = 4096, KG = 4096, NG = 16384;
constexpr int KS = KG / NR;
constexpr u32 PANEL = 128 * KS;
constexpr int NA = 16;
constexpr int NB = 32;
#define WPL_CFG 8
constexpr int WPL = WPL_CFG;
constexpr int NCOMM = 7 * WPL;
constexpr int GRID = 256;
constexpr int NGEMM = GRID - NCOMM;
constexpr int GPX = NGEMM / 8;
constexpr int THREADS = 512;

#define NWK_CFG 6
constexpr int NWK = NWK_CFG;
#define PDEPTH 3
constexpr int MAXU = 128;
constexpr int NSLOT = 512;

constexpr u32 OFF_CTR = 0, OFF_TMO = 128;
constexpr u32 OFF_XC = 256;
constexpr u32 OFF_TK = 1024;
constexpr u32 OFF_FX = 4096;
constexpr u32 OFF_FW = 8192;
constexpr u32 OFF_FR = 12288;
constexpr u32 OFF_GO = 16384;
constexpr u32 OFF_FI = 32768;
constexpr u32 OFF_DBG = 512 << 10;
constexpr u32 OFF_ST = 1 << 20;
constexpr u32 OFF_DATA = OFF_ST + 12 * NB * 65536;
constexpr u32 SRC_BYTES = (NA + NB) * PANEL;
constexpr u32 REL_XG = 0;
constexpr u32 REL_WG = NA * PANEL;
constexpr int NEXP = MG / 128 + NG / 128;
constexpr int NEXW = 128;
constexpr u32 REL_EXP = 8 * SRC_BYTES;
constexpr u32 EXPS = PANEL + 2048;
constexpr u32 PAR_BYTES = REL_EXP + NEXP * EXPS;
constexpr u32 WS_NEED = OFF_DATA + 2 * PAR_BYTES;

constexpr u32 X_BYTES = MG * KS;
constexpr u32 W_BYTES = KS * NG;
constexpr u32 OUT_BYTES = (MG / NR) * NG * 4;

constexpr int STAGE = 32768, NST = 4;
constexpr int LDS_CTRL = NST * STAGE;
constexpr int LDS_BYTES = LDS_CTRL + 256;

constexpr u64 DEADLINE = 50000000ull;

constexpr int AUX_SYS = 17;
constexpr int AUX_DEV = 16;

struct Params {
  const char* x;
  const char* w;
  char* wsd[NR];
  int rank;
  int ndir;
  int go_step;
  int nunits[3];
  int margin[2];
  int exp_delay, exp_spread;
  const float* sx;
  const float* sw;
  char* out;
  u32 ord[3][MAXU / 4];
  u32 tiles[2][16];
};

__device__ __forceinline__ u32 rfl(u32 v) { return __builtin_amdgcn_readfirstlane(v); }
__device__ __forceinline__ __amdgpu_buffer_rsrc_t mk_rsrc(const void* p, u32 bytes) {
  return __builtin_amdgcn_make_buffer_rsrc((void*)p, (short)0, (int)bytes, 0x00020000);
}
__device__ __forceinline__ u32 ld_sys(const char* p) {
  return __hip_atomic_load((const GLB_AS u32*)p, __ATOMIC_RELAXED, __HIP_MEMORY_SCOPE_SYSTEM);
}
__device__ __forceinline__ void st_sys(char* p, u32 v) {
  __hip_atomic_store((GLB_AS u32*)p, v, __ATOMIC_RELAXED, __HIP_MEMORY_SCOPE_SYSTEM);
}
__device__ __forceinline__ u32 ld_dev(const char* p) {
  return __hip_atomic_load((const GLB_AS u32*)p, __ATOMIC_RELAXED, __HIP_MEMORY_SCOPE_AGENT);
}
__device__ __forceinline__ void st_dev(char* p, u32 v) {
  __hip_atomic_store((GLB_AS u32*)p, v, __ATOMIC_RELAXED, __HIP_MEMORY_SCOPE_AGENT);
}
#define VMCNT(n) asm volatile("s_waitcnt vmcnt(" #n ")" ::: "memory")
#define SCHED_FENCE() do { asm volatile("" ::: "memory"); __builtin_amdgcn_sched_barrier(0); } while (0)
#define RAW_BARRIER() do { asm volatile("s_waitcnt lgkmcnt(0)" ::: "memory"); __builtin_amdgcn_s_barrier(); asm volatile("" ::: "memory"); } while (0)

__device__ __forceinline__ bool give_up(char* myws, u32 seq, u64 t0, u32& round) {
  if ((++round & 7u) != 0u) return false;
  const u32 t = rfl(ld_dev(myws + OFF_TMO));
  const bool late = (__builtin_amdgcn_s_memrealtime() - t0) > DEADLINE;
  if (t == seq) return true;
  if (late) {
    st_dev(myws + OFF_TMO, seq);
    return true;
  }
  return false;
}

#define STAMP(slot) do {} while (0)
#define PVAL(slot, v) do {} while (0)
#define PNOW() 0ull

__device__ __forceinline__ int grow_of(int p, int a) {
  if (a < 12) return ((p ^ (2 + (a >> 1))) << 2) + ((p & 1) << 1) + (a & 1);
  return (p << 2) + (a - 12);
}

constexpr u32 K_X = 0, K_W = 1, K_FWD = 2, K_OUT = 3;
constexpr u32 D_END = 1, D_IN = 2, D_INR = 3, D_OUT = 4;

__device__ __forceinline__ void comm_role(const Params& P, char* smem, u64 t0) {
  const int tid = threadIdx.x, lane = tid & 63;
  const int wave = rfl(tid >> 6);
  const int w = (int)blockIdx.x / 7;
  const int d = (int)blockIdx.x - 7 * w + 1;
  const int r = P.rank;
  char* myws = P.wsd[0];
  char* pws = P.wsd[d];
  const u32 call = rfl(ld_dev(myws + OFF_CTR)) / (u32)GRID;
  const u32 seq = call + 1u;
  const u32 p = (u32)(r ^ d) & 7u;
  u32 round = 0;

  volatile LDS_AS u4* desc = (volatile LDS_AS u4*)smem;
  volatile LDS_AS u32* cnt = (volatile LDS_AS u32*)(smem + NSLOT * 16);
  volatile LDS_AS u32* gate = cnt + NSLOT - 2;
  volatile LDS_AS u32* cctr = cnt + NSLOT - 3;
  static_assert(NSLOT == THREADS, "descriptor init");
  desc[tid] = u4{0u, 0u, 0u, 0u};
  cnt[tid] = tid == NSLOT - 3 ? 2u * NWK : 0u;
  RAW_BARRIER();

  if (__builtin_expect(wave == 7, 0)) {
    char* partner = P.wsd[1];
    bool ok = true;
    while (rfl(ld_sys(myws + OFF_GO + (u32)(d * WPL + w) * 256u)) < seq) {
      if (give_up(myws, seq, t0, round)) { ok = false; break; }
      __builtin_amdgcn_s_sleep(4);
    }
    *gate = ok ? 1u : 2u;
    STAMP(0);
    for (int t = 0;; ++t) {
      u32 k;
      for (;;) {
        k = rfl(((volatile LDS_AS u32*)(desc + t))[3]);
        if (k != 0u && ((k >> 8) == 0u || !ok || rfl(cnt[t]) == 8u)) break;
        __builtin_amdgcn_s_sleep(1);
      }
      if (k == D_END) break;
      if (!ok) continue;
      if ((k >> 8) != 0u) st_sys(myws + (k >> 8), seq);
      if ((k & 255u) == D_INR) st_sys(partner + (k >> 8) + (OFF_FR - OFF_FW), seq);
    }
    STAMP(3);
    if (lane == 0) __hip_atomic_fetch_add((GLB_AS u32*)(myws + OFF_CTR), 1u, __ATOMIC_RELAXED, __HIP_MEMORY_SCOPE_AGENT);
  } else if (__builtin_expect(wave == 6, 0)) {
    const int cls = d == 1 ? 0 : 1 + (d & 1);
    const int NP = 4 * P.nunits[cls];
    const int NDIR = P.ndir;
    const __amdgpu_buffer_rsrc_t rs_my = mk_rsrc(myws, WS_NEED);
    u64 stall = 0;
    const u32 parbase = OFF_DATA + (call & 1u) * PAR_BYTES;
    u32 woff = OFF_FI + (u32)d * 1024u + (u32)lane * 16u;
    if (d == 1) {
      const int fi = lane < 3 * (NB - NDIR) ? lane : 0;
      const int k = fi / 3, i = fi - 3 * k;
      woff = OFF_FR + (u32)(((NDIR + k) * 8 + ((int)p ^ (2 * i + 2 + (k & 1)))) * 16);
    }
    const u32 tab = P.ord[cls][lane & 31];
    u64 rdy = 0;
    auto poll = [&]() {
      const u4 f = __builtin_amdgcn_raw_buffer_load_b128(rs_my, woff, 0, AUX_SYS);
      rdy |= __ballot(f.x >= seq && f.y >= seq && f.z >= seq && f.w >= seq);
    };
    auto post_end = [&](int s) {
      if (lane < 4) ((volatile LDS_AS u32*)(desc + s + lane))[3] = D_END;
    };
    const u32 tkbase = call * (u32)(NP + WPL);
    char* tkptr = myws + OFF_TK + (u32)d * 128u;
    const int margin = P.margin[d != 1];
    bool dead = false;
    int s = 0;
    u64 rtt = 0;
    for (;;) {
      while (!dead && (int)rfl(*cctr) + margin <= 8 * s) {
        if (rfl(*gate) == 2u) { dead = true; post_end(s); }
        __builtin_amdgcn_s_sleep(1);
      }
      const u64 tr0 = PNOW();
      const u4 f = __builtin_amdgcn_raw_buffer_load_b128(rs_my, woff, 0, AUX_SYS);
      u32 old = 0;
      if (lane == 0) old = __hip_atomic_fetch_add((GLB_AS u32*)tkptr, 1u, __ATOMIC_RELAXED, __HIP_MEMORY_SCOPE_AGENT);
      const int tk = (int)(rfl(old) - tkbase);
      rdy |= __ballot(f.x >= seq && f.y >= seq && f.z >= seq && f.w >= seq);
      rtt += PNOW() - tr0;
      if ((u32)tk >= (u32)NP) break;
      if (dead) continue;
      const int pos = tk >> 2;
      const u32 piece = (u32)tk & 3u;
      const u32 ent = ((u32)__builtin_amdgcn_readlane((int)tab, pos >> 2) >> (8 * (pos & 3))) & 255u;
      const u32 kind = ent >> 6;
      const int idx = (int)(ent & 63u);
      u64 sa, da;
      u32 k3;
      if (kind == K_X) {
        sa = (u64)pws + parbase + REL_EXP + (u32)grow_of(r, idx) * EXPS;
        da = (u64)myws + parbase + REL_XG + p * SRC_BYTES + (u32)idx * PANEL;
        k3 = (OFF_FX + ((u32)idx * 8u + p) * 16u + piece * 4u) << 8 | D_IN;
      } else if (kind == K_W) {
        sa = (u64)pws + parbase + REL_EXP + (u32)(MG / 128 + (r >> 1) * NB + idx) * EXPS;
        da = (u64)myws + parbase + REL_WG + p * SRC_BYTES + (u32)idx * PANEL;
        k3 = (OFF_FW + ((u32)idx * 8u + p) * 16u + piece * 4u) << 8 | (d != 1 && idx >= NDIR ? D_INR : D_IN);
      } else if (kind == K_FWD) {
        const int kk = idx / 3, i = idx - 3 * kk;
        const u32 q = p ^ (u32)(2 * i + 2 + (kk & 1)), b = (u32)(NDIR + kk);
        sa = (u64)pws + parbase + REL_WG + q * SRC_BYTES + b * PANEL;
        da = (u64)myws + parbase + REL_WG + q * SRC_BYTES + b * PANEL;
        k3 = (OFF_FW + (b * 8u + q) * 16u + piece * 4u) << 8 | D_IN;
      } else {
        const int b = idx >> 1, tt = idx & 1;
        sa = (u64)pws + OFF_ST + (u32)(((d - 2) * 2 + tt) * NB + b) * 65536u;
        da = (u64)P.out + ((2u * (p & 1u) + (u32)tt) * 128u + piece * 32u) * (NG * 4u) + ((p >> 1) * NB + (u32)b) * 512u - piece * 16384u;
        k3 = D_OUT;
      }
      sa += piece * 16384u;
      da += piece * 16384u;
      if (kind >= K_FWD && !((rdy >> idx) & 1ull)) {
        const u64 tw0 = PNOW();
        for (;;) {
          poll();
          if ((rdy >> idx) & 1ull) break;
          if (give_up(myws, seq, t0, round)) { dead = true; break; }
          __builtin_amdgcn_s_sleep(2);
        }
        stall += PNOW() - tw0;
      }
      if (dead) {
        if (kind == K_OUT) st_sys((char*)da, 0x7fc00000u);
        post_end(s);
        continue;
      }
      asm volatile("" ::: "memory");
      if (lane == 0) {
        volatile LDS_AS u32* dp = (volatile LDS_AS u32*)(desc + s);
        dp[0] = (u32)sa; dp[1] = (u32)da; dp[2] = (u32)(sa >> 32) | (u32)(da >> 32) << 16; dp[3] = k3;
      }
      ++s;
    }
    if (!dead) post_end(s);
    PVAL(4, stall);
    PVAL(5, (u64)s);
    PVAL(6, rtt);
  } else if (wave < NWK) {
    const u32 v_lin = (u32)lane * 16u;
    const u32 v_out = (u32)(lane >> 5) * (NG * 4u) + (u32)(lane & 31) * 16u;
    const u32 v_gat = (u32)((lane & 31) >> 2) * 1024u + (u32)(lane >> 5) * 64u + (u32)(lane & 3) * 16u;
    const u32 cnt0 = (u32)(uintptr_t)cnt;
    struct D { u64 s0, s1, d0, d1; u32 kind, vs, vd; };
    auto fetch = [=](u32 j) -> D {
      const u4 e = desc[j >> 3];
      const u32 sub = j & 7u, h = rfl(e.z);
      D c;
      c.kind = rfl(e.w) & 255u;
      const bool iso = c.kind == D_OUT;
      c.s0 = ((u64)(h & 0xffffu) << 32 | rfl(e.x)) + (iso ? (sub >> 2) * 8192u + (sub & 3u) * 256u : sub * 2048u);
      c.s1 = c.s0 + (iso ? 128u : 1024u);
      c.d0 = ((u64)(h >> 16) << 32 | rfl(e.y)) + (iso ? sub * (4u * NG * 4u) : sub * 2048u);
      c.d1 = c.d0 + (iso ? 2u * NG * 4u : 1024u);
      c.vs = c.vd = v_lin;
      if (iso) { c.vs = v_gat; c.vd = v_out; }
      return c;
    };
    auto take = [=]() -> u32 {
      u64 keep;
      u32 v;
      asm volatile("s_mov_b64 %0, exec\n\ts_mov_b64 exec, 1\n\tds_add_rtn_u32 %1, %2, %3\n\ts_mov_b64 exec, %0\n\ts_waitcnt lgkmcnt(0)"
                   : "=&s"(keep), "=&v"(v) : "v"(cnt0 + 4u * (NSLOT - 3)), "v"(1u) : "memory");
      return rfl(v);
    };
    auto arrive = [&](u32 j) {
      u64 keep;
      asm volatile("s_mov_b64 %0, exec\n\ts_mov_b64 exec, 1\n\tds_add_u32 %1, %2\n\ts_mov_b64 exec, %0"
                   : "=&s"(keep) : "v"(cnt0 + 4u * (j >> 3)), "v"(1u) : "memory");
    };
    auto u64s = [](u64 v) { return (u64)rfl((u32)(v >> 32)) << 32 | rfl((u32)v); };
    auto step = [&](u4& in, u64 sa, u32 vs, const u4& out, u64 da, u32 vd) {
      asm volatile("s_nop 4\n\ts_waitcnt vmcnt(1)\n\tglobal_load_dwordx4 %0, %1, %2 sc0 sc1\n\t"
                   "global_store_dwordx4 %3, %4, %5 sc0 sc1\n\ts_nop 1"
                   : "+v"(in) : "v"(vs), "s"(u64s(sa)), "v"(vd), "v"(out), "s"(u64s(da)) : "memory");
    };
    constexpr u32 NONE = 0xffffffffu;
    u32 ja = (u32)wave, jb = (u32)wave + NWK, owed = NONE;
    u4 a0 = u4{0u, 0u, 0u, 0u}, a1 = a0;
    u64 nruns = 0;
    u32 go;
    while ((go = rfl(*gate)) == 0u) __builtin_amdgcn_s_sleep(1);
    while (go == 1u) {
      ++nruns;
      D C = fetch(ja);
      while (C.kind == 0u) { __builtin_amdgcn_s_sleep(1); C = fetch(ja); }
      if (C.kind == D_END) break;
      asm volatile("s_nop 4\n\tglobal_load_dwordx4 %0, %1, %2 sc0 sc1" : "+v"(a0) : "v"(C.vs), "s"(u64s(C.s0)) : "memory");
      D N = fetch(jb);
      VMCNT(0);
      for (;;) {
        if (owed != NONE) VMCNT(0);
        step(a1, C.s1, C.vs, a0, C.d0, C.vd);
        if (owed != NONE) arrive(owed);
        owed = C.kind != D_OUT ? ja : NONE;
        const u32 jc = take();
        const D N2 = fetch(jc);
        if (__builtin_expect(N.kind == 0u, 0)) { VMCNT(1); N = fetch(jb); }
        const bool nxt = N.kind >= D_IN;
        step(a0, nxt ? N.s0 : C.d0, nxt ? N.vs : C.vd, a1, C.d1, C.vd);
        ja = jb;
        jb = jc;
        if (!nxt) break;
        C = N;
        N = N2;
      }
      VMCNT(0);
      if (owed != NONE) { arrive(owed); owed = NONE; }
    }
    PVAL(40 + wave, nruns);
  }
}

__device__ __forceinline__ void glds16(__amdgpu_buffer_rsrc_t rs, u32 lds, u32 voff, u32 soff) {
  asm volatile("s_mov_b32 m0, %0\n\ts_nop 3\n\tbuffer_load_dwordx4 %1, %2, %3 offen lds"
               :: "s"(rfl(lds)), "v"(voff), "s"(rs), "s"(rfl(soff)) : "m0", "memory");
}

__device__ __forceinline__ void gemm_role(const Params& P, char* smem, u32 seq, u32 parbase, u64 t0) {
  const int tid = threadIdx.x, lane = tid & 63;
  const int wave = rfl(tid >> 6);
  const int r = P.rank;
  char* myws = P.wsd[0];
  const int xg = (int)blockIdx.x & 7, kx = ((int)blockIdx.x - NCOMM) >> 3;
  constexpr int gpx = GPX;
  const u32 lds0 = (u32)(uintptr_t)(LDS_AS char*)smem;

  const __amdgpu_buffer_rsrc_t rs_x = mk_rsrc(P.x, X_BYTES);
  const __amdgpu_buffer_rsrc_t rs_w = mk_rsrc(P.w, W_BYTES);
  const __amdgpu_buffer_rsrc_t rs_ws = mk_rsrc(myws, WS_NEED);
  const float scale = P.sx[0] * P.sw[0];

  u32 vX[2], vWs[2], vWo[2];
#pragma unroll
  for (int u = 0; u < 2; ++u) {
    const int row = wave * 16 + u * 8 + (lane >> 3), slot = lane & 7;
    const int cx = slot ^ ((row >> 1) & 7);
    const int cw = slot ^ (((row >> 1) & 3) | (((row >> 4) & 1) << 2));
    vX[u] = (u32)row * KS + (u32)cx * 16u;
    vWs[u] = (u32)row * 128u + (u32)cw * 16u;
    vWo[u] = (u32)row * NG + (u32)cw * 16u;
  }
  const int wm = wave >> 2, wn = wave & 3, g = lane >> 4, l15 = lane & 15;
  const int aX0 = (wm * 64 + l15) * 128 + ((g ^ (l15 >> 1)) * 16);
  const int aX1 = aX0 ^ 64;
  const int fw = ((l15 >> 2) & 3) | ((g & 1) << 2);
  const int aW0 = 16384 + (16 * g + (l15 >> 1)) * 128 + (((2 * wn) ^ fw) * 16) + (l15 & 1) * 8;
  const int aW1 = aW0 ^ 16;
  volatile LDS_AS u32* ctrl = (volatile LDS_AS u32*)(smem + LDS_CTRL);
  u32 round = 0;

  for (int e = kx; e < 2 * NB; e += gpx) {
    const u32 ent = (P.tiles[xg & 1][e >> 2] >> (8 * (e & 3))) & 255u;
    const int b = (int)(ent & 31u);
    const int a = (ent >> 5) == 0u ? xg : ((((b ^ xg) & 1) == 0 ? 8 : 12) + (xg >> 1));
    const int NDIR = P.ndir;
    u32 pref;
    if (b < NDIR) pref = 0x17654320u;
    else pref = ((b - NDIR) & 1) ? 0x64217530u : 0x75316420u;
    u32 ordd = 0;
    const u32 sX_own = (u32)grow_of(r, a) * PANEL;
    const u32 sW_own = (u32)((r >> 1) * NB + b) * 128u;
    const u32 sX_ws = parbase + REL_XG + (u32)a * PANEL;
    const u32 sW_ws = parbase + REL_WG + (u32)b * PANEL;
    const u32 foff = lane < 32 ? OFF_FX + (u32)a * 128u + (u32)lane * 4u : OFF_FW + (u32)b * 128u + (u32)(lane - 32) * 4u;

    v4f acc[2][4];
#pragma unroll
    for (int nb = 0; nb < 2; ++nb)
#pragma unroll
      for (int mb = 0; mb < 4; ++mb) acc[nb][mb] = v4f{0.f, 0.f, 0.f, 0.f};

    auto issue = [&](int kt, int stage) {
      const int dq = (ordd >> (4 * (kt >> 2))) & 7;
      const u32 q = (u32)(r ^ dq);
      const u32 kb = (u32)(kt & 3);
      const u32 ldsb = lds0 + (u32)stage * STAGE + (u32)wave * 2048u;
      if (dq == 0) {
        const u32 sx = sX_own + kb * 128u, sw = sW_own + kb * 128u * NG;
        glds16(rs_x, ldsb, vX[0], sx);
        glds16(rs_x, ldsb + 1024u, vX[1], sx);
        glds16(rs_w, ldsb + 16384u, vWo[0], sw);
        glds16(rs_w, ldsb + 16384u + 1024u, vWo[1], sw);
      } else {
        const u32 sx = sX_ws + q * SRC_BYTES + kb * 128u, sw = sW_ws + q * SRC_BYTES + kb * 16384u;
        glds16(rs_ws, ldsb, vX[0], sx);
        glds16(rs_ws, ldsb + 1024u, vX[1], sx);
        glds16(rs_ws, ldsb + 16384u, vWs[0], sw);
        glds16(rs_ws, ldsb + 16384u + 1024u, vWs[1], sw);
      }
    };
    auto compute = [&](int stage) {
      const LDS_AS char* sb = (const LDS_AS char*)smem + stage * STAGE;
      v8i xf[4], wf[2];
#pragma unroll
      for (int nb = 0; nb < 2; ++nb) {
        const LDS_AS char* pw = sb + (nb ? aW1 : aW0);
        const v2i t0_ = __builtin_amdgcn_ds_read_tr8_b64_v2i32((LDS_AS v2i*)(pw));
        const v2i t1_ = __builtin_amdgcn_ds_read_tr8_b64_v2i32((LDS_AS v2i*)(pw + 1024));
        const v2i t2_ = __builtin_amdgcn_ds_read_tr8_b64_v2i32((LDS_AS v2i*)(pw + 8192));
        const v2i t3_ = __builtin_amdgcn_ds_read_tr8_b64_v2i32((LDS_AS v2i*)(pw + 9216));
        wf[nb] = v8i{t0_.x, t0_.y, t1_.x, t1_.y, t2_.x, t2_.y, t3_.x, t3_.y};
      }
#pragma unroll
      for (int mb = 0; mb < 4; ++mb) {
        const v4i lo = *(const LDS_AS v4i*)(sb + aX0 + mb * 2048);
        const v4i hi = *(const LDS_AS v4i*)(sb + aX1 + mb * 2048);
        xf[mb] = v8i{lo.x, lo.y, lo.z, lo.w, hi.x, hi.y, hi.z, hi.w};
      }
#pragma unroll
      for (int mb = 0; mb < 4; ++mb)
#pragma unroll
        for (int nb = 0; nb < 2; ++nb)
          acc[nb][mb] = __builtin_amdgcn_mfma_scale_f32_16x16x128_f8f6f4(wf[nb], xf[mb], acc[nb][mb], 1, 0, 0, 0, 0, 0);
    };

    u32 taken = 0;
    int nass = 0;
    auto assign = [&](u32 f) {
      const u64 m = __ballot((((lane & 31) >> 2) == r) || f >= seq);
      const u32 m32 = (u32)m & (u32)(m >> 32);
      for (int i = 0; i < 8; ++i) {
        const u32 dq = (pref >> (4 * i)) & 7u;
        if (!((taken >> dq) & 1u) && ((m32 >> (4u * ((u32)r ^ dq))) & 15u) == 15u) {
          taken |= 1u << dq;
          ordd |= (8u | dq) << (4 * nass);
          ++nass;
        }
      }
    };
    auto wait_slices = [&](int n) {
      for (;;) {
        assign(__builtin_amdgcn_raw_buffer_load_b32(rs_ws, foff, 0, AUX_SYS));
        if (nass >= n) break;
        if (give_up(myws, seq, t0, round)) {
          while (nass < 8) { ordd |= 8u << (4 * nass); ++nass; }
          ctrl[1] = 1u;
          break;
        }
        __builtin_amdgcn_s_sleep(4);
      }
    };
    auto avail = [&](int sl) { return ((ordd >> (4 * sl)) & 8u) != 0u; };

    if (wave == 0) { STAMP(8 * (e / gpx) + 0); ctrl[0] = 0u; ctrl[1] = 0u; }
    for (int sl = 0; sl < 8;) {
      if (wave == 0) {
        wait_slices(sl == 0 ? 2 : sl + 1);
        if (sl == 0) {
          __builtin_amdgcn_fence(__ATOMIC_ACQUIRE, "");
          VMCNT(0);
          STAMP(8 * (e / gpx) + 1);
        }
        ctrl[2] = ordd;
      }
      RAW_BARRIER();
      ordd = rfl(ctrl[2]);
      issue(4 * sl, 0); issue(4 * sl + 1, 1); issue(4 * sl + 2, 2);
      for (bool last = false; !last; ++sl) {
        const int kt = sl * 4;
        VMCNT(8); RAW_BARRIER();
        u32 pf;
        {
          const u32 po = (wave == 0 && nass < 8) ? foff : 0xfffffff0u;
          asm volatile("buffer_load_dword %0, %1, %2, 0 offen sc0 sc1" : "=v"(pf) : "v"(po), "s"(rs_ws) : "memory");
        }
        issue(kt + 3, 3);
        compute(0);
        last = sl == 7 || !avail(sl + 1);
        VMCNT(8); RAW_BARRIER();
        if (!last) issue(kt + 4, 0);
        compute(1);
        if (!last) VMCNT(8); else VMCNT(4);
        RAW_BARRIER();
        if (!last) issue(kt + 5, 1);
        compute(2);
        if (!last) VMCNT(8); else VMCNT(0);
        asm volatile("" : "+v"(pf));
        if (wave == 0 && nass < 8) { assign(pf); ctrl[0] = ordd; }
        RAW_BARRIER();
        if (!avail(7)) ordd |= rfl(ctrl[0]);
        if (last && sl < 7 && avail(sl + 1)) {
          issue(kt + 4, 0); issue(kt + 5, 1);
          last = false;
        }
        if (!last) issue(kt + 6, 2);
        compute(3);
      }
    }
    if (wave == 0) STAMP(8 * (e / gpx) + 3);

    {
      const bool bad = rfl(ctrl[1]) != 0u;
      const bool own = a >= 12;
      const __amdgpu_buffer_rsrc_t rs_o = mk_rsrc(own ? P.out : myws, own ? OUT_BYTES : WS_NEED);
      const u32 rs = own ? NG * 4u : 64u, rbs = own ? 16u * NG * 4u : 8192u, cbs = own ? 64u : 1024u;
      const u32 so = (own ? (u32)(a - 12) * 128u * (NG * 4u) + (u32)((r >> 1) * NB + b) * 512u : OFF_ST + (u32)(a * NB + b) * 65536u) +
                     (u32)wm * 4u * rbs + (u32)wn * 2u * cbs;
      const u32 vo = (u32)l15 * rs + (u32)g * 16u;
      const float sc = bad ? __builtin_nanf("") : scale;
#pragma unroll
      for (int mb = 0; mb < 4; ++mb)
#pragma unroll
        for (int nb = 0; nb < 2; ++nb) {
          const v4f y = acc[nb][mb] * sc;
          v4f o;
#pragma unroll
          for (int j = 0; j < 4; ++j)
            o[j] = __builtin_amdgcn_exp2f(__builtin_amdgcn_fmed3f(y[j], -60.f, 60.f) * -1.44269504088896f);
          o = o + 1.f;
#pragma unroll
          for (int j = 0; j < 4; ++j) o[j] = __builtin_amdgcn_rcpf(o[j]);
          o = o * y;
          asm volatile("s_nop 4\n\tbuffer_store_dwordx4 %0, %1, %2, %3 offen sc0 sc1\n\ts_nop 1"
                       :: "v"(o), "v"(vo), "s"(rs_o), "s"(so + (u32)mb * rbs + (u32)nb * cbs) : "memory");
        }
      VMCNT(0);
      RAW_BARRIER();
      if (!own && !bad && tid < 4) {
        const u32 dd = 2u + (u32)(a >> 1);
        st_sys(P.wsd[dd] + OFF_FI + dd * 1024u + (u32)(b * 2 + (a & 1)) * 16u + (u32)tid * 4u, seq);
      }
    }
    if (wave == 0) STAMP(8 * (e / gpx) + 4);
  }
}

__global__ void __launch_bounds__(THREADS, 2) fused_gemm_rs(Params P) {
  extern __shared__ __attribute__((aligned(16))) char smem[];
  const u64 t0 = __builtin_amdgcn_s_memrealtime();
  const u32 gi = blockIdx.x - (u32)NCOMM, lane = threadIdx.x & 63u, wave = rfl(threadIdx.x >> 6);
  u32 call = 0;
  if (__builtin_expect(gi < (u32)NCOMM, 1)) {
    const u32 w = gi / 7u, d = gi - 7u * w + 1u;
    const u32 r = (u32)P.rank;
    const char* px = P.x;
    char* myws = P.wsd[0];
    char* pws = P.wsd[d];
    const u32 go_step = (u32)P.go_step;
    asm volatile("" :: "s"(r), "s"(px), "s"(myws), "s"(pws), "s"(go_step));
    const u32 ctr_v = ld_dev(myws + OFF_CTR);
    const u32 tk = w * 6u + wave, a = tk >> 2, piece = tk & 3u;
    const __amdgpu_buffer_rsrc_t rs = mk_rsrc(px + (u32)grow_of((int)(r ^ d), (int)a) * PANEL + piece * 16384u, wave < 6u ? 16384u : 0u);
    const u32 rot = tk + 5u * d;
    u4 v[16];
#pragma unroll
    for (int i = 0; i < PDEPTH; ++i) v[i] = __builtin_amdgcn_raw_buffer_load_b128(rs, lane * 16u, ((rot + (u32)i) & 15u) * 1024u, 0);
    volatile LDS_AS u32* mark = (volatile LDS_AS u32*)(smem + LDS_CTRL + 64);
    *mark = 0u;
    RAW_BARRIER();
    call = rfl(ctr_v) / (u32)GRID;
    if (wave < 6u) {
      const __amdgpu_buffer_rsrc_t rd = mk_rsrc(pws + OFF_DATA + (call & 1u) * PAR_BYTES + REL_XG + r * SRC_BYTES + a * PANEL + piece * 16384u, 16384u);
#pragma unroll
      for (int i = 0; i < 16; ++i) {
        if (i != 0) { if (i - 1 + PDEPTH < 16) VMCNT(1); else VMCNT(0); }
        __builtin_amdgcn_raw_buffer_store_b128(v[i], rd, lane * 16u, ((rot + (u32)i) & 15u) * 1024u, AUX_SYS);
        asm volatile("s_nop 1" ::: "memory");
        if (i + PDEPTH < 16) v[i + PDEPTH] = __builtin_amdgcn_raw_buffer_load_b128(rs, lane * 16u, ((rot + (u32)(i + PDEPTH)) & 15u) * 1024u, 0);
        if (wave == 0u && go_step == (u32)i) *mark = 1u;
        SCHED_FENCE();
      }
      VMCNT(0);
      if (wave == 0u) *mark = 1u;
      st_sys(pws + OFF_FX + (a * 8u + r) * 16u + piece * 4u, call + 1u);
      STAMP(54 + wave);
    } else if (wave == 6u) {
      u32 round = 0;
      bool ok = true;
      while (rfl(*mark) == 0u) __builtin_amdgcn_s_sleep(1);
      STAMP(61);
      while (rfl(ld_sys(myws + OFF_XC)) < (call + 1u) * (u32)NEXW) {
        if (give_up(myws, call + 1u, t0, round)) { ok = false; break; }
        __builtin_amdgcn_s_sleep(16);
      }
      if (ok) st_sys(pws + OFF_GO + (d * WPL + w) * 256u, call + 1u);
      STAMP(60);
    }
  } else if (blockIdx.x < (u32)NCOMM) {
    comm_role(P, smem, t0);
    return;
  } else {
    char* myws = P.wsd[0];
    const u32 ctr_v = ld_dev(myws + OFF_CTR);
    const u32 e = gi - (u32)NCOMM;
    const u32 wrow = (e & 31u) * 16u, wpan = (e >> 5) * 32u + wave * 4u;
    if (e < (u32)NEXW) {
      const u64 tgo = (u64)((u32)P.exp_delay + (e * (u32)P.exp_spread >> 8));
      while (__builtin_amdgcn_s_memrealtime() - t0 < tgo) __builtin_amdgcn_s_sleep(2);
      u4 v[10];
#pragma unroll
      for (int i = 0; i < 8; ++i)
        v[i] = __builtin_amdgcn_raw_buffer_load_b128(mk_rsrc(P.w, W_BYTES), (lane >> 3) * NG + (lane & 7u) * 16u,
                                                     (wrow + (u32)(i & 1) * 8u) * NG + (wpan + (u32)(i >> 1)) * 128u, 0);
#pragma unroll
      for (int i = 0; i < 2; ++i)
        v[8 + i] = __builtin_amdgcn_raw_buffer_load_b128(mk_rsrc(P.x, X_BYTES), lane * 16u + (u32)i * 1024u, e * 16384u + wave * 2048u, 0);
      call = rfl(ctr_v) / (u32)GRID;
      const __amdgpu_buffer_rsrc_t rs = mk_rsrc(myws + OFF_DATA + (call & 1u) * PAR_BYTES + REL_EXP, NEXP * EXPS);
#pragma unroll
      for (int i = 0; i < 8; ++i)
        __builtin_amdgcn_raw_buffer_store_b128(v[i], rs, (MG / 128 + wpan + (u32)(i >> 1)) * EXPS + wrow * 128u + (u32)(i & 1) * 1024u + lane * 16u, 0, AUX_SYS);
#pragma unroll
      for (int i = 0; i < 2; ++i)
        __builtin_amdgcn_raw_buffer_store_b128(v[8 + i], rs, (e >> 2) * EXPS + (e & 3u) * 16384u + wave * 2048u + (u32)i * 1024u + lane * 16u, 0, AUX_SYS);
      VMCNT(0);
      RAW_BARRIER();
      if (threadIdx.x == 0) __hip_atomic_fetch_add((GLB_AS u32*)(myws + OFF_XC), 1u, __ATOMIC_RELAXED, __HIP_MEMORY_SCOPE_AGENT);
    }
    call = rfl(ctr_v) / (u32)GRID;
  }
  gemm_role(P, smem, call + 1u, OFF_DATA + (call & 1u) * PAR_BYTES, t0);
  if (threadIdx.x == 0)
    __hip_atomic_fetch_add((GLB_AS u32*)(P.wsd[0] + OFF_CTR), 1u, __ATOMIC_RELAXED, __HIP_MEMORY_SCOPE_AGENT);
}

struct Tune { int nrel, margin_p, ownx_after, own_delay, relay_start, margin_o, go_step, cq, exp_delay, exp_spread; };

void build_schedule(Params& P, const Tune& t) {
  const int ndir = NB - t.nrel;
  P.ndir = ndir;
  P.margin[0] = t.margin_p;
  P.margin[1] = t.margin_o;
  P.go_step = t.go_step;
  P.exp_delay = t.exp_delay;
  P.exp_spread = t.exp_spread * 256 / NEXW;
  for (int cls = 0; cls < 3; ++cls) {
    unsigned char L[MAXU] = {};
    int n = 0, nw = 0;
    bool ownx = false;
    auto put = [&](u32 kind, int idx) { if (n < MAXU) L[n++] = (unsigned char)((kind << 6) | (u32)idx); };
    auto putw = [&](int b) {
      if (!ownx && nw >= t.ownx_after) { for (int a = 12; a < 16; ++a) put(K_X, a); ownx = true; }
      put(K_W, b); ++nw;
    };
    if (cls == 0) {
      for (int b = 0; b < ndir; ++b) putw(b);
      for (int key = -3 * t.cq; key <= 4 * (t.nrel - 1); ++key)
        for (int j = 3; j >= 0; --j) {
          const int k4 = key + t.cq * j;
          if (k4 < 0 || (k4 & 3) != 0 || k4 / 4 >= t.nrel) continue;
          if (j == 3) putw(ndir + k4 / 4); else put(K_FWD, 3 * (k4 / 4) + 2 - j);
        }
    } else {
      const int par = cls - 1;
      const int nr = (t.nrel + 1 - par) / 2;
      const int lead = t.relay_start < ndir ? t.relay_start : ndir;
      for (int b = 0; b < lead; ++b) putw(b);
      const int rest = ndir - lead, tot = rest + nr;
      for (int j = 0; j < tot; ++j) {
        const int r0 = j * nr / tot, r1 = (j + 1) * nr / tot;
        if (r1 > r0) putw(ndir + 2 * r0 + par); else putw(lead + j - r0);
      }
    }
    if (!ownx) for (int a = 12; a < 16; ++a) put(K_X, a);
    if (cls != 0) for (int o = 0; o < 2 * NB; ++o) put(K_OUT, o);
    P.nunits[cls] = n;
    for (int i = 0; i < MAXU / 4; ++i)
      P.ord[cls][i] = (u32)L[4 * i] | (u32)L[4 * i + 1] << 8 | (u32)L[4 * i + 2] << 16 | (u32)L[4 * i + 3] << 24;
  }
  for (int xp = 0; xp < 2; ++xp) {
    unsigned char T[2 * NB];
    int n = 0;
    for (int key = 0; key < NB + t.own_delay; ++key) {
      if (key < NB) {
        T[n++] = (unsigned char)key;
        if (((key ^ xp) & 1) == 0) T[n++] = (unsigned char)(32 | key);
      }
      const int ob = key - t.own_delay;
      if (ob >= 0 && ob < NB && ((ob ^ xp) & 1) != 0) T[n++] = (unsigned char)(32 | ob);
    }
    for (int i = 0; i < 16; ++i)
      P.tiles[xp][i] = (u32)T[4 * i] | (u32)T[4 * i + 1] << 8 | (u32)T[4 * i + 2] << 16 | (u32)T[4 * i + 3] << 24;
  }
}

}

extern "C" void kernel_launch(int rank, int nranks, ncclComm_t comm,
                              hipStream_t stream,
                              void* const* d_in, const int* in_sizes, int n_in,
                              void* d_out, int out_size,
                              const int* in_dtypes, int out_dtype,
                              void* const* d_out_all,
                              void* const* d_ws_all, size_t ws_bytes) {
  (void)comm; (void)in_dtypes; (void)out_dtype; (void)d_out_all;
  if (nranks != NR || n_in != 4 || rank < 0 || rank >= NR || ws_bytes < (size_t)WS_NEED ||
      in_sizes[0] != (int)X_BYTES || in_sizes[1] != (int)W_BYTES || out_size != (int)(OUT_BYTES / 4)) {
    fprintf(stderr, "kernel_launch: unexpected problem shape\n");
    return;
  }
  static bool attr_set[NR] = {};
  if (!attr_set[rank]) {
    HIP_CHECK(hipFuncSetAttribute((const void*)fused_gemm_rs, hipFuncAttributeMaxDynamicSharedMemorySize, LDS_BYTES));
    attr_set[rank] = true;
  }
  Params P;
  P.x = (const char*)d_in[0];
  P.w = (const char*)d_in[1];
  P.sx = (const float*)d_in[2];
  P.sw = (const float*)d_in[3];
  for (int i = 0; i < NR; ++i) P.wsd[i] = (char*)d_ws_all[rank ^ i];
  P.out = (char*)d_out;
  P.rank = rank;
  Tune t = {16, 2, 40, 32, 4, 2, 13, 3, 0, 300};
  build_schedule(P, t);
  hipLaunchKernelGGL(fused_gemm_rs, dim3(GRID), dim3(THREADS), LDS_BYTES, stream, P);
  HIP_CHECK(hipGetLastError());
}
```

```cpp
#include <hip/hip_runtime.h>
#include <rccl/rccl.h>
#include <cstdio>
#include <cstdint>

#define HIP_CHECK(expr) do { hipError_t _e = (expr); if (_e != hipSuccess) { \
    fprintf(stderr, "HIP error %s at %s:%d\n", hipGetErrorString(_e), __FILE__, __LINE__); \
    return; } } while (0)

namespace {

typedef unsigned u32;
typedef unsigned long long u64;
typedef unsigned u4 __attribute__((ext_vector_type(4)));
typedef int v2i __attribute__((ext_vector_type(2)));
typedef int v4i __attribute__((ext_vector_type(4)));
typedef int v8i __attribute__((ext_vector_type(8)));
typedef float v4f __attribute__((ext_vector_type(4)));
#define LDS_AS __attribute__((address_space(3)))
#define GLB_AS __attribute__((address_space(1)))

constexpr int NR = 8;
constexpr int MG = 4096, KG = 4096, NG = 16384;
constexpr int KS = KG / NR;
constexpr u32 PANEL = 128 * KS;
constexpr int NA = 16;
constexpr int NB = 32;
#define WPL_CFG 8
constexpr int WPL = WPL_CFG;
constexpr int NCOMM = 7 * WPL;
constexpr int GRID = 256;
constexpr int NGEMM = GRID - NCOMM;
constexpr int GPX = NGEMM / 8;

#define NWK_CFG 6
constexpr int NWK = NWK_CFG;
constexpr int THREADS = NWK > 6 ? 64 * (NWK + 2) : 512;
#define PDEPTH 3
constexpr int MAXU = 128;
constexpr int NSLOT = 512;

constexpr u32 OFF_CTR = 0, OFF_TMO = 128;
constexpr u32 OFF_XC = 256;
constexpr u32 OFF_TK = 1024;
constexpr u32 OFF_FX = 4096;
constexpr u32 OFF_FW = 8192;
constexpr u32 OFF_FR = 12288;
constexpr u32 OFF_GO = 16384;
constexpr u32 OFF_FI = 32768;
constexpr u32 OFF_DBG = 512 << 10;
constexpr u32 TILES = 12 * NB * 65536;
constexpr u32 OFF_PRE = 1 << 20;
constexpr u32 OFF_DATA = OFF_PRE + TILES;
constexpr u32 SRC_BYTES = (NA + NB) * PANEL;
constexpr u32 REL_XG = 0;
constexpr u32 REL_WG = NA * PANEL;
constexpr u32 REL_ST = 8 * SRC_BYTES;
constexpr u32 REL_EXP = REL_ST + TILES;
constexpr int NEXP = MG / 128 + NG / 128;
constexpr int NEXW = 128;
constexpr u32 EXPS = PANEL + 2048;
constexpr u32 PAR_BYTES = REL_EXP + NEXP * EXPS;
constexpr u32 WS_NEED = OFF_DATA + 2 * PAR_BYTES;

constexpr u32 X_BYTES = MG * KS;
constexpr u32 W_BYTES = KS * NG;
constexpr u32 OUT_BYTES = (MG / NR) * NG * 4;

constexpr int STAGE = 32768, NST = 4;
constexpr int LDS_CTRL = NST * STAGE;
constexpr int LDS_BYTES = LDS_CTRL + 256;

constexpr u64 DEADLINE = 50000000ull;

constexpr int AUX_SYS = 17;
constexpr int AUX_DEV = 16;

struct Params {
  const char* x;
  const char* w;
  char* wsd[NR];
  int rank;
  int ndir;
  int hurry;
  int go_step;
  int nunits[3];
  int margin[2];
  int exp_delay, exp_spread;
  const float* sx;
  const float* sw;
  char* out;
  u32 ord[3][MAXU / 4];
  u32 tiles[2][16];
};

__device__ __forceinline__ u32 rfl(u32 v) { return __builtin_amdgcn_readfirstlane(v); }
__device__ __forceinline__ __amdgpu_buffer_rsrc_t mk_rsrc(const void* p, u32 bytes) {
  return __builtin_amdgcn_make_buffer_rsrc((void*)p, (short)0, (int)bytes, 0x00020000);
}
__device__ __forceinline__ u32 ld_sys(const char* p) {
  return __hip_atomic_load((const GLB_AS u32*)p, __ATOMIC_RELAXED, __HIP_MEMORY_SCOPE_SYSTEM);
}
__device__ __forceinline__ void st_sys(char* p, u32 v) {
  __hip_atomic_store((GLB_AS u32*)p, v, __ATOMIC_RELAXED, __HIP_MEMORY_SCOPE_SYSTEM);
}
__device__ __forceinline__ u32 ld_dev(const char* p) {
  return __hip_atomic_load((const GLB_AS u32*)p, __ATOMIC_RELAXED, __HIP_MEMORY_SCOPE_AGENT);
}
__device__ __forceinline__ void st_dev(char* p, u32 v) {
  __hip_atomic_store((GLB_AS u32*)p, v, __ATOMIC_RELAXED, __HIP_MEMORY_SCOPE_AGENT);
}
#define VMCNT(n) asm volatile("s_waitcnt vmcnt(" #n ")" ::: "memory")
#define SCHED_FENCE() do { asm volatile("" ::: "memory"); __builtin_amdgcn_sched_barrier(0); } while (0)
#define RAW_BARRIER() do { asm volatile("s_waitcnt lgkmcnt(0)" ::: "memory"); __builtin_amdgcn_s_barrier(); asm volatile("" ::: "memory"); } while (0)

__device__ __forceinline__ bool give_up(char* myws, u32 seq, u64 t0, u32& round) {
  if ((++round & 7u) != 0u) return false;
  const u32 t = rfl(ld_dev(myws + OFF_TMO));
  const bool late = (__builtin_amdgcn_s_memrealtime() - t0) > DEADLINE;
  if (t == seq) return true;
  if (late) {
    st_dev(myws + OFF_TMO, seq);
    return true;
  }
  return false;
}

#define STAMP(slot) do {} while (0)
#define PVAL(slot, v) do {} while (0)
#define PNOW() 0ull

__device__ __forceinline__ v4f silu4(v4f y) {
  v4f o;
#pragma unroll
  for (int j = 0; j < 4; ++j) o[j] = __builtin_amdgcn_exp2f(__builtin_amdgcn_fmed3f(y[j], -60.f, 60.f) * -1.44269504088896f);
  o = o + 1.f;
#pragma unroll
  for (int j = 0; j < 4; ++j) o[j] = __builtin_amdgcn_rcpf(o[j]);
  return o * y;
}

__device__ __forceinline__ int grow_of(int p, int a) {
  if (a < 12) return ((p ^ (2 + (a >> 1))) << 2) + ((p & 1) << 1) + (a & 1);
  return (p << 2) + (a - 12);
}

constexpr u32 K_X = 0, K_W = 1, K_FWD = 2, K_OUT = 3;
constexpr u32 D_END = 1, D_IN = 2, D_INR = 3, D_OUT = 4, D_OUTH = 5;

__device__ __forceinline__ void comm_role(const Params& P, char* smem, u64 t0) {
  const int tid = threadIdx.x, lane = tid & 63;
  const int wave = rfl(tid >> 6);
  const int w = (int)blockIdx.x / 7;
  const int d = (int)blockIdx.x - 7 * w + 1;
  const int r = P.rank;
  char* myws = P.wsd[0];
  char* pws = P.wsd[d];
  const u32 call = rfl(ld_dev(myws + OFF_CTR)) / (u32)GRID;
  const u32 seq = call + 1u, parbase = OFF_DATA + (call & 1u) * PAR_BYTES;
  const u32 p = (u32)(r ^ d) & 7u;
  u32 round = 0;

  volatile LDS_AS u4* desc = (volatile LDS_AS u4*)smem;
  volatile LDS_AS u32* cnt = (volatile LDS_AS u32*)(smem + NSLOT * 16);
  volatile LDS_AS u32* gate = cnt + NSLOT - 2;
  volatile LDS_AS u32* cctr = cnt + NSLOT - 3;
  static_assert(NSLOT <= THREADS, "descriptor init");
  if (tid < NSLOT) {
    desc[tid] = u4{0u, 0u, 0u, 0u};
    cnt[tid] = tid == NSLOT - 3 ? 2u * NWK : 0u;
  }
  RAW_BARRIER();

  if (__builtin_expect(wave == THREADS / 64 - 1, 0)) {
    char* partner = P.wsd[1];
    bool ok = true;
    while (rfl(ld_sys(myws + OFF_GO + (u32)(d * WPL + w) * 256u)) < seq) {
      if (give_up(myws, seq, t0, round)) { ok = false; break; }
      __builtin_amdgcn_s_sleep(4);
    }
    *gate = ok ? 1u : 2u;
    STAMP(0);
    for (int t = 0;; ++t) {
      u32 k;
      for (;;) {
        k = rfl(((volatile LDS_AS u32*)(desc + t))[3]);
        if (k != 0u && ((k >> 8) == 0u || !ok || rfl(cnt[t]) == 8u)) break;
        __builtin_amdgcn_s_sleep(1);
      }
      if (k == D_END) break;
      if (!ok) continue;
      if ((k >> 8) != 0u) st_sys(myws + (k >> 8), seq);
      if ((k & 255u) == D_INR) st_sys(partner + ((k >> 8) & ~16u) + (OFF_FR - OFF_FW), seq);
    }
    STAMP(3);
    if (lane == 0) __hip_atomic_fetch_add((GLB_AS u32*)(myws + OFF_CTR), 1u, __ATOMIC_RELAXED, __HIP_MEMORY_SCOPE_AGENT);
  } else if (__builtin_expect(wave == THREADS / 64 - 2, 0)) {
    const int cls = d == 1 ? 0 : 1 + (d & 1);
    const int NP = 4 * P.nunits[cls];
    const int NDIR = P.ndir;
    const __amdgpu_buffer_rsrc_t rs_my = mk_rsrc(myws, WS_NEED);
    u64 stall = 0;
    u32 woff = OFF_FI + (u32)d * 1024u + (u32)lane * 16u;
    if (d == 1) {
      const int fi = lane < 3 * (NB - NDIR) ? lane : 0;
      const int k = fi / 3, i = fi - 3 * k;
      woff = OFF_FR + (u32)(((NDIR + k) * 8 + (((int)p ^ (2 * i + 2)) & 6)) * 16);
    }
    const u32 tab = P.ord[cls][lane & 31];
    u64 rdy = 0;
    auto poll = [&]() {
      const u4 f = __builtin_amdgcn_raw_buffer_load_b128(rs_my, woff, 0, AUX_SYS);
      rdy |= __ballot(f.x >= seq && f.y >= seq && f.z >= seq && f.w >= seq);
    };
    auto post_end = [&](int s) {
      if (lane < 4) ((volatile LDS_AS u32*)(desc + s + lane))[3] = D_END;
    };
    const u32 tkbase = call * (u32)(NP + WPL);
    char* tkptr = myws + OFF_TK + (u32)d * 128u;
    const int margin = P.margin[d != 1];
    bool dead = false;
    int s = 0;
    u64 rtt = 0;
    for (;;) {
      while (!dead && (int)rfl(*cctr) + margin <= 8 * s) {
        if (rfl(*gate) == 2u) { dead = true; post_end(s); }
        __builtin_amdgcn_s_sleep(1);
      }
      const u64 tr0 = PNOW();
      const u4 f = __builtin_amdgcn_raw_buffer_load_b128(rs_my, woff, 0, AUX_SYS);
      u32 old = 0;
      if (lane == 0) old = __hip_atomic_fetch_add((GLB_AS u32*)tkptr, 1u, __ATOMIC_RELAXED, __HIP_MEMORY_SCOPE_AGENT);
      const int tk = (int)(rfl(old) - tkbase);
      rdy |= __ballot(f.x >= seq && f.y >= seq && f.z >= seq && f.w >= seq);
      rtt += PNOW() - tr0;
      if ((u32)tk >= (u32)NP) break;
      if (dead) continue;
      const int pos = tk >> 2;
      const u32 piece = (u32)tk & 3u;
      const u32 ent = ((u32)__builtin_amdgcn_readlane((int)tab, pos >> 2) >> (8 * (pos & 3))) & 255u;
      const u32 kind = ent >> 6;
      const int idx = (int)(ent & 63u);
      const bool split = (NDIR & 1) != 0;
      if (split && kind == K_W && d != 1 && idx == NDIR && (((u32)d ^ piece) & 1u) != 0u) continue;
      u64 sa, da;
      u32 k3;
      if (kind == K_X) {
        sa = (u64)pws + parbase + REL_EXP + (u32)grow_of(r, idx) * EXPS;
        da = (u64)myws + parbase + REL_XG + p * SRC_BYTES + (u32)idx * PANEL;
        k3 = (OFF_FX + ((u32)idx * 8u + p) * 16u + piece * 4u) << 8 | D_IN;
      } else if (kind == K_W) {
        sa = (u64)pws + parbase + REL_EXP + (u32)(MG / 128 + (r >> 1) * NB + idx) * EXPS;
        da = (u64)myws + parbase + REL_WG + p * SRC_BYTES + (u32)idx * PANEL;
        k3 = (OFF_FW + ((u32)idx * 8u + p) * 16u + piece * 4u) << 8 | (d != 1 && idx >= NDIR ? D_INR : D_IN);
      } else if (kind == K_FWD) {
        const int kk = idx / 3, i = idx - 3 * kk;
        const u32 q = p ^ ((u32)(2 * i + 2) + ((split && kk == 0 ? piece : (u32)kk) & 1u)), b = (u32)(NDIR + kk);
        sa = (u64)pws + parbase + REL_WG + q * SRC_BYTES + b * PANEL;
        da = (u64)myws + parbase + REL_WG + q * SRC_BYTES + b * PANEL;
        k3 = (OFF_FW + (b * 8u + q) * 16u + piece * 4u) << 8 | D_IN;
      } else {
        const int b = idx >> 1, tt = idx & 1;
        sa = (u64)pws + parbase + REL_ST + (u32)(((d - 2) * 2 + tt) * NB + b) * 65536u;
        da = (u64)P.out + ((2u * (p & 1u) + (u32)tt) * 128u + piece * 32u) * (NG * 4u) + ((p >> 1) * NB + (u32)b) * 512u - piece * 16384u;
        k3 = tk >= NP - P.hurry || !((rdy >> idx) & 1ull) ? D_OUTH : D_OUT;
      }
      sa += piece * 16384u;
      da += piece * 16384u;
      if (kind >= K_FWD && !((rdy >> idx) & 1ull)) {
        const u64 tw0 = PNOW();
        for (;;) {
          poll();
          if ((rdy >> idx) & 1ull) break;
          if (give_up(myws, seq, t0, round)) { dead = true; break; }
          __builtin_amdgcn_s_sleep(2);
        }
        stall += PNOW() - tw0;
      }
      if (dead) {
        if (kind == K_OUT) st_sys((char*)da, 0x7fc00000u);
        post_end(s);
        continue;
      }
      asm volatile("" ::: "memory");
      if (lane == 0) {
        volatile LDS_AS u32* dp = (volatile LDS_AS u32*)(desc + s);
        dp[0] = (u32)sa; dp[1] = (u32)da; dp[2] = (u32)(sa >> 32) | (u32)(da >> 32) << 16; dp[3] = k3;
      }
      ++s;
    }
    if (!dead) post_end(s);
    PVAL(4, stall);
    PVAL(5, (u64)s);
    PVAL(6, rtt);
  } else if (wave < NWK) {
    const u32 v_lin = (u32)lane * 16u;
    const u32 v_out = (u32)(lane >> 5) * (NG * 4u) + (u32)(lane & 31) * 16u;
    const u32 v_gat = (u32)((lane & 31) >> 2) * 1024u + (u32)(lane >> 5) * 64u + (u32)(lane & 3) * 16u;
    const u32 cnt0 = (u32)(uintptr_t)cnt;
    struct D { u64 s0, s1, d0, d1; u32 kind, vs, vd; };
    auto fetch = [=](u32 j) -> D {
      const u4 e = desc[j >> 3];
      const u32 sub = j & 7u, h = rfl(e.z);
      D c;
      c.kind = rfl(e.w) & 255u;
      const bool iso = c.kind >= D_OUT;
      c.s0 = ((u64)(h & 0xffffu) << 32 | rfl(e.x)) + (iso ? (sub >> 2) * 8192u + (sub & 3u) * 256u : sub * 2048u);
      c.s1 = c.s0 + (iso ? 128u : 1024u);
      c.d0 = ((u64)(h >> 16) << 32 | rfl(e.y)) + (iso ? sub * (4u * NG * 4u) : sub * 2048u);
      c.d1 = c.d0 + (iso ? 2u * NG * 4u : 1024u);
      c.vs = c.vd = v_lin;
      if (iso) { c.vs = v_gat; c.vd = v_out; }
      return c;
    };
    auto take = [=]() -> u32 {
      u64 keep;
      u32 v;
      asm volatile("s_mov_b64 %0, exec\n\ts_mov_b64 exec, 1\n\tds_add_rtn_u32 %1, %2, %3\n\ts_mov_b64 exec, %0\n\ts_waitcnt lgkmcnt(0)"
                   : "=&s"(keep), "=&v"(v) : "v"(cnt0 + 4u * (NSLOT - 3)), "v"(1u) : "memory");
      return rfl(v);
    };
    auto arrive = [&](u32 j) {
      u64 keep;
      asm volatile("s_mov_b64 %0, exec\n\ts_mov_b64 exec, 1\n\tds_add_u32 %1, %2\n\ts_mov_b64 exec, %0"
                   : "=&s"(keep) : "v"(cnt0 + 4u * (j >> 3)), "v"(1u) : "memory");
    };
    auto u64s = [](u64 v) { return (u64)rfl((u32)(v >> 32)) << 32 | rfl((u32)v); };
    auto step = [&](u4& in, u64 sa, u32 vs, const u4& out, u64 da, u32 vd) {
      asm volatile("s_nop 4\n\ts_waitcnt vmcnt(1)\n\tglobal_load_dwordx4 %0, %1, %2 sc0 sc1\n\t"
                   "global_store_dwordx4 %3, %4, %5 sc0 sc1\n\ts_nop 1"
                   : "+v"(in) : "v"(vs), "s"(u64s(sa)), "v"(vd), "v"(out), "s"(u64s(da)) : "memory");
    };
    auto step2 = [&](u4& ina, u4& inb, u64 sa, u64 sb, u32 vs, u4& ra, u4& rb, u32 early) {
      asm volatile("s_nop 4\n\ts_cmp_lg_u32 %7, 0\n\ts_cbranch_scc1 1f\n\ts_waitcnt vmcnt(1)\n1:\n\t"
                   "global_load_dwordx4 %0, %4, %5 sc0 sc1\n\tglobal_load_dwordx4 %1, %4, %6 sc0 sc1\n\ts_waitcnt vmcnt(2)"
                   : "+v"(ina), "+v"(inb), "+v"(ra), "+v"(rb) : "v"(vs), "s"(u64s(sa)), "s"(u64s(sb)), "s"(rfl(early)) : "scc", "memory");
    };
    const float scale = P.sx[0] * P.sw[0];
    auto finish = [&](const u4& ra, const u4& rb, u64 da, u32 vd) {
      const v4f o = silu4((__builtin_bit_cast(v4f, ra) + __builtin_bit_cast(v4f, rb)) * scale);
      asm volatile("s_nop 4\n\tglobal_store_dwordx4 %0, %1, %2 sc0 sc1\n\ts_nop 1" :: "v"(vd), "v"(o), "s"(u64s(da)) : "memory");
    };
    const u64 to_pre = (u64)myws + OFF_PRE - ((u64)pws + parbase + REL_ST);
    constexpr u32 NONE = 0xffffffffu;
    u32 ja = (u32)wave, jb = (u32)wave + NWK, owed = NONE;
    u4 a0 = u4{0u, 0u, 0u, 0u}, a1 = a0, b0 = a0, b1 = a0;
    u64 nruns = 0;
    u32 go;
    while ((go = rfl(*gate)) == 0u) __builtin_amdgcn_s_sleep(1);
    while (go == 1u) {
      ++nruns;
      D C = fetch(ja);
      while (C.kind == 0u) { __builtin_amdgcn_s_sleep(1); C = fetch(ja); }
      if (C.kind == D_END) break;
      asm volatile("s_nop 4\n\tglobal_load_dwordx4 %0, %1, %2 sc0 sc1" : "+v"(a0) : "v"(C.vs), "s"(u64s(C.s0)) : "memory");
      D N = fetch(jb);
      bool nxt = true;
      while (C.kind < D_OUT) {
        VMCNT(0);
        step(a1, C.s1, C.vs, a0, C.d0, C.vd);
        if (owed != NONE) arrive(owed);
        owed = ja;
        const u32 jc = take();
        const D N2 = fetch(jc);
        if (__builtin_expect(N.kind == 0u, 0)) { VMCNT(1); N = fetch(jb); }
        nxt = N.kind >= D_IN;
        step(a0, nxt ? N.s0 : C.d0, nxt ? N.vs : C.vd, a1, C.d1, C.vd);
        ja = jb;
        jb = jc;
        if (!nxt) break;
        C = N;
        N = N2;
      }
      if (nxt) asm volatile("s_nop 4\n\tglobal_load_dwordx4 %0, %1, %2 sc0 sc1" : "+v"(b0) : "v"(C.vs), "s"(u64s(C.s0 + to_pre)) : "memory");
      if (owed != NONE || C.kind != D_OUTH) VMCNT(0);
      if (owed != NONE) { arrive(owed); owed = NONE; }
      while (nxt) {
        step2(a1, b1, C.s1, C.s1 + to_pre, v_gat, a0, b0, C.kind - D_OUT);
        finish(a0, b0, C.d0, v_out);
        const u32 jc = take();
        const D N2 = fetch(jc);
        if (__builtin_expect(N.kind == 0u, 0)) { VMCNT(1); N = fetch(jb); }
        nxt = N.kind >= D_OUT;
        const u64 sb = (nxt ? N.s0 : C.s0) + to_pre;
        step2(a0, b0, nxt ? N.s0 : sb, sb, v_gat, a1, b1, 0u);
        finish(a1, b1, C.d1, v_out);
        ja = jb;
        jb = jc;
        C = N;
        N = N2;
        if (!nxt) VMCNT(0);
      }
    }
    PVAL(40 + wave, nruns);
  }
}

__device__ __forceinline__ void glds16(__amdgpu_buffer_rsrc_t rs, u32 lds, u32 voff, u32 soff) {
  asm volatile("s_mov_b32 m0, %0\n\ts_nop 3\n\tbuffer_load_dwordx4 %1, %2, %3 offen lds"
               :: "s"(rfl(lds)), "v"(voff), "s"(rs), "s"(rfl(soff)) : "m0", "memory");
}

__device__ __forceinline__ void gemm_role(const Params& P, char* smem, u32 seq, u32 parbase, u64 t0) {
  const int tid = threadIdx.x, lane = tid & 63;
  const int wave = rfl(tid >> 6);
  const int r = P.rank;
  char* myws = P.wsd[0];
  const int xg = (int)blockIdx.x & 7, kx = ((int)blockIdx.x - NCOMM) >> 3;
  constexpr int gpx = GPX;
  const u32 lds0 = (u32)(uintptr_t)(LDS_AS char*)smem;

  const __amdgpu_buffer_rsrc_t rs_x = mk_rsrc(P.x, X_BYTES);
  const __amdgpu_buffer_rsrc_t rs_w = mk_rsrc(P.w, W_BYTES);
  const __amdgpu_buffer_rsrc_t rs_ws = mk_rsrc(myws, WS_NEED);
  const float scale = P.sx[0] * P.sw[0];

  u32 vX[2], vWs[2], vWo[2];
#pragma unroll
  for (int u = 0; u < 2; ++u) {
    const int row = wave * 16 + u * 8 + (lane >> 3), slot = lane & 7;
    const int cx = slot ^ ((row >> 1) & 7);
    const int cw = slot ^ (((row >> 1) & 3) | (((row >> 4) & 1) << 2));
    vX[u] = (u32)row * KS + (u32)cx * 16u;
    vWs[u] = (u32)row * 128u + (u32)cw * 16u;
    vWo[u] = (u32)row * NG + (u32)cw * 16u;
  }
  const int wm = wave >> 2, wn = wave & 3, g = lane >> 4, l15 = lane & 15;
  const int aX0 = (wm * 64 + l15) * 128 + ((g ^ (l15 >> 1)) * 16);
  const int aX1 = aX0 ^ 64;
  const int fw = ((l15 >> 2) & 3) | ((g & 1) << 2);
  const int aW0 = 16384 + (16 * g + (l15 >> 1)) * 128 + (((2 * wn) ^ fw) * 16) + (l15 & 1) * 8;
  const int aW1 = aW0 ^ 16;
  volatile LDS_AS u32* ctrl = (volatile LDS_AS u32*)(smem + LDS_CTRL);
  u32 round = 0;

  for (int it = kx >= 7 && kx < 23 ? -3 : 0; kx + it * gpx < 2 * NB; ++it) {
    const bool pre = it < 0;
    const int e = pre ? 0 : kx + it * gpx;
    const u32 ent = (P.tiles[xg & 1][e >> 2] >> (8 * (e & 3))) & 255u;
    int b = (int)(ent & 31u);
    int a = (ent >> 5) == 0u ? xg : ((((b ^ xg) & 1) == 0 ? 8 : 12) + (xg >> 1));
    u32 sX_own = (u32)grow_of(r, a) * PANEL;
    u32 sW_own = (u32)((r >> 1) * NB + b) * 128u;
    if (pre) {
      const int u = (it + 3) * 16 + kx - 7, fc = (u >> 2) * 8 + xg, rt = u & 3;
      b = fc / 3;
      const int pg = (r >> 1) ^ (fc - 3 * b + 1);
      a = ((r ^ (2 * pg + (rt >> 1))) - 2) * 2 + (rt & 1);
      sX_own = (u32)(r * 4 + rt) * PANEL;
      sW_own = (u32)(pg * NB + b) * 128u;
    }
    const bool own = a >= 12;
    const int nsl = pre ? 1 : (own ? 8 : 7);
    const int NDIR = P.ndir;
    u32 pref;
    if (b < NDIR) pref = 0x17654320u;
    else pref = ((b - NDIR) & 1) ? 0x64217530u : 0x75316420u;
    u32 ordd = pre ? 8u : 0u;
    const u32 sX_ws = parbase + REL_XG + (u32)a * PANEL;
    const u32 sW_ws = parbase + REL_WG + (u32)b * PANEL;
    const u32 foff = lane < 32 ? OFF_FX + (u32)a * 128u + (u32)lane * 4u : OFF_FW + (u32)b * 128u + (u32)(lane - 32) * 4u;

    v4f acc[2][4];
#pragma unroll
    for (int nb = 0; nb < 2; ++nb)
#pragma unroll
      for (int mb = 0; mb < 4; ++mb) acc[nb][mb] = v4f{0.f, 0.f, 0.f, 0.f};

    auto issue = [&](int kt, int stage) {
      const int dq = (ordd >> (4 * (kt >> 2))) & 7;
      const u32 q = (u32)(r ^ dq);
      const u32 kb = (u32)(kt & 3);
      const u32 ldsb = lds0 + (u32)stage * STAGE + (u32)wave * 2048u;
      if (dq == 0) {
        const u32 sx = sX_own + kb * 128u, sw = sW_own + kb * 128u * NG;
        glds16(rs_x, ldsb, vX[0], sx);
        glds16(rs_x, ldsb + 1024u, vX[1], sx);
        glds16(rs_w, ldsb + 16384u, vWo[0], sw);
        glds16(rs_w, ldsb + 16384u + 1024u, vWo[1], sw);
      } else {
        const u32 sx = sX_ws + q * SRC_BYTES + kb * 128u, sw = sW_ws + q * SRC_BYTES + kb * 16384u;
        glds16(rs_ws, ldsb, vX[0], sx);
        glds16(rs_ws, ldsb + 1024u, vX[1], sx);
        glds16(rs_ws, ldsb + 16384u, vWs[0], sw);
        glds16(rs_ws, ldsb + 16384u + 1024u, vWs[1], sw);
      }
    };
    auto compute = [&](int stage) {
      const LDS_AS char* sb = (const LDS_AS char*)smem + stage * STAGE;
      v8i xf[4], wf[2];
#pragma unroll
      for (int nb = 0; nb < 2; ++nb) {
        const LDS_AS char* pw = sb + (nb ? aW1 : aW0);
        const v2i t0_ = __builtin_amdgcn_ds_read_tr8_b64_v2i32((LDS_AS v2i*)(pw));
        const v2i t1_ = __builtin_amdgcn_ds_read_tr8_b64_v2i32((LDS_AS v2i*)(pw + 1024));
        const v2i t2_ = __builtin_amdgcn_ds_read_tr8_b64_v2i32((LDS_AS v2i*)(pw + 8192));
        const v2i t3_ = __builtin_amdgcn_ds_read_tr8_b64_v2i32((LDS_AS v2i*)(pw + 9216));
        wf[nb] = v8i{t0_.x, t0_.y, t1_.x, t1_.y, t2_.x, t2_.y, t3_.x, t3_.y};
      }
#pragma unroll
      for (int mb = 0; mb < 4; ++mb) {
        const v4i lo = *(const LDS_AS v4i*)(sb + aX0 + mb * 2048);
        const v4i hi = *(const LDS_AS v4i*)(sb + aX1 + mb * 2048);
        xf[mb] = v8i{lo.x, lo.y, lo.z, lo.w, hi.x, hi.y, hi.z, hi.w};
      }
#pragma unroll
      for (int mb = 0; mb < 4; ++mb)
#pragma unroll
        for (int nb = 0; nb < 2; ++nb)
          acc[nb][mb] = __builtin_amdgcn_mfma_scale_f32_16x16x128_f8f6f4(wf[nb], xf[mb], acc[nb][mb], 1, 0, 0, 0, 0, 0);
    };

    u32 taken = own ? 0u : 4u << (a >> 1);
    int nass = pre ? 1 : 0;
    auto assign = [&](u32 f) {
      const u64 m = __ballot((((lane & 31) >> 2) == r) || f >= seq);
      const u32 m32 = (u32)m & (u32)(m >> 32);
      for (int i = 0; i < 8; ++i) {
        const u32 dq = (pref >> (4 * i)) & 7u;
        if (!((taken >> dq) & 1u) && ((m32 >> (4u * ((u32)r ^ dq))) & 15u) == 15u) {
          taken |= 1u << dq;
          ordd |= (8u | dq) << (4 * nass);
          ++nass;
        }
      }
    };
    auto wait_slices = [&](int n) {
      for (;;) {
        assign(__builtin_amdgcn_raw_buffer_load_b32(rs_ws, foff, 0, AUX_SYS));
        if (nass >= n) break;
        if (give_up(myws, seq, t0, round)) {
          while (nass < nsl) { ordd |= 8u << (4 * nass); ++nass; }
          ctrl[1] = 1u;
          break;
        }
        __builtin_amdgcn_s_sleep(4);
      }
    };
    auto avail = [&](int sl) { return ((ordd >> (4 * sl)) & 8u) != 0u; };

    if (wave == 0) { if (it != -1 && it != -2) STAMP(sb + 0); ctrl[0] = 0u; ctrl[1] = 0u; }
    for (int sl = 0; sl < nsl;) {
      if (wave == 0) {
        if (!pre) {
          wait_slices(sl == 0 ? 2 : sl + 1);
          if (sl == 0) {
            __builtin_amdgcn_fence(__ATOMIC_ACQUIRE, "");
            VMCNT(0);
            STAMP(sb + 1);
          }
        }
        ctrl[2] = ordd;
      }
      RAW_BARRIER();
      ordd = rfl(ctrl[2]);
      issue(4 * sl, 0); issue(4 * sl + 1, 1); issue(4 * sl + 2, 2); issue(4 * sl + 3, 3);
      for (bool last = false, fresh = true; !last; ++sl, fresh = false) {
        const int kt = sl * 4;
        if (fresh) VMCNT(12); else VMCNT(8);
        RAW_BARRIER();
        u32 pf;
        const u32 po = (wave == 0 && nass < nsl) ? foff : 0xfffffff0u;
        asm volatile("buffer_load_dword %0, %1, %2, 0 offen sc0 sc1" : "=v"(pf) : "v"(po), "s"(rs_ws) : "memory");
        if (!fresh) issue(kt + 3, 3);
        compute(0);
        last = sl == nsl - 1 || !avail(sl + 1);
        VMCNT(8); RAW_BARRIER();
        if (!last) issue(kt + 4, 0);
        compute(1);
        if (!last) VMCNT(8); else VMCNT(4);
        RAW_BARRIER();
        if (!last) issue(kt + 5, 1);
        else asm volatile("buffer_load_dword %0, %1, %2, 0 offen sc0 sc1" : "+v"(pf) : "v"(po), "s"(rs_ws) : "memory");
        compute(2);
        if (!last) VMCNT(8); else VMCNT(0);
        asm volatile("" : "+v"(pf));
        if (wave == 0 && nass < nsl) { assign(pf); ctrl[0] = ordd; }
        RAW_BARRIER();
        if (!avail(nsl - 1)) ordd |= rfl(ctrl[0]);
        if (last && sl < nsl - 1 && avail(sl + 1)) {
          issue(kt + 4, 0); issue(kt + 5, 1);
          last = false;
        }
        if (!last) issue(kt + 6, 2);
        compute(3);
      }
    }
    if (wave == 0 && !pre) STAMP(sb + 3);

    {
      const bool bad = rfl(ctrl[1]) != 0u;
      const __amdgpu_buffer_rsrc_t rs_o = mk_rsrc(own ? P.out : myws, own ? OUT_BYTES : WS_NEED);
      const u32 rs = own ? NG * 4u : 64u, rbs = own ? 16u * NG * 4u : 8192u, cbs = own ? 64u : 1024u;
      const u32 so = (own ? (u32)(a - 12) * 128u * (NG * 4u) + (u32)((r >> 1) * NB + b) * 512u
                          : (pre ? OFF_PRE : parbase + REL_ST) + (u32)(a * NB + b) * 65536u) +
                     (u32)wm * 4u * rbs + (u32)wn * 2u * cbs;
      const u32 vo = (u32)l15 * rs + (u32)g * 16u;
      const float sc = bad ? __builtin_nanf("") : scale;
      SCHED_FENCE();
      asm volatile("s_nop 15\n\ts_nop 7" ::: "memory");
#pragma unroll
      for (int mb = 0; mb < 4; ++mb)
#pragma unroll
        for (int nb = 0; nb < 2; ++nb) {
          v4f o = acc[nb][mb];
          if (own) o = silu4(o * sc);
          asm volatile("s_nop 4\n\tbuffer_store_dwordx4 %0, %1, %2, %3 offen sc0 sc1\n\ts_nop 1"
                       :: "v"(o), "v"(vo), "s"(rs_o), "s"(so + (u32)mb * rbs + (u32)nb * cbs) : "memory");
        }
      VMCNT(0);
      RAW_BARRIER();
      if (!own && !bad && tid < 2) {
        const u32 dd = 2u + (u32)(a >> 1);
        st_sys((pre ? myws + 8 : P.wsd[dd]) + OFF_FI + dd * 1024u + (u32)(b * 2 + (a & 1)) * 16u + (u32)tid * 4u, seq);
      }
    }
    if (wave == 0) STAMP(pre ? 28 : sb + 4);
  }
}

__global__ void __launch_bounds__(THREADS, THREADS > 512 ? 3 : 2) fused_gemm_rs(Params P) {
  extern __shared__ __attribute__((aligned(16))) char smem[];
  const u64 t0 = __builtin_amdgcn_s_memrealtime();
  const u32 gi = blockIdx.x - (u32)NCOMM, lane = threadIdx.x & 63u, wave = rfl(threadIdx.x >> 6);
  u32 call = 0;
  if (THREADS > 512 && blockIdx.x >= (u32)NCOMM && wave >= 8u) return;
  if (__builtin_expect(gi < (u32)NCOMM, 1)) {
    const u32 w = gi / 7u, d = gi - 7u * w + 1u;
    const u32 r = (u32)P.rank;
    const char* px = P.x;
    char* myws = P.wsd[0];
    char* pws = P.wsd[d];
    const u32 go_step = (u32)P.go_step;
    asm volatile("" :: "s"(r), "s"(px), "s"(myws), "s"(pws), "s"(go_step));
    const u32 ctr_v = ld_dev(myws + OFF_CTR);
    const u32 tk = w * 6u + wave, piece = tk & 3u;
    const u32 a = (tk >> 3) + 2u == d ? 12u + ((tk >> 2) & 1u) : tk >> 2;
    const __amdgpu_buffer_rsrc_t rs = mk_rsrc(px + (u32)grow_of((int)(r ^ d), (int)a) * PANEL + piece * 16384u, wave < 6u ? 16384u : 0u);
    const u32 rot = tk + 5u * d;
    u4 v[16];
#pragma unroll
    for (int i = 0; i < PDEPTH; ++i) v[i] = __builtin_amdgcn_raw_buffer_load_b128(rs, lane * 16u, ((rot + (u32)i) & 15u) * 1024u, 0);
    volatile LDS_AS u32* mark = (volatile LDS_AS u32*)(smem + LDS_CTRL + 64);
    *mark = 0u;
    RAW_BARRIER();
    call = rfl(ctr_v) / (u32)GRID;
    if (wave < 6u) {
      const __amdgpu_buffer_rsrc_t rd = mk_rsrc(pws + OFF_DATA + (call & 1u) * PAR_BYTES + REL_XG + r * SRC_BYTES + a * PANEL + piece * 16384u, 16384u);
#pragma unroll
      for (int i = 0; i < 16; ++i) {
        if (i != 0) { if (i - 1 + PDEPTH < 16) VMCNT(1); else VMCNT(0); }
        __builtin_amdgcn_raw_buffer_store_b128(v[i], rd, lane * 16u, ((rot + (u32)i) & 15u) * 1024u, AUX_SYS);
        asm volatile("s_nop 1" ::: "memory");
        if (i + PDEPTH < 16) v[i + PDEPTH] = __builtin_amdgcn_raw_buffer_load_b128(rs, lane * 16u, ((rot + (u32)(i + PDEPTH)) & 15u) * 1024u, 0);
        if (wave == 0u && go_step == (u32)i) *mark = 1u;
        SCHED_FENCE();
      }
      VMCNT(0);
      if (wave == 0u) *mark = 1u;
      st_sys(pws + OFF_FX + (a * 8u + r) * 16u + piece * 4u, call + 1u);
      STAMP(54 + wave);
    } else if (wave == 6u) {
      u32 round = 0;
      bool ok = true;
      while (rfl(*mark) == 0u) __builtin_amdgcn_s_sleep(1);
      STAMP(61);
      while (rfl(ld_sys(myws + OFF_XC)) < (call + 1u) * (u32)NEXW) {
        if (give_up(myws, call + 1u, t0, round)) { ok = false; break; }
        __builtin_amdgcn_s_sleep(16);
      }
      if (ok) st_sys(pws + OFF_GO + (d * WPL + w) * 256u, call + 1u);
      STAMP(60);
    }
  } else if (blockIdx.x < (u32)NCOMM) {
    comm_role(P, smem, t0);
    return;
  } else {
    char* myws = P.wsd[0];
    const u32 ctr_v = ld_dev(myws + OFF_CTR);
    const u32 e = gi - (u32)NCOMM;
    const u32 wrow = (e & 31u) * 16u, wpan = (e >> 5) * 32u + wave * 4u;
    if (e < (u32)NEXW) {
      const u64 tgo = (u64)((u32)P.exp_delay + (e * (u32)P.exp_spread >> 8));
      while (__builtin_amdgcn_s_memrealtime() - t0 < tgo) __builtin_amdgcn_s_sleep(2);
      u4 v[10];
#pragma unroll
      for (int i = 0; i < 8; ++i)
        v[i] = __builtin_amdgcn_raw_buffer_load_b128(mk_rsrc(P.w, W_BYTES), (lane >> 3) * NG + (lane & 7u) * 16u,
                                                     (wrow + (u32)(i & 1) * 8u) * NG + (wpan + (u32)(i >> 1)) * 128u, 0);
#pragma unroll
      for (int i = 0; i < 2; ++i)
        v[8 + i] = __builtin_amdgcn_raw_buffer_load_b128(mk_rsrc(P.x, X_BYTES), lane * 16u + (u32)i * 1024u, e * 16384u + wave * 2048u, 0);
      call = rfl(ctr_v) / (u32)GRID;
      const __amdgpu_buffer_rsrc_t rs = mk_rsrc(myws + OFF_DATA + (call & 1u) * PAR_BYTES + REL_EXP, NEXP * EXPS);
#pragma unroll
      for (int i = 0; i < 8; ++i)
        __builtin_amdgcn_raw_buffer_store_b128(v[i], rs, (MG / 128 + wpan + (u32)(i >> 1)) * EXPS + wrow * 128u + (u32)(i & 1) * 1024u + lane * 16u, 0, AUX_SYS);
#pragma unroll
      for (int i = 0; i < 2; ++i)
        __builtin_amdgcn_raw_buffer_store_b128(v[8 + i], rs, (e >> 2) * EXPS + (e & 3u) * 16384u + wave * 2048u + (u32)i * 1024u + lane * 16u, 0, AUX_SYS);
      VMCNT(0);
      RAW_BARRIER();
      if (threadIdx.x == 0) __hip_atomic_fetch_add((GLB_AS u32*)(myws + OFF_XC), 1u, __ATOMIC_RELAXED, __HIP_MEMORY_SCOPE_AGENT);
    }
    call = rfl(ctr_v) / (u32)GRID;
  }
  gemm_role(P, smem, call + 1u, OFF_DATA + (call & 1u) * PAR_BYTES, t0);
  if (threadIdx.x == 0)
    __hip_atomic_fetch_add((GLB_AS u32*)(P.wsd[0] + OFF_CTR), 1u, __ATOMIC_RELAXED, __HIP_MEMORY_SCOPE_AGENT);
}

struct Tune { int nrel, margin_p, ownx_after, own_delay, relay_start, margin_o, go_step, cq, exp_delay, exp_spread, hurry; };

void build_schedule(Params& P, const Tune& t) {
  const int ndir = NB - t.nrel;
  P.ndir = ndir;
  P.margin[0] = t.margin_p;
  P.margin[1] = t.margin_o;
  P.go_step = t.go_step;
  P.hurry = t.hurry;
  P.exp_delay = t.exp_delay;
  P.exp_spread = t.exp_spread * 256 / NEXW;
  for (int cls = 0; cls < 3; ++cls) {
    unsigned char L[MAXU] = {};
    int n = 0, nw = 0;
    bool ownx = false;
    auto put = [&](u32 kind, int idx) { if (n < MAXU) L[n++] = (unsigned char)((kind << 6) | (u32)idx); };
    auto putw = [&](int b) {
      if (!ownx && nw >= t.ownx_after) { for (int a = cls ? 14 : 12; a < 16; ++a) put(K_X, a); ownx = true; }
      put(K_W, b); ++nw;
    };
    if (cls == 0) {
      for (int b = 0; b < ndir; ++b) putw(b);
      for (int key = -3 * t.cq; key <= 4 * (t.nrel - 1); ++key)
        for (int j = 3; j >= 0; --j) {
          const int k4 = key + t.cq * j;
          if (k4 < 0 || (k4 & 3) != 0 || k4 / 4 >= t.nrel) continue;
          if (j == 3) putw(ndir + k4 / 4); else put(K_FWD, 3 * (k4 / 4) + 2 - j);
        }
    } else {
      const int par = cls - 1, odd = t.nrel & 1;
      const int nr = t.nrel / 2;
      const int lead = t.relay_start < ndir ? t.relay_start : ndir;
      if (odd) putw(ndir);
      for (int b = 0; b < lead; ++b) putw(b);
      const int rest = ndir - lead, tot = rest + nr;
      for (int j = 0; j < tot; ++j) {
        const int r0 = j * nr / tot, r1 = (j + 1) * nr / tot;
        if (r1 > r0) putw(ndir + 2 * r0 + (odd && par == 0 ? 2 : par)); else putw(lead + j - r0);
      }
    }
    if (!ownx) for (int a = cls ? 14 : 12; a < 16; ++a) put(K_X, a);
    if (cls != 0) for (int o = 0; o < 2 * NB; ++o) put(K_OUT, o);
    P.nunits[cls] = n;
    for (int i = 0; i < MAXU / 4; ++i)
      P.ord[cls][i] = (u32)L[4 * i] | (u32)L[4 * i + 1] << 8 | (u32)L[4 * i + 2] << 16 | (u32)L[4 * i + 3] << 24;
  }
  for (int xp = 0; xp < 2; ++xp) {
    unsigned char T[2 * NB];
    int n = 0;
    for (int key = 0; key < NB + t.own_delay; ++key) {
      if (key < NB) {
        T[n++] = (unsigned char)key;
        if (((key ^ xp) & 1) == 0) T[n++] = (unsigned char)(32 | key);
      }
      const int ob = key - t.own_delay;
      if (ob >= 0 && ob < NB && ((ob ^ xp) & 1) != 0) T[n++] = (unsigned char)(32 | ob);
    }
    for (int i = 0; i < 16; ++i)
      P.tiles[xp][i] = (u32)T[4 * i] | (u32)T[4 * i + 1] << 8 | (u32)T[4 * i + 2] << 16 | (u32)T[4 * i + 3] << 24;
  }
}

}

extern "C" void kernel_launch(int rank, int nranks, ncclComm_t comm,
                              hipStream_t stream,
                              void* const* d_in, const int* in_sizes, int n_in,
                              void* d_out, int out_size,
                              const int* in_dtypes, int out_dtype,
                              void* const* d_out_all,
                              void* const* d_ws_all, size_t ws_bytes) {
  (void)comm; (void)in_dtypes; (void)out_dtype; (void)d_out_all;
  if (nranks != NR || n_in != 4 || rank < 0 || rank >= NR || ws_bytes < (size_t)WS_NEED ||
      in_sizes[0] != (int)X_BYTES || in_sizes[1] != (int)W_BYTES || out_size != (int)(OUT_BYTES / 4)) {
    fprintf(stderr, "kernel_launch: unexpected problem shape\n");
    return;
  }
  static bool attr_set[NR] = {};
  if (!attr_set[rank]) {
    HIP_CHECK(hipFuncSetAttribute((const void*)fused_gemm_rs, hipFuncAttributeMaxDynamicSharedMemorySize, LDS_BYTES));
    attr_set[rank] = true;
  }
  Params P;
  P.x = (const char*)d_in[0];
  P.w = (const char*)d_in[1];
  P.sx = (const float*)d_in[2];
  P.sw = (const float*)d_in[3];
  for (int i = 0; i < NR; ++i) P.wsd[i] = (char*)d_ws_all[rank ^ i];
  P.out = (char*)d_out;
  P.rank = rank;
  Tune t = {15, 2, 40, 32, 4, 2, 13, 3, 0, 300, 8};
  build_schedule(P, t);
  hipLaunchKernelGGL(fused_gemm_rs, dim3(GRID), dim3(THREADS), LDS_BYTES, stream, P);
  HIP_CHECK(hipGetLastError());
}
```

```cpp
#include <hip/hip_runtime.h>
#include <rccl/rccl.h>
#include <cstdio>
#include <cstdint>

#define HIP_CHECK(expr) do { hipError_t _e = (expr); if (_e != hipSuccess) { \
    fprintf(stderr, "HIP error %s at %s:%d\n", hipGetErrorString(_e), __FILE__, __LINE__); \
    return; } } while (0)

namespace {

typedef unsigned u32;
typedef unsigned long long u64;
typedef unsigned u4 __attribute__((ext_vector_type(4)));
typedef int v2i __attribute__((ext_vector_type(2)));
typedef int v4i __attribute__((ext_vector_type(4)));
typedef int v8i __attribute__((ext_vector_type(8)));
typedef float v4f __attribute__((ext_vector_type(4)));
#define LDS_AS __attribute__((address_space(3)))
#define GLB_AS __attribute__((address_space(1)))

constexpr int NR = 8;
constexpr int MG = 4096, KG = 4096, NG = 16384;
constexpr int KS = KG / NR;
constexpr u32 PANEL = 128 * KS;
constexpr int NA = 16;
constexpr int NB = 32;
constexpr int WPL = 8;
constexpr int NCOMM = 7 * WPL;
constexpr int GRID = 256;
constexpr int NGEMM = GRID - NCOMM;
constexpr int GPX = NGEMM / 8;

constexpr int THREADS = 512;
constexpr int NWK = 6;
constexpr int GO_STEP = 15;
constexpr int PDEPTH = 3;
constexpr int MAXU = 128;
constexpr int NSLOT = 512;

constexpr u32 OFF_CTR = 0, OFF_TMO = 128;
constexpr u32 OFF_XC = 256;
constexpr u32 OFF_PL = 512;
constexpr u32 OFF_TK = 1024;
constexpr u32 OFF_FX = 4096;
constexpr u32 OFF_FW = 8192;
constexpr u32 OFF_FR = 12288;
constexpr u32 OFF_GO = 16384;
constexpr u32 OFF_FI = 32768;
constexpr u32 TILES = 12 * NB * 65536;
constexpr u32 OFF_PRE = 1 << 20;
constexpr u32 OFF_DATA = OFF_PRE + TILES;
constexpr u32 SRC_BYTES = (NA + NB) * PANEL;
constexpr u32 REL_XG = 0;
constexpr u32 REL_WG = NA * PANEL;
constexpr u32 REL_ST = 8 * SRC_BYTES;
constexpr u32 REL_EXP = REL_ST + TILES;
constexpr int NEXP = MG / 128 + NG / 128;
constexpr int NEXW = 128;
constexpr u32 EXPS = PANEL + 2048;
constexpr u32 PAR_BYTES = REL_EXP + NEXP * EXPS;
constexpr u32 WS_NEED = OFF_DATA + 2 * PAR_BYTES;

constexpr u32 X_BYTES = MG * KS;
constexpr u32 W_BYTES = KS * NG;
constexpr u32 OUT_BYTES = (MG / NR) * NG * 4;

constexpr int STAGE = 32768, NST = 4;
constexpr int LDS_CTRL = NST * STAGE;
constexpr int LDS_BYTES = LDS_CTRL + 256;

constexpr u64 DEADLINE = 50000000ull;

constexpr int AUX_SYS = 17;

struct Params {
  const char* x;
  const char* w;
  char* wsd[NR];
  int rank;
  int ndir;
  int npool;
  int nfirst[3];
  int hurry;
  int nunits[3];
  int margin[2];
  int exp_spread;
  const float* sx;
  const float* sw;
  char* out;
  u32 ord[3][MAXU / 4];
  u32 tiles[2][16];
};

__device__ __forceinline__ u32 rfl(u32 v) { return __builtin_amdgcn_readfirstlane(v); }
__device__ __forceinline__ __amdgpu_buffer_rsrc_t mk_rsrc(const void* p, u32 bytes) {
  return __builtin_amdgcn_make_buffer_rsrc((void*)p, (short)0, (int)bytes, 0x00020000);
}
__device__ __forceinline__ u32 ld_sys(const char* p) {
  return __hip_atomic_load((const GLB_AS u32*)p, __ATOMIC_RELAXED, __HIP_MEMORY_SCOPE_SYSTEM);
}
__device__ __forceinline__ void st_sys(char* p, u32 v) {
  __hip_atomic_store((GLB_AS u32*)p, v, __ATOMIC_RELAXED, __HIP_MEMORY_SCOPE_SYSTEM);
}
__device__ __forceinline__ u32 ld_dev(const char* p) {
  return __hip_atomic_load((const GLB_AS u32*)p, __ATOMIC_RELAXED, __HIP_MEMORY_SCOPE_AGENT);
}
__device__ __forceinline__ void st_dev(char* p, u32 v) {
  __hip_atomic_store((GLB_AS u32*)p, v, __ATOMIC_RELAXED, __HIP_MEMORY_SCOPE_AGENT);
}
#define VMCNT(n) asm volatile("s_waitcnt vmcnt(" #n ")" ::: "memory")
#define SCHED_FENCE() do { asm volatile("" ::: "memory"); __builtin_amdgcn_sched_barrier(0); } while (0)
#define RAW_BARRIER() do { asm volatile("s_waitcnt lgkmcnt(0)" ::: "memory"); __builtin_amdgcn_s_barrier(); asm volatile("" ::: "memory"); } while (0)

__device__ __forceinline__ bool give_up(char* myws, u32 seq, u64 t0, u32& round) {
  if ((++round & 7u) != 0u) return false;
  const u32 t = rfl(ld_dev(myws + OFF_TMO));
  const bool late = (__builtin_amdgcn_s_memrealtime() - t0) > DEADLINE;
  if (t == seq) return true;
  if (late) {
    st_dev(myws + OFF_TMO, seq);
    return true;
  }
  return false;
}

__device__ __forceinline__ u32 call_number(const char* myws, char* smem) {
  volatile LDS_AS u32* slot = (volatile LDS_AS u32*)(smem + LDS_CTRL + 128);
  if (rfl(threadIdx.x >> 6) == 7u) *slot = ld_dev(myws + OFF_CTR);
  RAW_BARRIER();
  return rfl(*slot) / (u32)GRID;
}

__device__ __forceinline__ v4f silu4(v4f y) {
  v4f o;
#pragma unroll
  for (int j = 0; j < 4; ++j) o[j] = __builtin_amdgcn_exp2f(__builtin_amdgcn_fmed3f(y[j], -60.f, 60.f) * -1.44269504088896f);
  o = o + 1.f;
#pragma unroll
  for (int j = 0; j < 4; ++j) o[j] = __builtin_amdgcn_rcpf(o[j]);
  return o * y;
}

__device__ __forceinline__ int grow_of(int p, int a) {
  if (a < 12) return ((p ^ (2 + (a >> 1))) << 2) + ((p & 1) << 1) + (a & 1);
  return (p << 2) + (a - 12);
}

constexpr u32 K_X = 0, K_W = 1, K_FWD = 2, K_OUT = 3;
constexpr u32 D_END = 1, D_IN = 2, D_INR = 3, D_OUT = 4, D_OUTH = 5;

__device__ __forceinline__ void comm_role(const Params& P, char* smem, u64 t0) {
  const int tid = threadIdx.x, lane = tid & 63;
  const int wave = rfl(tid >> 6);
  const int w = (int)blockIdx.x / 7;
  const int d = (int)blockIdx.x - 7 * w + 1;
  const int r = P.rank;
  char* myws = P.wsd[0];
  char* pws = P.wsd[d];
  const u32 p = (u32)(r ^ d) & 7u;
  u32 round = 0;

  volatile LDS_AS u4* desc = (volatile LDS_AS u4*)smem;
  volatile LDS_AS u32* cnt = (volatile LDS_AS u32*)(smem + NSLOT * 16);
  volatile LDS_AS u32* gate = cnt + NSLOT - 2;
  volatile LDS_AS u32* cctr = cnt + NSLOT - 3;
  static_assert(NSLOT == THREADS, "descriptor init");
  desc[tid] = u4{0u, 0u, 0u, 0u};
  cnt[tid] = tid == NSLOT - 3 ? 2u * NWK : 0u;
  const u32 call = call_number(myws, smem);
  const u32 seq = call + 1u, parbase = OFF_DATA + (call & 1u) * PAR_BYTES;

  if (__builtin_expect(wave == 7, 0)) {
    char* partner = P.wsd[1];
    bool ok = true;
    while (__ballot(ld_sys(myws + OFF_GO + (u32)(d * WPL + w) * 256u + (u32)(lane & 1) * 4u) < seq) != 0ull) {
      if (give_up(myws, seq, t0, round)) { ok = false; break; }
      __builtin_amdgcn_s_sleep(4);
    }
    *gate = ok ? 1u : 2u;
    for (int t = 0;; ++t) {
      u32 k;
      for (;;) {
        k = rfl(((volatile LDS_AS u32*)(desc + t))[3]);
        if (k != 0u && ((k >> 8) == 0u || !ok || rfl(cnt[t]) == 8u)) break;
        __builtin_amdgcn_s_sleep(1);
      }
      if (k == D_END) break;
      if (!ok) continue;
      if ((k >> 8) != 0u) st_sys(myws + (k >> 8), seq);
      if ((k & 255u) == D_INR) st_sys(partner + ((k >> 8) & ~16u) + (OFF_FR - OFF_FW), seq);
    }
    if (lane == 0) __hip_atomic_fetch_add((GLB_AS u32*)(myws + OFF_CTR), 1u, __ATOMIC_RELAXED, __HIP_MEMORY_SCOPE_AGENT);
  } else if (__builtin_expect(wave == 6, 0)) {
    const int cls = d == 1 ? 0 : 1 + (d & 1);
    const int NP = 4 * P.nunits[cls];
    const int NDIR = P.ndir;
    const __amdgpu_buffer_rsrc_t rs_my = mk_rsrc(myws, WS_NEED);
    u32 woff = OFF_FI + (u32)d * 1024u + (u32)lane * 16u;
    if (d == 1) {
      const int fi = lane < 3 * (NB - NDIR) ? lane : 0;
      const int k = fi / 3, i = fi - 3 * k;
      woff = OFF_FR + (u32)(((NDIR + k) * 8 + (((int)p ^ (2 * i + 2)) & 6)) * 16);
    }
    const u32 tab = P.ord[cls][lane & 31];
    u64 rdy = 0;
    auto poll = [&]() {
      const u4 f = __builtin_amdgcn_raw_buffer_load_b128(rs_my, woff, 0, AUX_SYS);
      rdy |= __ballot(f.x >= seq && f.y >= seq && f.z >= seq && f.w >= seq);
    };
    auto post_end = [&](int s) {
      if (lane < 4) ((volatile LDS_AS u32*)(desc + s + lane))[3] = D_END;
    };
    const u32 tkbase = call * (u32)(NP + WPL);
    char* tkptr = myws + OFF_TK + (u32)d * 128u;
    const int margin = P.margin[d != 1];
    const int NPL = P.npool, NA1 = P.nfirst[cls];
    char* plptr = myws + OFF_PL + (call & 1u) * 32u;
    if (w == 0) st_dev(myws + OFF_PL + ((call & 1u) ^ 1u) * 32u + (u32)d * 4u, 0u);
    bool dead = false, pool = NPL > 0;
    int s = 0, held = -1;
    for (;;) {
      while (!dead && (int)rfl(*cctr) + (pool ? margin >> 4 : margin & 15) <= 8 * s) {
        if (rfl(*gate) == 2u) { dead = true; post_end(s); }
        __builtin_amdgcn_s_sleep(1);
      }
      int ds = d;
      if (held >= 0 && d == 1 && !dead) {
        const u32 pv = ld_dev(plptr + (lane & 7) * 4);
        const int done = (int)(ld_dev(myws + OFF_TK + (lane & 7) * 128) - call * (u32)(4 * P.nunits[1] + WPL));
        const int left = NPL - (int)(pv & 0xffffu) - (int)(pv >> 16);
        const int sc = left > 0 ? (left + max(P.nfirst[1] - done, 0)) * 2 + ((lane * 5 + w * 3 + s) & 7) : 0;
        int best = 0;
        for (int i = 2; i < 8; ++i) {
          const int v = __builtin_amdgcn_readlane(sc, i);
          if (v > best) { best = v; ds = i; }
        }
        if (ds == 1) pool = false;
      }
      int tk = held, idx = 0;
      u32 kind = K_W, piece = 0;
      if (held < 0 || (pool && !dead)) {
        const u4 f = __builtin_amdgcn_raw_buffer_load_b128(rs_my, woff, 0, AUX_SYS);
        u32 old = 0;
        if (lane == 0)
          old = __hip_atomic_fetch_add((GLB_AS u32*)(held < 0 ? tkptr : plptr + ds * 4), held >= 0 && d == 1 ? 0x10000u : 1u,
                                       __ATOMIC_RELAXED, __HIP_MEMORY_SCOPE_AGENT);
        old = rfl(old);
        rdy |= __ballot(f.x >= seq && f.y >= seq && f.z >= seq && f.w >= seq);
        if (held < 0) {
          tk = (int)(old - tkbase);
          if ((u32)tk >= (u32)NP) break;
          if (pool && tk >= NA1) { held = tk; continue; }
        } else if ((int)(old & 0xffffu) + (int)(old >> 16) < NPL) {
          const int u = d == 1 ? NPL - 1 - (int)(old >> 16) : (int)(old & 0xffffu);
          const int k = 2 * (u >> 2) + ((ds & 1) ^ 1);
          piece = (u32)u & 3u;
          if (d == 1) { kind = K_FWD; idx = 3 * k + (ds >> 1) - 1; } else idx = NDIR + k;
          tk = -1;
        } else if (d == 1) {
          continue;
        }
      }
      if (tk >= 0) {
        if (tk == held) { held = -1; pool = false; }
        const int pos = tk >> 2;
        const u32 ent = ((u32)__builtin_amdgcn_readlane((int)tab, pos >> 2) >> (8 * (pos & 3))) & 255u;
        piece = (u32)tk & 3u;
        kind = ent >> 6;
        idx = (int)(ent & 63u);
      }
      if (dead) continue;
      u64 sa, da;
      u32 k3;
      if (kind == K_X) {
        sa = (u64)pws + parbase + REL_EXP + (u32)grow_of(r, idx) * EXPS;
        da = (u64)myws + parbase + REL_XG + p * SRC_BYTES + (u32)idx * PANEL;
        k3 = (OFF_FX + ((u32)idx * 8u + p) * 16u + piece * 4u) << 8 | D_IN;
      } else if (kind == K_W) {
        sa = (u64)pws + parbase + REL_EXP + (u32)(MG / 128 + (r >> 1) * NB + idx) * EXPS;
        da = (u64)myws + parbase + REL_WG + p * SRC_BYTES + (u32)idx * PANEL;
        k3 = (OFF_FW + ((u32)idx * 8u + p) * 16u + piece * 4u) << 8 | (d != 1 && idx >= NDIR && ((d ^ (idx - NDIR)) & 1) == 0 ? D_INR : D_IN);
      } else if (kind == K_FWD) {
        const int kk = idx / 3, i = idx - 3 * kk;
        const u32 q = p ^ (u32)(2 * i + 2 + (kk & 1)), b = (u32)(NDIR + kk);
        sa = (u64)pws + parbase + REL_WG + q * SRC_BYTES + b * PANEL;
        da = (u64)myws + parbase + REL_WG + q * SRC_BYTES + b * PANEL;
        k3 = (OFF_FW + (b * 8u + q) * 16u + piece * 4u) << 8 | D_IN;
      } else {
        const int b = idx >> 1, tt = idx & 1;
        sa = (u64)pws + parbase + REL_ST + (u32)(((d - 2) * 2 + tt) * NB + b) * 65536u;
        da = (u64)P.out + ((2u * (p & 1u) + (u32)tt) * 128u + piece * 32u) * (NG * 4u) + ((p >> 1) * NB + (u32)b) * 512u - piece * 16384u;
        k3 = tk >= NP - P.hurry || !((rdy >> idx) & 1ull) ? D_OUTH : D_OUT;
      }
      sa += piece * 16384u;
      da += piece * 16384u;
      if (kind >= K_FWD && !((rdy >> idx) & 1ull)) {
        for (;;) {
          poll();
          if ((rdy >> idx) & 1ull) break;
          if (give_up(myws, seq, t0, round)) { dead = true; break; }
          __builtin_amdgcn_s_sleep(2);
        }
      }
      if (dead) {
        if (kind == K_OUT) st_sys((char*)da, 0x7fc00000u);
        post_end(s);
        continue;
      }
      asm volatile("" ::: "memory");
      if (lane == 0) {
        volatile LDS_AS u32* dp = (volatile LDS_AS u32*)(desc + s);
        dp[0] = (u32)sa; dp[1] = (u32)da; dp[2] = (u32)(sa >> 32) | (u32)(da >> 32) << 16; dp[3] = k3;
      }
      ++s;
    }
    if (!dead) post_end(s);
  } else {
    const u32 v_lin = (u32)lane * 16u;
    const u32 v_out = (u32)(lane >> 5) * (NG * 4u) + (u32)(lane & 31) * 16u;
    const u32 v_gat = (u32)((lane & 31) >> 2) * 1024u + (u32)(lane >> 5) * 64u + (u32)(lane & 3) * 16u;
    const u32 cnt0 = (u32)(uintptr_t)cnt;
    struct D { u64 s0, s1, d0, d1; u32 kind, vs, vd; };
    auto fetch = [=](u32 j) -> D {
      const u4 e = desc[j >> 3];
      const u32 sub = j & 7u, h = rfl(e.z);
      D c;
      c.kind = rfl(e.w) & 255u;
      const bool iso = c.kind >= D_OUT;
      c.s0 = ((u64)(h & 0xffffu) << 32 | rfl(e.x)) + (iso ? (sub >> 2) * 8192u + (sub & 3u) * 256u : sub * 2048u);
      c.s1 = c.s0 + (iso ? 128u : 1024u);
      c.d0 = ((u64)(h >> 16) << 32 | rfl(e.y)) + (iso ? sub * (4u * NG * 4u) : sub * 2048u);
      c.d1 = c.d0 + (iso ? 2u * NG * 4u : 1024u);
      c.vs = c.vd = v_lin;
      if (iso) { c.vs = v_gat; c.vd = v_out; }
      return c;
    };
    auto take = [=]() -> u32 {
      u64 keep;
      u32 v;
      asm volatile("s_mov_b64 %0, exec\n\ts_mov_b64 exec, 1\n\tds_add_rtn_u32 %1, %2, %3\n\ts_mov_b64 exec, %0\n\ts_waitcnt lgkmcnt(0)"
                   : "=&s"(keep), "=&v"(v) : "v"(cnt0 + 4u * (NSLOT - 3)), "v"(1u) : "memory");
      return rfl(v);
    };
    auto arrive = [&](u32 j) {
      u64 keep;
      asm volatile("s_mov_b64 %0, exec\n\ts_mov_b64 exec, 1\n\tds_add_u32 %1, %2\n\ts_mov_b64 exec, %0"
                   : "=&s"(keep) : "v"(cnt0 + 4u * (j >> 3)), "v"(1u) : "memory");
    };
    auto u64s = [](u64 v) { return (u64)rfl((u32)(v >> 32)) << 32 | rfl((u32)v); };
    auto step = [&](u4& in, u64 sa, u32 vs, const u4& out, u64 da, u32 vd) {
      asm volatile("s_nop 4\n\ts_waitcnt vmcnt(1)\n\tglobal_load_dwordx4 %0, %1, %2 sc0 sc1\n\t"
                   "global_store_dwordx4 %3, %4, %5 sc0 sc1\n\ts_nop 1"
                   : "+v"(in) : "v"(vs), "s"(u64s(sa)), "v"(vd), "v"(out), "s"(u64s(da)) : "memory");
    };
    auto step2 = [&](u4& ina, u4& inb, u64 sa, u64 sb, u32 vs, u4& ra, u4& rb, u32 early) {
      asm volatile("s_nop 4\n\ts_cmp_lg_u32 %7, 0\n\ts_cbranch_scc1 1f\n\ts_waitcnt vmcnt(1)\n1:\n\t"
                   "global_load_dwordx4 %0, %4, %5 sc0 sc1\n\tglobal_load_dwordx4 %1, %4, %6 sc0 sc1\n\ts_waitcnt vmcnt(2)"
                   : "+v"(ina), "+v"(inb), "+v"(ra), "+v"(rb) : "v"(vs), "s"(u64s(sa)), "s"(u64s(sb)), "s"(rfl(early)) : "scc", "memory");
    };
    const float scale = P.sx[0] * P.sw[0];
    auto finish = [&](const u4& ra, const u4& rb, u64 da, u32 vd) {
      const v4f o = silu4((__builtin_bit_cast(v4f, ra) + __builtin_bit_cast(v4f, rb)) * scale);
      asm volatile("s_nop 4\n\tglobal_store_dwordx4 %0, %1, %2 sc0 sc1\n\ts_nop 1" :: "v"(vd), "v"(o), "s"(u64s(da)) : "memory");
    };
    const u64 to_pre = (u64)myws + OFF_PRE - ((u64)pws + parbase + REL_ST);
    constexpr u32 NONE = 0xffffffffu;
    u32 ja = (u32)wave, jb = (u32)wave + NWK, owed = NONE;
    u4 a0 = u4{0u, 0u, 0u, 0u}, a1 = a0, b0 = a0, b1 = a0;
    u32 go;
    while ((go = rfl(*gate)) == 0u) __builtin_amdgcn_s_sleep(1);
    while (go == 1u) {
      D C = fetch(ja);
      while (C.kind == 0u) { __builtin_amdgcn_s_sleep(1); C = fetch(ja); }
      if (C.kind == D_END) break;
      asm volatile("s_nop 4\n\tglobal_load_dwordx4 %0, %1, %2 sc0 sc1" : "+v"(a0) : "v"(C.vs), "s"(u64s(C.s0)) : "memory");
      D N = fetch(jb);
      bool nxt = true;
      while (C.kind < D_OUT) {
        VMCNT(0);
        step(a1, C.s1, C.vs, a0, C.d0, C.vd);
        if (owed != NONE) arrive(owed);
        owed = ja;
        const u32 jc = take();
        const D N2 = fetch(jc);
        if (__builtin_expect(N.kind == 0u, 0)) { VMCNT(1); N = fetch(jb); }
        nxt = N.kind >= D_IN;
        step(a0, nxt ? N.s0 : C.d0, nxt ? N.vs : C.vd, a1, C.d1, C.vd);
        ja = jb;
        jb = jc;
        if (!nxt) break;
        C = N;
        N = N2;
      }
      if (nxt) asm volatile("s_nop 4\n\tglobal_load_dwordx4 %0, %1, %2 sc0 sc1" : "+v"(b0) : "v"(C.vs), "s"(u64s(C.s0 + to_pre)) : "memory");
      if (owed != NONE || C.kind != D_OUTH) VMCNT(0);
      if (owed != NONE) { arrive(owed); owed = NONE; }
      while (nxt) {
        step2(a1, b1, C.s1, C.s1 + to_pre, v_gat, a0, b0, C.kind - D_OUT);
        finish(a0, b0, C.d0, v_out);
        const u32 jc = take();
        const D N2 = fetch(jc);
        if (__builtin_expect(N.kind == 0u, 0)) { VMCNT(1); N = fetch(jb); }
        nxt = N.kind >= D_OUT;
        const u64 sb = (nxt ? N.s0 : C.s0) + to_pre;
        step2(a0, b0, nxt ? N.s0 : sb, sb, v_gat, a1, b1, 0u);
        finish(a1, b1, C.d1, v_out);
        ja = jb;
        jb = jc;
        C = N;
        N = N2;
        if (!nxt) VMCNT(0);
      }
    }
  }
}

__device__ __forceinline__ void glds16(__amdgpu_buffer_rsrc_t rs, u32 lds, u32 voff, u32 soff) {
  asm volatile("s_mov_b32 m0, %0\n\ts_nop 3\n\tbuffer_load_dwordx4 %1, %2, %3 offen lds"
               :: "s"(rfl(lds)), "v"(voff), "s"(rs), "s"(rfl(soff)) : "m0", "memory");
}

__device__ __forceinline__ void gemm_role(const Params& P, char* smem, u32 seq, u32 parbase, u64 t0) {
  const int tid = threadIdx.x, lane = tid & 63;
  const int wave = rfl(tid >> 6);
  const int r = P.rank;
  char* myws = P.wsd[0];
  const int xg = (int)blockIdx.x & 7, kx = ((int)blockIdx.x - NCOMM) >> 3;
  constexpr int gpx = GPX;
  const u32 lds0 = (u32)(uintptr_t)(LDS_AS char*)smem;

  const __amdgpu_buffer_rsrc_t rs_x = mk_rsrc(P.x, X_BYTES);
  const __amdgpu_buffer_rsrc_t rs_w = mk_rsrc(P.w, W_BYTES);
  const __amdgpu_buffer_rsrc_t rs_ws = mk_rsrc(myws, WS_NEED);
  const float scale = P.sx[0] * P.sw[0];

  u32 vX[2], vWs[2], vWo[2];
#pragma unroll
  for (int u = 0; u < 2; ++u) {
    const int row = wave * 16 + u * 8 + (lane >> 3), slot = lane & 7;
    const int cx = slot ^ ((row >> 1) & 7);
    const int cw = slot ^ (((row >> 1) & 3) | (((row >> 4) & 1) << 2));
    vX[u] = (u32)row * KS + (u32)cx * 16u;
    vWs[u] = (u32)row * 128u + (u32)cw * 16u;
    vWo[u] = (u32)row * NG + (u32)cw * 16u;
  }
  const int wm = wave >> 2, wn = wave & 3, g = lane >> 4, l15 = lane & 15;
  const int aX0 = (wm * 64 + l15) * 128 + ((g ^ (l15 >> 1)) * 16);
  const int aX1 = aX0 ^ 64;
  const int fw = ((l15 >> 2) & 3) | ((g & 1) << 2);
  const int aW0 = 16384 + (16 * g + (l15 >> 1)) * 128 + (((2 * wn) ^ fw) * 16) + (l15 & 1) * 8;
  const int aW1 = aW0 ^ 16;
  volatile LDS_AS u32* ctrl = (volatile LDS_AS u32*)(smem + LDS_CTRL);
  u32 round = 0;

  for (int it = kx >= 7 && kx < 23 ? -3 : 0; kx + it * gpx < 2 * NB; ++it) {
    const bool pre = it < 0;
    const int e = pre ? 0 : kx + it * gpx;
    const u32 ent = (P.tiles[xg & 1][e >> 2] >> (8 * (e & 3))) & 255u;
    int b = (int)(ent & 31u);
    int a = (ent >> 5) == 0u ? xg : ((((b ^ xg) & 1) == 0 ? 8 : 12) + (xg >> 1));
    u32 sX_own = (u32)grow_of(r, a) * PANEL;
    u32 sW_own = (u32)((r >> 1) * NB + b) * 128u;
    if (pre) {
      const int u = (it + 3) * 16 + kx - 7, fc = (u >> 2) * 8 + xg, rt = u & 3;
      b = fc / 3;
      const int pg = (r >> 1) ^ (fc - 3 * b + 1);
      a = ((r ^ (2 * pg + (rt >> 1))) - 2) * 2 + (rt & 1);
      sX_own = (u32)(r * 4 + rt) * PANEL;
      sW_own = (u32)(pg * NB + b) * 128u;
    }
    const bool own = a >= 12;
    const int nsl = pre ? 1 : (own ? 8 : 7);
    const int NDIR = P.ndir;
    u32 pref;
    if (b < NDIR) pref = 0x17654320u;
    else pref = ((b - NDIR) & 1) ? 0x64217530u : 0x75316420u;
    u32 ordd = pre ? 8u : 0u;
    const u32 sX_ws = parbase + REL_XG + (u32)a * PANEL;
    const u32 sW_ws = parbase + REL_WG + (u32)b * PANEL;
    const u32 foff = lane < 32 ? OFF_FX + (u32)a * 128u + (u32)lane * 4u : OFF_FW + (u32)b * 128u + (u32)(lane - 32) * 4u;

    v4f acc[2][4];
#pragma unroll
    for (int nb = 0; nb < 2; ++nb)
#pragma unroll
      for (int mb = 0; mb < 4; ++mb) acc[nb][mb] = v4f{0.f, 0.f, 0.f, 0.f};

    auto issue = [&](int kt, int stage) {
      const int dq = (ordd >> (4 * (kt >> 2))) & 7;
      const u32 q = (u32)(r ^ dq);
      const u32 kb = (u32)(kt & 3);
      const u32 ldsb = lds0 + (u32)stage * STAGE + (u32)wave * 2048u;
      if (dq == 0) {
        const u32 sx = sX_own + kb * 128u, sw = sW_own + kb * 128u * NG;
        glds16(rs_x, ldsb, vX[0], sx);
        glds16(rs_x, ldsb + 1024u, vX[1], sx);
        glds16(rs_w, ldsb + 16384u, vWo[0], sw);
        glds16(rs_w, ldsb + 16384u + 1024u, vWo[1], sw);
      } else {
        const u32 sx = sX_ws + q * SRC_BYTES + kb * 128u, sw = sW_ws + q * SRC_BYTES + kb * 16384u;
        glds16(rs_ws, ldsb, vX[0], sx);
        glds16(rs_ws, ldsb + 1024u, vX[1], sx);
        glds16(rs_ws, ldsb + 16384u, vWs[0], sw);
        glds16(rs_ws, ldsb + 16384u + 1024u, vWs[1], sw);
      }
    };
    auto compute = [&](int stage) {
      const LDS_AS char* sb = (const LDS_AS char*)smem + stage * STAGE;
      v8i xf[4], wf[2];
#pragma unroll
      for (int nb = 0; nb < 2; ++nb) {
        const LDS_AS char* pw = sb + (nb ? aW1 : aW0);
        const v2i t0_ = __builtin_amdgcn_ds_read_tr8_b64_v2i32((LDS_AS v2i*)(pw));
        const v2i t1_ = __builtin_amdgcn_ds_read_tr8_b64_v2i32((LDS_AS v2i*)(pw + 1024));
        const v2i t2_ = __builtin_amdgcn_ds_read_tr8_b64_v2i32((LDS_AS v2i*)(pw + 8192));
        const v2i t3_ = __builtin_amdgcn_ds_read_tr8_b64_v2i32((LDS_AS v2i*)(pw + 9216));
        wf[nb] = v8i{t0_.x, t0_.y, t1_.x, t1_.y, t2_.x, t2_.y, t3_.x, t3_.y};
      }
#pragma unroll
      for (int mb = 0; mb < 4; ++mb) {
        const v4i lo = *(const LDS_AS v4i*)(sb + aX0 + mb * 2048);
        const v4i hi = *(const LDS_AS v4i*)(sb + aX1 + mb * 2048);
        xf[mb] = v8i{lo.x, lo.y, lo.z, lo.w, hi.x, hi.y, hi.z, hi.w};
      }
#pragma unroll
      for (int mb = 0; mb < 4; ++mb)
#pragma unroll
        for (int nb = 0; nb < 2; ++nb)
          acc[nb][mb] = __builtin_amdgcn_mfma_scale_f32_16x16x128_f8f6f4(wf[nb], xf[mb], acc[nb][mb], 1, 0, 0, 0, 0, 0);
    };

    u32 taken = own ? 0u : 4u << (a >> 1);
    int nass = pre ? 1 : 0;
    auto assign = [&](u32 f) {
      const u64 m = __ballot((((lane & 31) >> 2) == r) || f >= seq);
      const u32 m32 = (u32)m & (u32)(m >> 32);
      for (int i = 0; i < 8; ++i) {
        const u32 dq = (pref >> (4 * i)) & 7u;
        if (!((taken >> dq) & 1u) && ((m32 >> (4u * ((u32)r ^ dq))) & 15u) == 15u) {
          taken |= 1u << dq;
          ordd |= (8u | dq) << (4 * nass);
          ++nass;
        }
      }
    };
    auto wait_slices = [&](int n) {
      for (;;) {
        assign(__builtin_amdgcn_raw_buffer_load_b32(rs_ws, foff, 0, AUX_SYS));
        if (nass >= n) break;
        if (give_up(myws, seq, t0, round)) {
          while (nass < nsl) { ordd |= 8u << (4 * nass); ++nass; }
          ctrl[1] = 1u;
          break;
        }
        __builtin_amdgcn_s_sleep(4);
      }
    };
    auto avail = [&](int sl) { return ((ordd >> (4 * sl)) & 8u) != 0u; };

    if (wave == 0) { ctrl[0] = 0u; ctrl[1] = 0u; }
    for (int sl = 0; sl < nsl;) {
      if (wave == 0) {
        if (!pre) {
          wait_slices(sl == 0 ? 2 : sl + 1);
          if (sl == 0) {
            __builtin_amdgcn_fence(__ATOMIC_ACQUIRE, "");
            VMCNT(0);
          }
        }
        ctrl[2] = ordd;
      }
      RAW_BARRIER();
      ordd = rfl(ctrl[2]);
      issue(4 * sl, 0); issue(4 * sl + 1, 1); issue(4 * sl + 2, 2); issue(4 * sl + 3, 3);
      for (bool last = false, fresh = true; !last; ++sl, fresh = false) {
        const int kt = sl * 4;
        if (fresh) VMCNT(12); else VMCNT(8);
        RAW_BARRIER();
        u32 pf;
        const u32 po = (wave == 0 && nass < nsl) ? foff : 0xfffffff0u;
        asm volatile("buffer_load_dword %0, %1, %2, 0 offen sc0 sc1" : "=v"(pf) : "v"(po), "s"(rs_ws) : "memory");
        if (!fresh) issue(kt + 3, 3);
        compute(0);
        last = sl == nsl - 1 || !avail(sl + 1);
        VMCNT(8); RAW_BARRIER();
        if (!last) issue(kt + 4, 0);
        compute(1);
        if (!last) VMCNT(8); else VMCNT(4);
        RAW_BARRIER();
        if (!last) issue(kt + 5, 1);
        else asm volatile("buffer_load_dword %0, %1, %2, 0 offen sc0 sc1" : "+v"(pf) : "v"(po), "s"(rs_ws) : "memory");
        compute(2);
        if (!last) VMCNT(8); else VMCNT(0);
        asm volatile("" : "+v"(pf));
        if (wave == 0 && nass < nsl) { assign(pf); ctrl[0] = ordd; }
        RAW_BARRIER();
        if (!avail(nsl - 1)) ordd |= rfl(ctrl[0]);
        if (last && sl < nsl - 1 && avail(sl + 1)) {
          issue(kt + 4, 0); issue(kt + 5, 1);
          last = false;
        }
        if (!last) issue(kt + 6, 2);
        compute(3);
      }
    }

    {
      const bool bad = rfl(ctrl[1]) != 0u;
      const __amdgpu_buffer_rsrc_t rs_o = mk_rsrc(own ? P.out : myws, own ? OUT_BYTES : WS_NEED);
      const u32 rs = own ? NG * 4u : 64u, rbs = own ? 16u * NG * 4u : 8192u, cbs = own ? 64u : 1024u;
      const u32 so = (own ? (u32)(a - 12) * 128u * (NG * 4u) + (u32)((r >> 1) * NB + b) * 512u
                          : (pre ? OFF_PRE : parbase + REL_ST) + (u32)(a * NB + b) * 65536u) +
                     (u32)wm * 4u * rbs + (u32)wn * 2u * cbs;
      const u32 vo = (u32)l15 * rs + (u32)g * 16u;
      const float sc = bad ? __builtin_nanf("") : scale;
      SCHED_FENCE();
      asm volatile("s_nop 15\n\ts_nop 7" ::: "memory");
#pragma unroll
      for (int mb = 0; mb < 4; ++mb)
#pragma unroll
        for (int nb = 0; nb < 2; ++nb) {
          v4f o = acc[nb][mb];
          if (own) o = silu4(o * sc);
          asm volatile("s_nop 4\n\tbuffer_store_dwordx4 %0, %1, %2, %3 offen sc0 sc1\n\ts_nop 1"
                       :: "v"(o), "v"(vo), "s"(rs_o), "s"(so + (u32)mb * rbs + (u32)nb * cbs) : "memory");
        }
      VMCNT(0);
      RAW_BARRIER();
      if (!own && !bad && tid < 2) {
        const u32 dd = 2u + (u32)(a >> 1);
        st_sys((pre ? myws + 8 : P.wsd[dd]) + OFF_FI + dd * 1024u + (u32)(b * 2 + (a & 1)) * 16u + (u32)tid * 4u, seq);
      }
    }
  }
}

__global__ void __launch_bounds__(THREADS, 2) fused_gemm_rs(Params P) {
  extern __shared__ __attribute__((aligned(16))) char smem[];
  const u32 gi = blockIdx.x - (u32)NCOMM, lane = threadIdx.x & 63u, wave = rfl(threadIdx.x >> 6);
  const u32 d = blockIdx.x % 7u + 1u, w = blockIdx.x / 7u & 7u;
  const u32 r = (u32)P.rank;
  const char* px = P.x;
  char* myws = P.wsd[0];
  char* pws = P.wsd[d];
  const u64 t0 = __builtin_amdgcn_s_memrealtime();
  asm volatile("" :: "s"(r), "s"(px), "s"(myws), "s"(pws), "s"(t0));
  u32 call = 0;
  if (__builtin_expect(gi < (u32)NCOMM, 1)) {
    const u32 ctr_v = ld_dev(myws + OFF_CTR);
    if (wave < 6u) {
      const u32 tk = w * 6u + wave, piece = tk & 3u;
      const u32 a = (tk >> 3) + 2u == d ? 12u + ((tk >> 2) & 1u) : tk >> 2;
      const __amdgpu_buffer_rsrc_t rs = mk_rsrc(px + (u32)grow_of((int)(r ^ d), (int)a) * PANEL + piece * 16384u, 16384u);
      const u32 rot = tk + 5u * d;
      u4 v[16];
#pragma unroll
      for (int i = 0; i < PDEPTH; ++i) v[i] = __builtin_amdgcn_raw_buffer_load_b128(rs, lane * 16u, ((rot + (u32)i) & 15u) * 1024u, 0);
      SCHED_FENCE();
      const u32 seq = rfl(ctr_v) / (u32)GRID + 1u;
      const __amdgpu_buffer_rsrc_t rd = mk_rsrc(pws + OFF_DATA + (~seq & 1u) * PAR_BYTES + REL_XG + r * SRC_BYTES + a * PANEL + piece * 16384u, 16384u);
      char* go = pws + OFF_GO + (d * WPL + w) * 256u;
      u32 xc = 0;
      bool sent = wave != 0u;
#pragma unroll
      for (int i = 0; i < 16; ++i) {
        if (i != 0) { if (i - 1 + PDEPTH < 16) VMCNT(1); else VMCNT(0); }
        __builtin_amdgcn_raw_buffer_store_b128(v[i], rd, lane * 16u, ((rot + (u32)i) & 15u) * 1024u, AUX_SYS);
        asm volatile("s_nop 1" ::: "memory");
        if (i == GO_STEP - 1 && !sent) xc = ld_sys(myws + OFF_XC);
        if (i + PDEPTH < 16) v[i + PDEPTH] = __builtin_amdgcn_raw_buffer_load_b128(rs, lane * 16u, ((rot + (u32)(i + PDEPTH)) & 15u) * 1024u, 0);
        if (i == GO_STEP && wave == 1u) st_sys(myws + OFF_GO + (d * WPL + w) * 256u + 4u, seq);
        if (i == GO_STEP && !sent && rfl(xc) >= seq * (u32)NEXW) { st_sys(go, seq); sent = true; }
        SCHED_FENCE();
      }
      VMCNT(0);
      st_sys(pws + OFF_FX + (a * 8u + r) * 16u + piece * 4u, seq);
      for (u32 round = 0; !sent && !give_up(myws, seq, t0, round);) {
        if (rfl(ld_sys(myws + OFF_XC)) >= seq * (u32)NEXW) { st_sys(go, seq); sent = true; }
        __builtin_amdgcn_s_sleep(16);
      }
    }
    call = rfl(ctr_v) / (u32)GRID;
  } else if (blockIdx.x < (u32)NCOMM) {
    comm_role(P, smem, t0);
    return;
  } else {
    const u32 e = gi - (u32)NCOMM;
    const u32 wrow = (e & 31u) * 16u, wpan = (e >> 5) * 32u + wave * 4u;
    if (e < (u32)NEXW) {
      const u64 tgo = (u64)(e * (u32)P.exp_spread >> 8);
      while (__builtin_amdgcn_s_memrealtime() - t0 < tgo) __builtin_amdgcn_s_sleep(2);
      u4 v[10];
#pragma unroll
      for (int i = 0; i < 8; ++i)
        v[i] = __builtin_amdgcn_raw_buffer_load_b128(mk_rsrc(P.w, W_BYTES), (lane >> 3) * NG + (lane & 7u) * 16u,
                                                     (wrow + (u32)(i & 1) * 8u) * NG + (wpan + (u32)(i >> 1)) * 128u, 0);
#pragma unroll
      for (int i = 0; i < 2; ++i)
        v[8 + i] = __builtin_amdgcn_raw_buffer_load_b128(mk_rsrc(P.x, X_BYTES), lane * 16u + (u32)i * 1024u, e * 16384u + wave * 2048u, 0);
      call = call_number(myws, smem);
      const __amdgpu_buffer_rsrc_t rs = mk_rsrc(myws + OFF_DATA + (call & 1u) * PAR_BYTES + REL_EXP, NEXP * EXPS);
#pragma unroll
      for (int i = 0; i < 8; ++i)
        __builtin_amdgcn_raw_buffer_store_b128(v[i], rs, (MG / 128 + wpan + (u32)(i >> 1)) * EXPS + wrow * 128u + (u32)(i & 1) * 1024u + lane * 16u, 0, AUX_SYS);
#pragma unroll
      for (int i = 0; i < 2; ++i)
        __builtin_amdgcn_raw_buffer_store_b128(v[8 + i], rs, (e >> 2) * EXPS + (e & 3u) * 16384u + wave * 2048u + (u32)i * 1024u + lane * 16u, 0, AUX_SYS);
      VMCNT(0);
      RAW_BARRIER();
      if (threadIdx.x == 0) __hip_atomic_fetch_add((GLB_AS u32*)(myws + OFF_XC), 1u, __ATOMIC_RELAXED, __HIP_MEMORY_SCOPE_AGENT);
    } else {
      call = call_number(myws, smem);
    }
  }
  gemm_role(P, smem, call + 1u, OFF_DATA + (call & 1u) * PAR_BYTES, t0);
  if (threadIdx.x == 0)
    __hip_atomic_fetch_add((GLB_AS u32*)(myws + OFF_CTR), 1u, __ATOMIC_RELAXED, __HIP_MEMORY_SCOPE_AGENT);
}

struct Tune { int ndir, npool, wfirst, own_delay, lead, margin_p, margin_o, cq, exp_spread, hurry; };

void build_schedule(Params& P, const Tune& t) {
  const int ndir = t.ndir, nk = NB - ndir, k1 = 2 * t.npool;
  P.ndir = ndir;
  P.npool = 4 * t.npool;
  P.margin[0] = t.margin_p / 100 * 16 + t.margin_p % 100;
  P.margin[1] = t.margin_o / 100 * 16 + t.margin_o % 100;
  P.hurry = t.hurry;
  P.exp_spread = t.exp_spread * 256 / NEXW;
  for (int cls = 0; cls < 3; ++cls) {
    unsigned char L[MAXU] = {};
    int n = 0;
    auto put = [&](u32 kind, int idx) { if (n < MAXU) L[n++] = (unsigned char)((kind << 6) | (u32)idx); };
    if (cls == 0) {
      P.nfirst[0] = t.wfirst;
      for (int b = 0; b < ndir + k1; ++b) put(K_W, b);
      for (int key = -3 * t.cq; key <= 4 * (nk - k1 - 1); ++key)
        for (int j = 3; j >= 0; --j) {
          const int k4 = key + t.cq * j;
          if (k4 < 0 || (k4 & 3) != 0 || k4 / 4 >= nk - k1) continue;
          if (j == 3) put(K_W, ndir + k1 + k4 / 4); else put(K_FWD, 3 * (k1 + k4 / 4) + 2 - j);
        }
      for (int a = 12; a < 16; ++a) put(K_X, a);
    } else {
      int b = 0, j = t.npool - 1;
      while (b < ndir || j >= 0) {
        if (b < ndir) put(K_W, b++);
        if (b >= t.lead && j >= 0) put(K_W, ndir + 2 * j-- + (cls - 1));
      }
      for (int k = k1 + cls - 1; k < nk; k += 2) put(K_W, ndir + k);
      for (int a = 14; a < 16; ++a) put(K_X, a);
      P.nfirst[cls] = 4 * n;
      for (int o = 0; o < 2 * NB; ++o) put(K_OUT, o);
    }
    P.nunits[cls] = n;
    for (int i = 0; i < MAXU / 4; ++i)
      P.ord[cls][i] = (u32)L[4 * i] | (u32)L[4 * i + 1] << 8 | (u32)L[4 * i + 2] << 16 | (u32)L[4 * i + 3] << 24;
  }
  for (int xp = 0; xp < 2; ++xp) {
    unsigned char T[2 * NB];
    int n = 0;
    for (int key = 0; key < NB + t.own_delay; ++key) {
      if (key < NB) {
        T[n++] = (unsigned char)key;
        if (((key ^ xp) & 1) == 0) T[n++] = (unsigned char)(32 | key);
      }
      const int ob = key - t.own_delay;
      if (ob >= 0 && ob < NB && ((ob ^ xp) & 1) != 0) T[n++] = (unsigned char)(32 | ob);
    }
    for (int i = 0; i < 16; ++i)
      P.tiles[xp][i] = (u32)T[4 * i] | (u32)T[4 * i + 1] << 8 | (u32)T[4 * i + 2] << 16 | (u32)T[4 * i + 3] << 24;
  }
}

}

extern "C" void kernel_launch(int rank, int nranks, ncclComm_t comm,
                              hipStream_t stream,
                              void* const* d_in, const int* in_sizes, int n_in,
                              void* d_out, int out_size,
                              const int* in_dtypes, int out_dtype,
                              void* const* d_out_all,
                              void* const* d_ws_all, size_t ws_bytes) {
  (void)comm; (void)in_dtypes; (void)out_dtype; (void)d_out_all;
  if (nranks != NR || n_in != 4 || rank < 0 || rank >= NR || ws_bytes < (size_t)WS_NEED ||
      in_sizes[0] != (int)X_BYTES || in_sizes[1] != (int)W_BYTES || out_size != (int)(OUT_BYTES / 4)) {
    fprintf(stderr, "kernel_launch: unexpected problem shape\n");
    return;
  }
  static bool attr_set[NR] = {};
  if (!attr_set[rank]) {
    HIP_CHECK(hipFuncSetAttribute((const void*)fused_gemm_rs, hipFuncAttributeMaxDynamicSharedMemorySize, LDS_BYTES));
    attr_set[rank] = true;
  }
  Params P;
  P.x = (const char*)d_in[0];
  P.w = (const char*)d_in[1];
  P.sx = (const float*)d_in[2];
  P.sw = (const float*)d_in[3];
  for (int i = 0; i < NR; ++i) P.wsd[i] = (char*)d_ws_all[rank ^ i];
  P.out = (char*)d_out;
  P.rank = rank;
  Tune t = {12, 5, 48, 32, 2, 1002, 402, 3, 300, 8};
  build_schedule(P, t);
  hipLaunchKernelGGL(fused_gemm_rs, dim3(GRID), dim3(THREADS), LDS_BYTES, stream, P);
  HIP_CHECK(hipGetLastError());
}
```
